```python
import numpy as np

kernel_name = "gpu_gemm_ar_m4096_k4096_n4096_e4m3e5m2_dqsilu"


ABSMAX_RTOL = 0.05

COMPUTE_DTYPE = "float32"
EPILOGUE = "dequant_silu"
IN_QUANT = None
OUT_QUANT = None
IN_FP8 = "e4m3e5m2"
IN_DTYPES = {"x": "e4m3", "w_mat": "e5m2",
             "scale_x": "float32", "scale_w": "float32"}
WORLD_SIZE = 8

def _snap_fp8(x, mant_bits, max_abs, min_normal):
    a = np.abs(x.astype(np.float64))
    _m, _e = np.frexp(a)
    q = np.where(a >= min_normal, np.ldexp(1.0, _e - 1 - mant_bits),
                 min_normal * 2.0 ** -mant_bits)
    snapped = np.minimum(np.round(a / q) * q, max_abs)
    return (np.sign(x) * snapped).astype(np.float32)

def setup_inputs():

    x = _snap_fp8(np.random.randn(4096, 4096).astype(np.float32), 3, 448.0, 2.0 ** -6)
    w_mat = _snap_fp8(np.random.randn(4096, 4096).astype(np.float32), 2, 57344.0, 2.0 ** -14)
    scale_x = (0.02 * (1.0 + np.random.rand(1))).astype(np.float32)
    scale_w = (0.02 * (1.0 + np.random.rand(1))).astype(np.float32)
    return {"x": x, "w_mat": w_mat, "scale_x": scale_x, "scale_w": scale_w}

def reference(x, w_mat, scale_x, scale_w):
    acc = (x @ w_mat).astype(np.float32)
    y = (acc * (scale_x * scale_w)).astype(np.float32)
    return (y / (1.0 + np.exp(-np.clip(y, -60.0, 60.0)))).astype(np.float32)
```

```cpp
#include <hip/hip_runtime.h>
#include <rccl/rccl.h>
#include <cstdio>

#define HIP_CHECK(expr) do { hipError_t _e = (expr); if (_e != hipSuccess) { \
    fprintf(stderr, "HIP error %s at %s:%d\n", hipGetErrorString(_e), __FILE__, __LINE__); \
    return; } } while (0)

#define LAS __attribute__((address_space(3)))
#define GAS __attribute__((address_space(1)))
typedef unsigned u32;
typedef unsigned long long u64;
typedef int v2i __attribute__((ext_vector_type(2)));
typedef int v4i __attribute__((ext_vector_type(4)));
typedef int v8i __attribute__((ext_vector_type(8)));
typedef float v4f __attribute__((ext_vector_type(4)));
typedef unsigned u32x2 __attribute__((ext_vector_type(2)));
typedef unsigned u32x4 __attribute__((ext_vector_type(4)));

namespace {
constexpr int NR = 8;
constexpr int DIM = 4096;
constexpr int KS = DIM / NR;
constexpr int NT = 32;
constexpr size_t STAGE_OFF = 0;
constexpr size_t STAGE_BYTES = (size_t)NT * 2 * NR * DIM * 16;
constexpr size_t OUTBOX_OFF = STAGE_OFF + STAGE_BYTES;
constexpr size_t OUTBOX_HALF = (size_t)2 * 16 * NT * 2 * 2048;
constexpr size_t RDY_OFF = OUTBOX_OFF + 2 * OUTBOX_HALF;
constexpr size_t RDY_HALF = (size_t)NR * 256 * 16;
constexpr size_t DONE_OFF = RDY_OFF + 2 * RDY_HALF;
constexpr size_t CNT_OFF = DONE_OFF + 32 * 128;
constexpr size_t TMO_OFF = CNT_OFF + 256 * 4;
constexpr size_t DBG_OFF = TMO_OFF + 128;
constexpr size_t WS_NEED = DBG_OFF + 256 * 64;
constexpr u64 DEADLINE = 40000000ull;
constexpr int BM = 256, BN = 320, XS = BM * 16, WS = BN * 16, XB = 8 * XS, WB = 8 * WS, STG = XB + WB;
constexpr int LDS_BYTES = 2 * STG + 256;
constexpr int GEMM_SLOTS = 26, PULL_SLOTS = 4;

struct Args {
    const unsigned char* x; const unsigned char* w; const float* sx; const float* sw; float* out;
    char* ws[NR]; int rank; int pad;
};

__device__ __forceinline__ void st16_sys(void* p, u32x4 v) {
    asm volatile("global_store_dwordx4 %0, %1, off sc0 sc1\n\ts_nop 1" ::"v"(p), "v"(v) : "memory");
}
__device__ __forceinline__ void st16_agent(void* p, u32x4 v) {
    asm volatile("global_store_dwordx4 %0, %1, off sc1\n\ts_nop 1" ::"v"(p), "v"(v) : "memory");
}
__device__ __forceinline__ bool give_up(u32* tmo, u32 call, u64 t0) {
    if (__hip_atomic_load(tmo, __ATOMIC_RELAXED, __HIP_MEMORY_SCOPE_AGENT) == call || __builtin_amdgcn_s_memrealtime() - t0 > DEADLINE) {
        __hip_atomic_store(tmo, call, __ATOMIC_RELAXED, __HIP_MEMORY_SCOPE_AGENT);
        return true;
    }
    return false;
}
__device__ __forceinline__ v8i ldfrag(LAS const char* p, int off, int stride) {
    v4i a = *(LAS const v4i*)(p + off), b = *(LAS const v4i*)(p + off + stride);
    return __builtin_shufflevector(a, b, 0, 1, 2, 3, 4, 5, 6, 7);
}
#define SCHED __builtin_amdgcn_sched_barrier(0)
#define STAMP(i) do { } while (0)

__device__ __forceinline__ u32 stage_out(const Args& a, LAS char* lds) {
    const int tid = threadIdx.x, wid = __builtin_amdgcn_readfirstlane(tid >> 6), lane = tid & 63;
    const int b = blockIdx.x, g = b & 7, p = (b >> 3) & 15, o = b >> 7;
    const int tt = wid >> 1, h = wid & 1;
    char* myws = a.ws[a.rank];
    u32* cnt = (u32*)(myws + CNT_OFF) + b;
    const u32 prev = __hip_atomic_load(cnt, __ATOMIC_RELAXED, __HIP_MEMORY_SCOPE_AGENT);
    u32x4 d0, d1;
    if (o == 0) {
        const unsigned char* src = a.x + (size_t)(256 * p + 128 * h + lane) * KS + 64 * g + 16 * tt;
        d0 = *(const u32x4*)src; d1 = *(const u32x4*)(src + 64 * KS);
    } else {
        const unsigned char* src = a.w + (size_t)(64 * g + (tid >> 4)) * DIM + 256 * p + (tid & 15) * 16;
        const u32x4 r0 = *(const u32x4*)src, r1 = *(const u32x4*)(src + (size_t)32 * DIM);
        *(LAS u32x4*)(lds + (tid >> 4) * 288 + (tid & 15) * 16) = r0;
        *(LAS u32x4*)(lds + (32 + (tid >> 4)) * 288 + (tid & 15) * 16) = r1;
        __syncthreads();
        const int base = (16 * tt + ((lane & 15) >> 1)) * 288 + 128 * h + (lane >> 4) * 16 + (lane & 1) * 8;
        const v2i c00 = __builtin_amdgcn_ds_read_tr8_b64_v2i32((LAS v2i*)(lds + base));
        const v2i c01 = __builtin_amdgcn_ds_read_tr8_b64_v2i32((LAS v2i*)(lds + base + 8 * 288));
        const v2i c10 = __builtin_amdgcn_ds_read_tr8_b64_v2i32((LAS v2i*)(lds + base + 64));
        const v2i c11 = __builtin_amdgcn_ds_read_tr8_b64_v2i32((LAS v2i*)(lds + base + 8 * 288 + 64));
        d0 = (u32x4){(u32)c00[0], (u32)c00[1], (u32)c01[0], (u32)c01[1]};
        d1 = (u32x4){(u32)c10[0], (u32)c10[1], (u32)c11[0], (u32)c11[1]};
    }
    const u32 call = (u32)__builtin_amdgcn_readfirstlane(prev) + 1u;
    const u32 par = call & 1u;
    char* slot = myws + OUTBOX_OFF + par * OUTBOX_HALF + (size_t)(((o * 16 + p) * NT + 4 * g + tt) * 2 + h) * 2048 + lane * 16;
    st16_sys(slot, d0); st16_sys(slot + 1024, d1);
    asm volatile("s_waitcnt vmcnt(0)" ::: "memory");
    __syncthreads();
    if (tid < NR) {
        const u32x4 f = {call, 0xffffffffu, call, 0xffffffffu};
        st16_sys(a.ws[tid] + RDY_OFF + par * RDY_HALF + (size_t)(a.rank * 256 + (o * 16 + p) * 8 + g) * 16, f);
    }
    if (tid == 0) __hip_atomic_store(cnt, call, __ATOMIC_RELAXED, __HIP_MEMORY_SCOPE_AGENT);
    return call;
}

__device__ __forceinline__ void pull(const Args& a, u32 call, int c) {
    const int tid = threadIdx.x, wid = __builtin_amdgcn_readfirstlane(tid >> 6), lane = tid & 63;
    const int o = c >> 4, p = c & 15;
    const u32 s = (u32)(a.rank + wid) & 7u, par = call & 1u;
    char* myws = a.ws[a.rank];
    u32* tmo = (u32*)(myws + TMO_OFF);
    bool dead = false;
    {
        const u64* rdy = (const u64*)(myws + RDY_OFF + par * RDY_HALF + (size_t)(s * 256 + c * 8) * 16) + (lane & 15);
        const u64 t0 = __builtin_amdgcn_s_memrealtime();
        for (;;) {
            const u64 v = __hip_atomic_load(rdy, __ATOMIC_RELAXED, __HIP_MEMORY_SCOPE_SYSTEM);
            if (__all((u32)v == call)) break;
            if (give_up(tmo, call, t0)) { dead = true; break; }
            __builtin_amdgcn_s_sleep(1);
        }
    }
    STAMP(2);
    u32* done = (u32*)(myws + DONE_OFF) + c * 32 + s;
    if (!dead) {
        const char* src = a.ws[s] + OUTBOX_OFF + par * OUTBOX_HALF + (size_t)c * (NT * 2 * 2048) + lane * 16;
        char* dst = myws + STAGE_OFF + (size_t)(o * 8 + s) * 65536 + (size_t)(256 * p + lane) * 16;
        u32x4 a0, a1, b0, b1;
#define LD2(r0, r1, u) asm volatile("global_load_dwordx4 %0, %2, off sc0 sc1\n\tglobal_load_dwordx4 %1, %2, off offset:1024 sc0 sc1" : "=&v"(r0), "=&v"(r1) : "v"(src + (size_t)(u) * 2048) : "memory")
        LD2(a0, a1, 0);
        for (int t = 0; t < NT; ++t) {
            asm volatile("s_waitcnt vmcnt(0)" : "+v"(a0), "+v"(a1) :: "memory");
            if (t > 0 && lane == 0) __hip_atomic_store(done, call * 64u + (u32)t, __ATOMIC_RELAXED, __HIP_MEMORY_SCOPE_AGENT);
            LD2(b0, b1, 2 * t + 1);
            char* d = dst + (size_t)t * (1 << 20);
            st16_agent(d, a0); st16_agent(d + 1024, a1);
            asm volatile("s_waitcnt vmcnt(2)" : "+v"(b0), "+v"(b1) :: "memory");
            if (t + 1 < NT) LD2(a0, a1, 2 * t + 2);
            st16_agent(d + 2048, b0); st16_agent(d + 3072, b1);
        }
#undef LD2
    }
    asm volatile("s_waitcnt vmcnt(0)" ::: "memory");
    if (lane == 0) __hip_atomic_store(done, call * 64u + (u32)NT, __ATOMIC_RELAXED, __HIP_MEMORY_SCOPE_AGENT);
    STAMP(3);
}

__device__ __forceinline__ void gemm(const Args& a, u32 call, int xcd, int slot, LAS char* lds) {
    const int tid = threadIdx.x, wid = __builtin_amdgcn_readfirstlane(tid >> 6), lane = tid & 63;
    const int li = (xcd & 1) * GEMM_SLOTS + slot, ti = (xcd >> 1) * 4 + (li & 3), tj = li >> 2;
    const int row0 = ti * BM, col0 = tj * BN > DIM - BN ? DIM - BN : tj * BN;
    const int wr = wid >> 2, wc = wid & 3, fr = lane & 15, fq = lane >> 4;
    char* myws = a.ws[a.rank];
    u32* tmo = (u32*)(myws + TMO_OFF);
    const char* gx = myws + STAGE_OFF + (size_t)wid * 65536 + (size_t)(row0 + lane) * 16;
    const char* gw = myws + STAGE_OFF + (size_t)(8 + wid) * 65536 + (size_t)(col0 + lane) * 16;
    const int xoff = 2 * fq * XS + (wr * 128 + fr) * 16, woff = XB + 2 * fq * WS + (wc * 80 + fr) * 16;
    const int fg = (lane >> 3) > 2 ? 0 : (lane >> 3);
    const int fline = fg == 0 ? ti : (fg == 1 ? 16 + (col0 >> 8) : 16 + ((col0 + BN - 1) >> 8));
    const u32* fl = (const u32*)(myws + DONE_OFF) + fline * 32 + (lane & 7);
    const u32 base = call * 64u;
    u32 seen = 0;
    auto wait_tile = [&](int t) {
        if (__all(seen > base + (u32)t)) return;
        const u64 t0 = __builtin_amdgcn_s_memrealtime();
        for (;;) {
            seen = __hip_atomic_load(fl, __ATOMIC_RELAXED, __HIP_MEMORY_SCOPE_AGENT);
            if (__all(seen > base + (u32)t)) break;
            if (give_up(tmo, call, t0)) { seen = 0xffffffffu; break; }
            __builtin_amdgcn_s_sleep(4);
        }
    };
#define DMA_X(t, b, i) __builtin_amdgcn_global_load_lds((const GAS u32*)(gx + (size_t)(t) * (1 << 20)), (LAS u32*)(lds + (b) * STG + wid * XS), 16, (i) * 1024, 17)
#define DMA_W(t, b, i) __builtin_amdgcn_global_load_lds((const GAS u32*)(gw + (size_t)(t) * (1 << 20) + ((i) == 4 ? 4096 : 0)), (LAS u32*)(lds + (b) * STG + XB + wid * WS + ((i) == 4 ? 4096 : 0)), 16, ((i) & 3) * 1024, 17)
#define DMA_ALL(t, b) do { DMA_W(t, b, 0); DMA_W(t, b, 1); DMA_W(t, b, 2); DMA_W(t, b, 3); DMA_W(t, b, 4); DMA_X(t, b, 0); DMA_X(t, b, 1); DMA_X(t, b, 2); DMA_X(t, b, 3); } while (0)
    v4f acc[8][5];
#pragma unroll
    for (int m = 0; m < 8; ++m)
#pragma unroll
        for (int n = 0; n < 5; ++n) acc[m][n] = (v4f){0.f, 0.f, 0.f, 0.f};
    if (wid == 0) wait_tile(1);
    asm volatile("s_waitcnt lgkmcnt(0)" ::: "memory");
    __builtin_amdgcn_s_barrier(); asm volatile("" ::: "memory");
    STAMP(2);
    DMA_ALL(0, 0); DMA_ALL(1, 1);
    asm volatile("s_waitcnt vmcnt(9)" ::: "memory");
    __builtin_amdgcn_s_barrier(); asm volatile("" ::: "memory");
    v8i wf[5], xa, xb;
    xa = ldfrag(lds, xoff, XS);
#pragma unroll
    for (int n = 0; n < 5; ++n) wf[n] = ldfrag(lds, woff + n * 256, WS);
#define MM(m, xf) do { _Pragma("unroll") for (int n = 0; n < 5; ++n) acc[m][n] = __builtin_amdgcn_mfma_scale_f32_16x16x128_f8f6f4(wf[n], xf, acc[m][n], 1, 0, 0, 0, 0, 0); } while (0)
    for (int t = 0; t < NT; ++t) {
        const int b = t & 1;
        LAS const char* cur = lds + b * STG; LAS const char* nxt = lds + (b ^ 1) * STG;
        const int t1 = t + 1 < NT ? t + 1 : t;
        xb = ldfrag(cur, xoff + 1 * 256, XS); if (t > 0 && t + 1 < NT) { DMA_W(t1, b ^ 1, 3); DMA_W(t1, b ^ 1, 4); DMA_X(t1, b ^ 1, 0); } SCHED; MM(0, xa); SCHED;
        xa = ldfrag(cur, xoff + 2 * 256, XS); if (t > 0 && t + 1 < NT) { DMA_X(t1, b ^ 1, 1); DMA_X(t1, b ^ 1, 2); DMA_X(t1, b ^ 1, 3); } SCHED; MM(1, xb); SCHED;
        xb = ldfrag(cur, xoff + 3 * 256, XS); SCHED; MM(2, xa); SCHED;
        xa = ldfrag(cur, xoff + 4 * 256, XS); SCHED; MM(3, xb); SCHED;
        xb = ldfrag(cur, xoff + 5 * 256, XS); SCHED; MM(4, xa); SCHED;
        xa = ldfrag(cur, xoff + 6 * 256, XS); SCHED; MM(5, xb); SCHED;
        xb = ldfrag(cur, xoff + 7 * 256, XS); SCHED; MM(6, xa); SCHED;
        if (wid == 0 && t + 2 < NT) wait_tile(t + 2);
        asm volatile("s_waitcnt vmcnt(0) lgkmcnt(0)" ::: "memory");
        __builtin_amdgcn_s_barrier(); asm volatile("" ::: "memory"); SCHED;
        if (t + 2 < NT) { DMA_W(t + 2, b, 0); DMA_W(t + 2, b, 1); DMA_W(t + 2, b, 2); }
        xa = ldfrag(nxt, xoff, XS); SCHED;
#pragma unroll
        for (int n = 0; n < 5; ++n) { acc[7][n] = __builtin_amdgcn_mfma_scale_f32_16x16x128_f8f6f4(wf[n], xb, acc[7][n], 1, 0, 0, 0, 0, 0); SCHED; wf[n] = ldfrag(nxt, woff + n * 256, WS); SCHED; }
    }
    STAMP(3);
    const bool bad = __hip_atomic_load(tmo, __ATOMIC_RELAXED, __HIP_MEMORY_SCOPE_AGENT) == call;
    const float scale = bad ? __builtin_nanf("") : a.sx[0] * a.sw[0];
    float* op = a.out + (size_t)(row0 + wr * 128 + fr) * DIM + col0 + wc * 80 + fq * 4;
    const int nskip = (tj == 12 && wc == 0) ? 4 : 0;
#pragma unroll
    for (int m = 0; m < 8; ++m)
#pragma unroll
        for (int n = 0; n < 5; ++n) {
            const v4f v = acc[m][n] * scale; v4f r;
#pragma unroll
            for (int j = 0; j < 4; ++j) {
                const float y = v[j], c = __builtin_amdgcn_fmed3f(y, -60.f, 60.f);
                r[j] = y * __builtin_amdgcn_rcpf(1.f + __builtin_amdgcn_exp2f(c * -1.44269504089f));
            }
            if (n >= nskip) *(v4f*)(op + (size_t)m * 16 * DIM + n * 16) = r;
        }
    STAMP(4);
}
}

__global__ void __launch_bounds__(512, 2) fused_allgather_gemm(Args a) {
    extern __shared__ __attribute__((aligned(16))) char lds_dyn[];
    LAS char* lds = (LAS char*)lds_dyn;
    STAMP(0);
    const u32 call = stage_out(a, lds);
    STAMP(1);
    const int xcd = blockIdx.x & 7, slot = blockIdx.x >> 3;
    if (slot < GEMM_SLOTS) gemm(a, call, xcd, slot, lds);
    else if (slot < GEMM_SLOTS + PULL_SLOTS) pull(a, call, (slot - GEMM_SLOTS) * 8 + xcd);
}

extern "C" void kernel_launch(int rank, int nranks, ncclComm_t comm, hipStream_t stream,
                              void* const* d_in, const int* in_sizes, int n_in,
                              void* d_out, int out_size, const int* in_dtypes, int out_dtype,
                              void* const* d_out_all, void* const* d_ws_all, size_t ws_bytes) {
    if (nranks != NR || (unsigned)rank >= (unsigned)NR || n_in != 4 || ws_bytes < WS_NEED || out_size != DIM * DIM ||
        in_sizes[0] != DIM * KS || in_sizes[1] != DIM * KS) {
        fprintf(stderr, "kernel_launch: unexpected problem shape\n");
        return;
    }
    static bool attr_set[NR] = {};
    if (!attr_set[rank]) {
        HIP_CHECK(hipFuncSetAttribute((const void*)fused_allgather_gemm, hipFuncAttributeMaxDynamicSharedMemorySize, LDS_BYTES));
        attr_set[rank] = true;
    }
    Args a;
    a.x = (const unsigned char*)d_in[0]; a.w = (const unsigned char*)d_in[1];
    a.sx = (const float*)d_in[2]; a.sw = (const float*)d_in[3];
    a.out = (float*)d_out;
    for (int p = 0; p < NR; ++p) a.ws[p] = (char*)d_ws_all[p];
    a.rank = rank; a.pad = 0;
    fused_allgather_gemm<<<256, 512, LDS_BYTES, stream>>>(a);
    HIP_CHECK(hipGetLastError());
}
```

```cpp
#include <hip/hip_runtime.h>
#include <rccl/rccl.h>
#include <cstdio>

#define HIP_CHECK(expr) do { hipError_t _e = (expr); if (_e != hipSuccess) { \
    fprintf(stderr, "HIP error %s at %s:%d\n", hipGetErrorString(_e), __FILE__, __LINE__); \
    return; } } while (0)

#define LAS __attribute__((address_space(3)))
#define GAS __attribute__((address_space(1)))
typedef unsigned u32;
typedef unsigned long long u64;
typedef int v2i __attribute__((ext_vector_type(2)));
typedef int v4i __attribute__((ext_vector_type(4)));
typedef int v8i __attribute__((ext_vector_type(8)));
typedef float v4f __attribute__((ext_vector_type(4)));
typedef unsigned u32x2 __attribute__((ext_vector_type(2)));
typedef unsigned u32x4 __attribute__((ext_vector_type(4)));

namespace {
constexpr int NR = 8;
constexpr int DIM = 4096;
constexpr int KS = DIM / NR;
constexpr int NT = 32;
constexpr size_t STAGE_OFF = 0;
constexpr size_t STAGE_BYTES = (size_t)NT * 2 * NR * DIM * 16;
constexpr size_t OUTBOX_OFF = STAGE_OFF + STAGE_BYTES;
constexpr size_t OUTBOX_HALF = (size_t)2 * 16 * NT * 2 * 2048;
constexpr size_t RDY_OFF = OUTBOX_OFF + 2 * OUTBOX_HALF;
constexpr size_t RDY_HALF = (size_t)NR * 256 * 16;
constexpr size_t DONE_OFF = RDY_OFF + 2 * RDY_HALF;
constexpr size_t CLAIM_OFF = DONE_OFF + (size_t)NT * 32 * 64;
constexpr size_t CNT_OFF = CLAIM_OFF + NR * 128;
constexpr size_t TMO_OFF = CNT_OFF + 256 * 4;
constexpr size_t DBG_OFF = TMO_OFF + 128;
constexpr size_t WS_NEED = DBG_OFF + 256 * 64 + 256 * 8 * 80 * 8;
constexpr u32 NUNITS = NT * 64;
constexpr u32 RAW_TILES = 1;
#define POLL_SLEEP 4
constexpr u32 RAW = 0xffu;
constexpr u64 DEADLINE = 150000000ull;
constexpr int BM = 256, BN = 320, XS = BM * 16, WS = BN * 16, XB = 8 * XS, WB = 8 * WS, STG = XB + WB;
constexpr int LDS_BYTES = 2 * STG + 2048 + 256;
constexpr int GEMM_SLOTS = 26, PULL_SLOTS = 6;

struct Args {
    const unsigned char* x; const unsigned char* w; const float* sx; const float* sw; float* out;
    char* me; char* ws[NR]; int rank; int pad;
};

__device__ __forceinline__ void st16_sys(void* p, u32x4 v) {
    asm volatile("global_store_dwordx4 %0, %1, off sc0 sc1\n\ts_nop 1" ::"v"(p), "v"(v) : "memory");
}
__device__ __forceinline__ void st16_agent(void* p, u32x4 v) {
    asm volatile("global_store_dwordx4 %0, %1, off sc1\n\ts_nop 1" ::"v"(p), "v"(v) : "memory");
}
__device__ __forceinline__ u32 ld32_agent(const void* p) {
    u32 v;
    asm volatile("global_load_dword %0, %1, off sc1\n\ts_waitcnt vmcnt(0)" : "=&v"(v) : "v"(p) : "memory");
    return v;
}
__device__ __forceinline__ void st32_agent_lane0(void* p, u32 v) {
    asm volatile("s_mov_b64 exec, 1\n\tglobal_store_dword %0, %1, off sc1\n\ts_mov_b64 exec, -1" ::"v"(p), "v"(v) : "memory");
}
__device__ __forceinline__ bool give_up(u32* tmo, u32 call, u64 t0) {
    if ((u32)__builtin_amdgcn_readfirstlane((int)ld32_agent(tmo)) == call || __builtin_amdgcn_s_memrealtime() - t0 > DEADLINE) {
        st32_agent_lane0(tmo, call);
        return true;
    }
    return false;
}
template <class T> __device__ __forceinline__ T* uniform_ptr(T* p) {
    const u64 v = (u64)p;
    return (T*)(((u64)(u32)__builtin_amdgcn_readfirstlane((int)(u32)(v >> 32)) << 32) | (u32)__builtin_amdgcn_readfirstlane((int)(u32)v));
}
__device__ __forceinline__ v8i ldfrag(LAS const char* p, int off, int stride) {
    v4i a = *(LAS const v4i*)(p + off), b = *(LAS const v4i*)(p + off + stride);
    return __builtin_shufflevector(a, b, 0, 1, 2, 3, 4, 5, 6, 7);
}
#define SCHED __builtin_amdgcn_sched_barrier(0)
#define STAMP(i) do { } while (0)
#define WSTAMP(i) do { } while (0)

__device__ __forceinline__ u32 wave_incl_scan(u32 x) {
    x += (u32)__builtin_amdgcn_update_dpp(0, (int)x, 0x111, 0xf, 0xf, false);
    x += (u32)__builtin_amdgcn_update_dpp(0, (int)x, 0x112, 0xf, 0xf, false);
    x += (u32)__builtin_amdgcn_update_dpp(0, (int)x, 0x114, 0xf, 0xf, false);
    x += (u32)__builtin_amdgcn_update_dpp(0, (int)x, 0x118, 0xf, 0xf, false);
    x += (u32)__builtin_amdgcn_update_dpp(0, (int)x, 0x142, 0xa, 0xf, false);
    x += (u32)__builtin_amdgcn_update_dpp(0, (int)x, 0x143, 0xc, 0xf, false);
    return x;
}
__device__ __forceinline__ u32 rotr(u32 x, u32 n) { return __builtin_amdgcn_alignbit(x, x, n); }
__device__ __forceinline__ u32 bfi(u32 m, u32 a, u32 b) { return (a & m) | (b & ~m); }
__device__ __forceinline__ u32 lds_window(u32 addr) {
    LAS const volatile u32* p = (LAS const volatile u32*)(size_t)(addr & ~3u);
    const u32 w0 = p[0], w1 = p[1];
    return __builtin_amdgcn_alignbyte(w1, w0, addr);
}

constexpr int P0_VAR = 2560, P0_LUT = P0_VAR + 1024, P0_WAVE = P0_LUT + 64, P0_SIZE = 8 * P0_WAVE;

template <int O>
__device__ __forceinline__ u32 pack_unit(const u32 (&d)[8], char* slot, LAS char* var, u32 lut, int lane) {
    constexpr int SH = O ? 2 : 3;
    constexpr u32 EM = O ? 0x1f1f1f1fu : 0x0f0f0f0fu, VOFF = O ? 1280u : 1536u, CAP = 2048u - VOFF;
    u32 ex[8], sum = 0;
#pragma unroll
    for (int j = 0; j < 8; ++j) ex[j] = (d[j] >> SH) & EM;
#pragma unroll
    for (int j = 0; j < 8; j += 2) sum = __builtin_amdgcn_udot4(ex[j], 0x01010101u, sum, false);
    const u32 tot = (u32)__builtin_amdgcn_readlane((int)wave_incl_scan(sum), 63);
    const u32 lo = tot > 50u ? (tot - 50u) >> 10 : 0u;
    const u32 bias = (0x80u - lo) * 0x01010101u;
    u32 P[2] = {0u, 0u}, X = 0, nb = 0, q[8], nbj[8];
#pragma unroll
    for (int j = 0; j < 8; ++j) {
        const u32 a7 = ex[j] + bias;
        const u32 t = ((~a7 | (a7 - 0x03030303u)) & 0x80808080u) >> 7;
        P[j >> 2] |= ((a7 | t | (t << 1)) & 0x03030303u) << (2 * (j & 3));
        const u32 sc = a7 & ((a7 | 0x80808080u) >> 5) & 0x07070707u;
        X = (X >> 1) | ((~(a7 + 0x07070707u) | (a7 - 0x04040404u)) & 0x80808080u);
        const u32 cz = __builtin_amdgcn_perm(0u, sc, *(LAS const u32*)(size_t)__builtin_amdgcn_udot4(t, 0x20100804u, lut, false));
        q[j] = __builtin_amdgcn_udot4(cz, 0x003e0801u, cz >> 15, false);
        nbj[j] = nb;
        nb = __builtin_amdgcn_udot4(t, 0x03030303u, nb, false);
    }
    u64 buf = (u64)(q[0] | (q[1] << nbj[1]));
#pragma unroll
    for (int j = 2; j < 8; j += 2) buf |= (u64)(q[j] | (q[j + 1] << (nbj[j + 1] - nbj[j]))) << (nbj[j] & 63u);
    const u32 incl = wave_incl_scan(nb), preb = incl - nb;
    const u32 nbits = (u32)__builtin_amdgcn_readlane((int)incl, 63);
    const u32 xoff = 4u + (((nbits + 15u) >> 4) << 1);
    bool raw = __any(nb > 63u) || xoff > CAP;
    u32 nexc = 0;
    if (!raw) {
        const u32 sh = preb & 31u;
        LAS u32* w = (LAS u32*)(var + 4u + ((preb >> 5) << 2));
        const u64 up = buf << sh;
        __hip_atomic_fetch_or(w, (u32)up, __ATOMIC_RELAXED, __HIP_MEMORY_SCOPE_WAVEFRONT);
        __hip_atomic_fetch_or(w + 1, (u32)(up >> 32), __ATOMIC_RELAXED, __HIP_MEMORY_SCOPE_WAVEFRONT);
        __hip_atomic_fetch_or(w + 2, (u32)(buf >> 32) >> 1 >> (31u - sh), __ATOMIC_RELAXED, __HIP_MEMORY_SCOPE_WAVEFRONT);
        u64 bal;
        while ((bal = __ballot(X != 0u)) != 0ull) {
            const u32 slotn = nexc + __builtin_amdgcn_mbcnt_hi((u32)(bal >> 32), __builtin_amdgcn_mbcnt_lo((u32)bal, 0u));
            nexc += (u32)__builtin_popcountll(bal);
            if (nexc > 64u || xoff + 2u * nexc > CAP) { raw = true; break; }
            if (X != 0u) {
                const u32 k = (u32)__builtin_ctz(X), j = k & 7u, i8 = k & 24u;
                const u32 e01 = (j & 1) ? ex[1] : ex[0], e23 = (j & 1) ? ex[3] : ex[2], e45 = (j & 1) ? ex[5] : ex[4], e67 = (j & 1) ? ex[7] : ex[6];
                const u32 e03 = (j & 2) ? e23 : e01, e47 = (j & 2) ? e67 : e45, ej = (j & 4) ? e47 : e03;
                const u32 n01 = (j & 1) ? nbj[1] : nbj[0], n23 = (j & 1) ? nbj[3] : nbj[2], n45 = (j & 1) ? nbj[5] : nbj[4], n67 = (j & 1) ? nbj[7] : nbj[6];
                const u32 n03 = (j & 2) ? n23 : n01, n47 = (j & 2) ? n67 : n45, nj = (j & 4) ? n47 : n03;
                const u32 Pj = ((j & 4) ? P[1] : P[0]) >> (2u * (j & 3u));
                const u32 pos = (((preb + nj) * 43691u) >> 17) + (u32)__builtin_popcount(Pj & (Pj >> 1) & 0x01010101u & ((1u << i8) - 1u));
                *(LAS unsigned short*)(var + xoff + 2u * slotn) = (unsigned short)(pos | (((ej >> i8) & 31u) << 11));
                X &= X - 1u;
            }
        }
    }
    if (raw) {
        st16_sys(slot + lane * 16, (u32x4){d[0], d[1], d[2], d[3]});
        st16_sys(slot + 1024 + lane * 16, (u32x4){d[4], d[5], d[6], d[7]});
        return RAW;
    }
    if (lane == 0) *(LAS u32*)var = lo | (nexc << 8);
    const u32 nvar = (xoff + 2u * nexc + 15u) >> 4;
    const u32x4 v = *(LAS const u32x4*)(var + lane * 16);
    if (O == 0) {
        u32x4 sm;
#pragma unroll
        for (int k = 0; k < 4; ++k) sm[k] = bfi(0x87878787u, d[2 * k], rotr(d[2 * k + 1], 28));
        st16_sys(slot + lane * 16, sm);
        asm volatile("global_store_dwordx2 %0, %1, off offset:1024 sc0 sc1\n\ts_nop 1" ::"v"(slot + lane * 8), "v"((u32x2){P[0], P[1]}) : "memory");
    } else {
        u32x4 sm;
        sm[0] = bfi(0x60606060u, d[6] << 5, bfi(0x83838383u, d[0], rotr(d[1], 29)));
        sm[1] = bfi(0x60606060u, d[7] << 5, bfi(0x83838383u, d[2], rotr(d[3], 29)));
        sm[2] = bfi(0x60606060u, bfi(0x40404040u, d[6] >> 1, d[7] >> 2), bfi(0x83838383u, d[4], rotr(d[5], 29)));
        sm[3] = P[0];
        st16_sys(slot + lane * 16, sm);
        const u32 up = __builtin_amdgcn_permlane32_swap(P[1], P[1], false, false)[1];
        if (lane < 32) asm volatile("global_store_dwordx2 %0, %1, off offset:1024 sc0 sc1\n\ts_nop 1" ::"v"(slot + lane * 8), "v"((u32x2){P[1], up}) : "memory");
    }
    if ((u32)lane < nvar) st16_sys(slot + VOFF + lane * 16, v);
    return nvar;
}

template <int O, class F>
__device__ __forceinline__ void unpack_unit(const u32x4 A, const u32x2 B, const u32x4 V, u32 (&d)[8], LAS char* var, LAS char* exp, u32 lut, int lane, F&& midway) {
    constexpr int SH = O ? 2 : 3;
    *(LAS u32x4*)(var + lane * 16) = V;
    const u32 P[2] = {O ? A[3] : B[0], O ? __builtin_amdgcn_permlane32_swap(B[0], B[1], false, false)[0] : B[1]};
    const u32 hdr = (u32)__builtin_amdgcn_readfirstlane((int)V[0]), lo = hdr & 31u, nexc = (hdr >> 8) & 127u;
    const u32 E[2] = {P[0] & (P[0] >> 1) & 0x55555555u, P[1] & (P[1] >> 1) & 0x55555555u};
    const u32 cnt = (u32)__builtin_popcount(E[0]) + (u32)__builtin_popcount(E[1]), incl = wave_incl_scan(cnt);
    const u32 n = min((u32)__builtin_amdgcn_readlane((int)incl, 63), 1344u);
    const u32 prim = ((lo * 0x01010101u + 0x00020100u) << SH);
    const u32 b8 = ((lo + 24u) & 31u) * 0x01010101u;
    const u32 seclo = ((((b8 + 0x03020100u) & 0x1f1f1f00u) | ((lo + 3u) & 31u)) << SH);
    const u32 sechi = ((b8 + 0x07060504u) & 0x1f1f1f1fu) << SH;
    for (u32 base = 0; base < n; base += 512u) {
        const u32 v = lds_window((u32)(size_t)var + 4u + 3u * ((base >> 3) + (u32)lane));
        const u32 v2 = v >> 12;
        u32 x0 = bfi(0x3fu, v, v << 10), x1 = bfi(0x3fu, v2, v2 << 10);
        x0 = bfi(0x00070007u, x0, x0 << 5) & 0x07070707u; x1 = bfi(0x00070007u, x1, x1 << 5) & 0x07070707u;
        *(LAS u32x2*)(exp + base + lane * 8) = (u32x2){__builtin_amdgcn_perm(sechi, seclo, x0), __builtin_amdgcn_perm(sechi, seclo, x1)};
    }
    if ((u32)lane < nexc) {
        const u32 e = *(LAS const unsigned short*)(var + 4u + (((3u * n + 15u) >> 4) << 1) + 2u * (u32)lane);
        *(LAS unsigned char*)(exp + min(e & 0x7ffu, 1535u)) = (unsigned char)((e >> 11) << SH);
    }
    midway();
    u32 ptr = (u32)(size_t)exp + min(incl - cnt, 1344u), ex[8];
#pragma unroll
    for (int j = 0; j < 8; ++j) {
        const int g = j & 3;
        const u32 Eh = g < 2 ? E[j >> 2] : E[j >> 2] >> 4;
        const u32 m = Eh & ((g & 1) ? 0x04040404u : 0x01010101u);
        const u32 dep = *(LAS const u32*)(size_t)__builtin_amdgcn_udot4(m, (g & 1) ? 0x08040201u : 0x20100804u, lut, false);
        const u32 win = lds_window(ptr);
        ptr += (u32)__builtin_popcount(m);
        ex[j] = __builtin_amdgcn_perm(win, prim, ((P[j >> 2] >> (2 * g)) & 0x03030303u) ^ dep);
    }
    if (O == 0) {
#pragma unroll
        for (int k = 0; k < 4; ++k) { d[2 * k] = (A[k] & 0x87878787u) | ex[2 * k]; d[2 * k + 1] = (rotr(A[k], 4) & 0x87878787u) | ex[2 * k + 1]; }
    } else {
#pragma unroll
        for (int k = 0; k < 3; ++k) { d[2 * k] = (A[k] & 0x83838383u) | ex[2 * k]; d[2 * k + 1] = (rotr(A[k], 3) & 0x83838383u) | ex[2 * k + 1]; }
        d[6] = (bfi(0x80808080u, A[2] << 1, A[0] >> 5) & 0x83838383u) | ex[6];
        d[7] = (bfi(0x80808080u, A[2] << 2, A[1] >> 5) & 0x83838383u) | ex[7];
    }
}

__device__ __forceinline__ size_t stage_off(u32 k, u32 s) {
    return STAGE_OFF + (size_t)(k >> 6) * (1u << 20) + (size_t)(((k >> 1) & 1u) * 8u + s) * 65536u + (size_t)(((k >> 2) & 15u) * 4096u + (k & 1u) * 2048u);
}
__device__ __forceinline__ u32 done_word(u32 k, u32 s) { return ((k >> 6) * 32u + ((k >> 1) & 1u) * 16u + ((k >> 2) & 15u)) * 16u + s * 2u + (k & 1u); }

__device__ __forceinline__ void load_unit(const Args& a, u32 k, int lane, u32x4& r0, u32x4& r1) {
    const u32 t = k >> 6, rows = ((k >> 2) & 15u) * 256u + (k & 1u) * 128u;
    const unsigned char* src = (k & 2u) ? a.w + (size_t)(16u * t + (lane >> 3)) * DIM + rows + (lane & 7) * 16
                                        : a.x + (size_t)(rows + lane) * KS + 16u * t;
    r0 = *(const u32x4*)src; r1 = *(const u32x4*)(src + 32768);
}
__device__ __forceinline__ void stage_out(const Args& a, LAS char* lds, const u32 b, const u32 call, const int round, const u32x4 r0, const u32x4 r1) {
    const int tid = threadIdx.x, wid = __builtin_amdgcn_readfirstlane(tid >> 6), lane = tid & 63;
    const u32 k = b * 8u + (u32)wid, t = k >> 6, o = (k >> 1) & 1u;
    char* myws = a.me;
    LAS char* wl = lds + wid * P0_WAVE;
    {
        u32 sel = 0x0c0c0c0cu; int n = 0;
#pragma unroll
        for (int i = 0; i < 4; ++i) if ((lane >> i) & 1) { sel = (sel & ~(0xffu << (8 * n))) | ((u32)i << (8 * n)); ++n; }
        if (lane < 16) *(LAS u32*)(wl + P0_LUT + lane * 4) = sel;
        *(LAS u32x4*)(wl + P0_VAR + lane * 16) = (u32x4){0u, 0u, 0u, 0u};
    }
    u32 d[8];
    if (o == 0) {
#pragma unroll
        for (int j = 0; j < 4; ++j) { d[j] = r0[j]; d[4 + j] = r1[j]; }
    } else {
        *(LAS u32x4*)(wl + (lane >> 3) * 160 + (lane & 7) * 16) = r0;
        *(LAS u32x4*)(wl + (8 + (lane >> 3)) * 160 + (lane & 7) * 16) = r1;
        LAS char* tb = wl + ((lane & 15) >> 1) * 160 + (lane >> 4) * 16 + (lane & 1) * 8;
        asm volatile("" ::: "memory");
        const v2i c00 = __builtin_amdgcn_ds_read_tr8_b64_v2i32((LAS v2i*)tb);
        const v2i c01 = __builtin_amdgcn_ds_read_tr8_b64_v2i32((LAS v2i*)(tb + 8 * 160));
        const v2i c10 = __builtin_amdgcn_ds_read_tr8_b64_v2i32((LAS v2i*)(tb + 64));
        const v2i c11 = __builtin_amdgcn_ds_read_tr8_b64_v2i32((LAS v2i*)(tb + 8 * 160 + 64));
        d[0] = c00[0]; d[1] = c00[1]; d[2] = c01[0]; d[3] = c01[1]; d[4] = c10[0]; d[5] = c10[1]; d[6] = c11[0]; d[7] = c11[1];
    }
    const u32 par = call & 1u;
    char* slot = myws + OUTBOX_OFF + par * OUTBOX_HALF + (size_t)k * 2048;
    char* own = myws + stage_off(k, (u32)a.rank) + lane * 16;
    const u32x4 w0 = {d[0], d[1], d[2], d[3]}, w1 = {d[4], d[5], d[6], d[7]};
    u32 size = RAW;
    if (t < RAW_TILES) { st16_sys(slot + lane * 16, w0); st16_sys(slot + 1024 + lane * 16, w1); }
    st16_agent(own, w0); st16_agent(own + 1024, w1);
    if (t >= RAW_TILES) {
        if (o == 0) size = pack_unit<0>(d, slot, wl + P0_VAR, (u32)(size_t)(wl + P0_LUT), lane);
        else size = pack_unit<1>(d, slot, wl + P0_VAR, (u32)(size_t)(wl + P0_LUT), lane);
    }
    if (lane == 0) *(LAS unsigned char*)(lds + P0_SIZE + round * 8 + wid) = (unsigned char)size;
    asm volatile("s_waitcnt vmcnt(0)" ::: "memory");
    __syncthreads();
    if (wid == 0) {
        const u32x2 sz = *(LAS const u32x2*)(lds + P0_SIZE + round * 8);
        const u32x4 f = {call, sz[0], call, sz[1]};
        const u32 off = (u32)(RDY_OFF + par * RDY_HALF) + ((u32)a.rank * 256u + b) * 16u;
#pragma unroll
        for (int p = 0; p < NR; ++p)
            if (p != a.rank) asm volatile("s_mov_b64 exec, 1\n\tglobal_store_dwordx4 %0, %1, %2 sc0 sc1\n\ts_mov_b64 exec, -1\n\ts_nop 0" ::"v"(off), "v"(f), "s"(a.ws[p]) : "memory");
    }
    st32_agent_lane0((u32*)(myws + DONE_OFF) + done_word(k, (u32)a.rank), call);
}

constexpr int PL_BASE = 32768, PL_WAVE = 2048 + 1024 + 2048 + 64;
constexpr u32 PULL_WAVES = PULL_SLOTS * 8 * 8;

__device__ __forceinline__ void pull(const Args& a, u32 call, int c, LAS char* lds) {
    const int tid = threadIdx.x, wid = __builtin_amdgcn_readfirstlane(tid >> 6), lane = tid & 63;
    const u32 gw = (u32)c * 8u + (u32)wid, link = gw % 7u, rk = gw / 7u;
    const u32 nw = (PULL_WAVES + 6u - link) / 7u;
    const u32 s = ((u32)a.rank + 1u + link) & 7u, par = call & 1u;
    char* myws = a.me;
    u32* tmo = (u32*)(myws + TMO_OFF);
    LAS char* sizes = lds + PL_BASE + wid * PL_WAVE;
    LAS char* var = sizes + 2048;
    LAS char* exp = var + 1024;
    LAS char* lutp = exp + 2048;
    if (lane < 16) {
        u32 dep = 0; u32 n = 0;
#pragma unroll
        for (int i = 0; i < 4; ++i) if ((lane >> i) & 1) { dep |= (3u ^ (4u + n)) << (8 * i); ++n; }
        *(LAS u32*)(lutp + lane * 4) = dep;
    }
    const char* box = uniform_ptr(a.ws[s] + OUTBOX_OFF + par * OUTBOX_HALF);
    const char* rdy = myws + RDY_OFF + par * RDY_HALF + (size_t)s * 4096 + lane * 16;
    __builtin_amdgcn_s_waitcnt(0x0F70);
    u32 known = 0;
    auto wait_units = [&](u32 need) -> bool {
        const u64 t0 = __builtin_amdgcn_s_memrealtime();
        for (;;) {
            const u32 q = known >> 9;
            u32x4 f;
            asm volatile("global_load_dwordx4 %0, %1, off sc0 sc1\n\ts_waitcnt vmcnt(0)" : "=&v"(f) : "v"(rdy + q * 1024u) : "memory");
            const u64 bad = __ballot(f[0] != call || f[2] != call);
            *(LAS u32x2*)(sizes + q * 512u + lane * 8) = (u32x2){f[1], f[3]};
            known = q * 512u + (bad ? (u32)__builtin_ctzll(bad) : 64u) * 8u;
            if (known > need) return true;
            if (!bad) continue;
            if (__any((int)(f[0] - call) > 0)) { st32_agent_lane0(tmo, call); return false; }
            if (give_up(tmo, call, t0)) return false;
            __builtin_amdgcn_s_sleep(POLL_SLEEP);
        }
    };
    u32* claim = (u32*)(myws + CLAIM_OFF) + link * 32u;
    const u32 tbase = 2u * nw - (call - 1u) * (NUNITS - nw);
    auto drain = [&](u32 k) {
        while (k < NUNITS) {
            u32 r;
            asm volatile("s_mov_b64 exec, 1\n\tglobal_atomic_add %0, %1, %2, off sc0\n\ts_mov_b64 exec, -1\n\ts_waitcnt vmcnt(0)" : "=&v"(r) : "v"(claim), "v"(1u) : "memory");
            k = (u32)__builtin_amdgcn_readfirstlane((int)r) + tbase;
        }
    };
    if (!wait_units(rk)) { drain(rk + nw); return; }
    STAMP(2); WSTAMP(33);
    char* stage = myws + lane * 16;
    u32* done = (u32*)(myws + DONE_OFF);
    const u32 lut = (u32)(size_t)lutp;
    u32x4 LA, LV; u32x2 LB; u32 tk;
#define ISSUE(kk, sz) do { \
        const u32 k = (u32)__builtin_amdgcn_readfirstlane((int)(kk)); \
        sz = (u32)__builtin_amdgcn_readfirstlane(*(LAS const unsigned char*)(sizes + (k))); \
        const char* sl_ = uniform_ptr(box + (size_t)(k) * 2048); \
        const bool r_ = sz == RAW, w_ = ((k) & 2u) != 0u; \
        const u64 mb_ = r_ ? 1ull : w_ ? 0xffffffffull : ~0ull; \
        const u64 mv_ = r_ ? ~0ull : (1ull << min(max(sz, 1u), w_ ? 48u : 32u)) - 1ull; \
        asm volatile("s_nop 4\n\tglobal_load_dwordx4 %0, %3, %5 sc0 sc1\n\ts_mov_b64 exec, %7\n\tglobal_load_dwordx2 %1, %4, %5 offset:1024 sc0 sc1\n\t" \
                     "s_mov_b64 exec, %8\n\tglobal_load_dwordx4 %2, %3, %6 sc0 sc1\n\ts_mov_b64 exec, -1" \
                     : "=&v"(LA), "=&v"(LB), "=&v"(LV) : "v"(lane * 16), "v"(lane * 8), "s"(sl_), "s"(sl_ + (r_ ? 1024u : w_ ? 1280u : 1536u)), "s"(mb_), "s"(mv_) : "memory"); \
    } while (0)
#define FLAG(k) st32_agent_lane0(done + done_word(k, s), call)
    u32 cur = rk, nxt = rk + nw, prv = NUNITS, szc, szn = 0;
    ISSUE(cur, szc);
    asm volatile("s_waitcnt vmcnt(0)" : "+v"(LA), "+v"(LB), "+v"(LV) :: "memory");
    for (;;) {
        const u32x4 A = LA, V = LV; const u32x2 B = LB;
        const bool more = nxt < NUNITS;
        if (more) {
            if (nxt >= known && !wait_units(nxt)) { drain(nxt); return; }
            asm volatile("s_mov_b64 exec, 1\n\tglobal_atomic_add %0, %1, %2, off sc0\n\ts_mov_b64 exec, -1" : "=&v"(tk) : "v"(claim), "v"(1u) : "memory");
            ISSUE(nxt, szn);
        }
        u32 d[8];
        auto publish_prev = [&]() {
            if (more) asm volatile("s_waitcnt vmcnt(4)" ::: "memory"); else asm volatile("s_waitcnt vmcnt(0)" ::: "memory");
            if (prv < NUNITS) FLAG(prv);
        };
        if (szc == RAW) {
#pragma unroll
            for (int j = 0; j < 4; ++j) { d[j] = A[j]; d[4 + j] = V[j]; }
            publish_prev();
        } else if (cur & 2u) unpack_unit<1>(A, B, V, d, var, exp, lut, lane, publish_prev);
        else unpack_unit<0>(A, B, V, d, var, exp, lut, lane, publish_prev);
        char* dst = stage + stage_off(cur, s);
        if (!more) {
            st16_agent(dst, (u32x4){d[0], d[1], d[2], d[3]}); st16_agent(dst + 1024, (u32x4){d[4], d[5], d[6], d[7]});
            asm volatile("s_waitcnt vmcnt(0)" ::: "memory");
            FLAG(cur);
            break;
        }
        st16_agent(dst, (u32x4){d[0], d[1], d[2], d[3]}); st16_agent(dst + 1024, (u32x4){d[4], d[5], d[6], d[7]});
        asm volatile("s_waitcnt vmcnt(2)" : "+v"(LA), "+v"(LB), "+v"(LV), "+v"(tk) :: "memory");
        prv = cur; cur = nxt; szc = szn;
        nxt = (u32)__builtin_amdgcn_readfirstlane((int)tk) + tbase;
    }
#undef ISSUE
#undef FLAG
    WSTAMP(32);
    STAMP(3);
}

__device__ __forceinline__ void gemm(const Args& a, u32 call, int xcd, int slot, LAS char* lds) {
    const int tid = threadIdx.x, wid = __builtin_amdgcn_readfirstlane(tid >> 6), lane = tid & 63;
    const int li = (xcd & 1) * GEMM_SLOTS + slot, ti = (xcd >> 1) * 4 + (li & 3), tj = li >> 2;
    const int row0 = ti * BM, col0 = tj * BN > DIM - BN ? DIM - BN : tj * BN;
    const int wr = wid >> 2, wc = wid & 3, fr = lane & 15, fq = lane >> 4;
    char* myws = a.me;
    u32* tmo = (u32*)(myws + TMO_OFF);
    const char* gx = myws + STAGE_OFF + (size_t)wid * 65536 + (size_t)(row0 + lane) * 16;
    const char* gw = myws + STAGE_OFF + (size_t)(8 + wid) * 65536 + (size_t)(col0 + lane) * 16;
    const int xoff = 2 * fq * XS + (wr * 128 + fr) * 16, woff = XB + 2 * fq * WS + (wc * 80 + fr) * 16;
    const int fg = (lane >> 4) > 2 ? 0 : (lane >> 4);
    const int fline = fg == 0 ? ti : (fg == 1 ? 16 + (col0 >> 8) : 16 + ((col0 + BN - 1) >> 8));
    const u32* fl = (const u32*)(myws + DONE_OFF) + fline * 16 + (lane & 15);
    bool dead = false;
    LAS u32* snap = (LAS u32*)(lds + 2 * STG + 2048);
    auto wait_tile = [&](int t) {
        if (dead) return;
        const u64 t0 = __builtin_amdgcn_s_memrealtime();
        for (;;) {
            const u32 seen = ld32_agent(fl + t * 512);
            if (__all(seen == call)) break;
            if (give_up(tmo, call, t0)) { dead = true; break; }
            __builtin_amdgcn_s_sleep(4);
        }
    };
#define DMA_X(t, b, i) __builtin_amdgcn_global_load_lds((const GAS u32*)(gx + (size_t)(t) * (1 << 20)), (LAS u32*)(lds + (b) * STG + wid * XS), 16, (i) * 1024, 17)
#define DMA_W(t, b, i) __builtin_amdgcn_global_load_lds((const GAS u32*)(gw + (size_t)(t) * (1 << 20) + ((i) == 4 ? 4096 : 0)), (LAS u32*)(lds + (b) * STG + XB + wid * WS + ((i) == 4 ? 4096 : 0)), 16, ((i) & 3) * 1024, 17)
#define DMA_ALL(t, b) do { DMA_W(t, b, 0); DMA_W(t, b, 1); DMA_W(t, b, 2); DMA_W(t, b, 3); DMA_W(t, b, 4); DMA_X(t, b, 0); DMA_X(t, b, 1); DMA_X(t, b, 2); DMA_X(t, b, 3); } while (0)
    v4f acc[8][5];
#pragma unroll
    for (int m = 0; m < 8; ++m)
#pragma unroll
        for (int n = 0; n < 5; ++n) acc[m][n] = (v4f){0.f, 0.f, 0.f, 0.f};
    const float scale = __builtin_bit_cast(float, __builtin_amdgcn_readfirstlane(__builtin_bit_cast(int, a.sx[0] * a.sw[0])));
    __builtin_amdgcn_s_waitcnt(0x0F70);
    if (wid == 0) { wait_tile(0); wait_tile(1); }
    asm volatile("s_waitcnt lgkmcnt(0)" ::: "memory");
    __builtin_amdgcn_s_barrier(); asm volatile("" ::: "memory");
    STAMP(2);
    DMA_ALL(0, 0); DMA_ALL(1, 1);
    asm volatile("s_waitcnt vmcnt(9)" ::: "memory");
    __builtin_amdgcn_s_barrier(); asm volatile("" ::: "memory");
    v8i wf[5], xa, xb;
    xa = ldfrag(lds, xoff, XS);
#pragma unroll
    for (int n = 0; n < 5; ++n) wf[n] = ldfrag(lds, woff + n * 256, WS);
#define MM(m, xf) do { _Pragma("unroll") for (int n = 0; n < 5; ++n) acc[m][n] = __builtin_amdgcn_mfma_scale_f32_16x16x128_f8f6f4(wf[n], xf, acc[m][n], 1, 0, 0, 0, 0, 0); } while (0)
    for (int t = 0; t < NT; ++t) {
        const int b = t & 1;
        LAS const char* cur = lds + b * STG; LAS const char* nxt = lds + (b ^ 1) * STG;
        const int t1 = t + 1 < NT ? t + 1 : t;
        xb = ldfrag(cur, xoff + 1 * 256, XS); if (t > 0 && t + 1 < NT) { DMA_W(t1, b ^ 1, 3); DMA_W(t1, b ^ 1, 4); DMA_X(t1, b ^ 1, 0); } SCHED; MM(0, xa); SCHED;
        xa = ldfrag(cur, xoff + 2 * 256, XS); if (t > 0 && t + 1 < NT) { DMA_X(t1, b ^ 1, 1); DMA_X(t1, b ^ 1, 2); DMA_X(t1, b ^ 1, 3); } SCHED; MM(1, xb); SCHED;
        xb = ldfrag(cur, xoff + 3 * 256, XS); SCHED; MM(2, xa); SCHED;
        xa = ldfrag(cur, xoff + 4 * 256, XS); SCHED; MM(3, xb); SCHED;
        xb = ldfrag(cur, xoff + 5 * 256, XS);
        if (wid == 0 && t + 2 < NT) __builtin_amdgcn_global_load_lds((const GAS u32*)(fl + (t + 2) * 512), snap, 4, 0, 16);
        SCHED; MM(4, xa); SCHED;
        xa = ldfrag(cur, xoff + 6 * 256, XS); SCHED; MM(5, xb); SCHED;
        xb = ldfrag(cur, xoff + 7 * 256, XS); SCHED; MM(6, xa); SCHED;
        asm volatile("s_waitcnt vmcnt(0) lgkmcnt(0)" ::: "memory");
        if (wid == 0) {
            WSTAMP(t);
            if (t + 2 < NT && !__all(*(LAS volatile u32*)(snap + lane) == call)) wait_tile(t + 2);
            WSTAMP(32 + t);
        }
        __builtin_amdgcn_s_barrier(); asm volatile("" ::: "memory"); SCHED;
        if (t + 2 < NT) { DMA_W(t + 2, b, 0); DMA_W(t + 2, b, 1); DMA_W(t + 2, b, 2); }
        else if (t + 1 < NT) __builtin_amdgcn_global_load_lds((const GAS u32*)tmo, (LAS u32*)(lds + 2 * STG + wid * 256), 4, 0, 16);
        xa = ldfrag(nxt, xoff, XS); SCHED;
#pragma unroll
        for (int n = 0; n < 5; ++n) { acc[7][n] = __builtin_amdgcn_mfma_scale_f32_16x16x128_f8f6f4(wf[n], xb, acc[7][n], 1, 0, 0, 0, 0, 0); SCHED; wf[n] = ldfrag(nxt, woff + n * 256, WS); SCHED; }
    }
    STAMP(3);
    const float sc = *(LAS const u32*)(lds + 2 * STG + wid * 256) == call ? __builtin_nanf("") : scale;
    float* op = a.out + (size_t)(row0 + wr * 128 + fr) * DIM + col0 + wc * 80 + fq * 4;
    const int nskip = (tj == 12 && wc == 0) ? 4 : 0;
#pragma unroll
    for (int m = 0; m < 8; ++m)
#pragma unroll
        for (int n = 0; n < 5; ++n) {
            const v4f v = acc[m][n] * sc; v4f r;
#pragma unroll
            for (int j = 0; j < 4; ++j) {
                const float y = v[j], c = __builtin_amdgcn_fmed3f(y, -60.f, 60.f);
                r[j] = y * __builtin_amdgcn_rcpf(1.f + __builtin_amdgcn_exp2f(c * -1.44269504089f));
            }
            if (n >= nskip) st16_sys(op + (size_t)m * 16 * DIM + n * 16, __builtin_bit_cast(u32x4, r));
        }
    STAMP(4);
}
}

__global__ void __launch_bounds__(512, 2) fused_allgather_gemm(Args a) {
    extern __shared__ __attribute__((aligned(16))) char lds_dyn[];
    LAS char* lds = (LAS char*)lds_dyn;
    STAMP(0);
    const int xcd = blockIdx.x & 7, slot = blockIdx.x >> 3;
    u32* cnt = (u32*)(a.me + CNT_OFF) + blockIdx.x;
    if (slot < GEMM_SLOTS) {
        const u32 k0 = blockIdx.x * 8u + (u32)__builtin_amdgcn_readfirstlane(threadIdx.x >> 6);
        const int rounds = slot < PULL_SLOTS ? 2 : 1;
        u32x4 r0, r1;
        load_unit(a, k0, threadIdx.x & 63, r0, r1);
        const u32 prev = __hip_atomic_load(cnt, __ATOMIC_RELAXED, __HIP_MEMORY_SCOPE_AGENT);
        SCHED;
        const u32 call = (u32)__builtin_amdgcn_readfirstlane(prev) + 1u;
        for (int r = 0; r < rounds; ++r) {
            if (r) load_unit(a, k0 + GEMM_SLOTS * 64u, threadIdx.x & 63, r0, r1);
            stage_out(a, lds, blockIdx.x + r * GEMM_SLOTS * 8, call, r, r0, r1);
        }
        if (__builtin_amdgcn_readfirstlane(threadIdx.x) == 0) st32_agent_lane0(cnt, call);
        STAMP(1);
        gemm(a, call, xcd, slot, lds);
    } else {
        const u32 call = (u32)__builtin_amdgcn_readfirstlane((int)ld32_agent(cnt)) + 1u;
        __syncthreads();
        if (__builtin_amdgcn_readfirstlane(threadIdx.x) == 0) st32_agent_lane0(cnt, call);
        STAMP(1);
        pull(a, call, (slot - GEMM_SLOTS) * 8 + xcd, lds);
    }
}

extern "C" void kernel_launch(int rank, int nranks, ncclComm_t comm, hipStream_t stream,
                              void* const* d_in, const int* in_sizes, int n_in,
                              void* d_out, int out_size, const int* in_dtypes, int out_dtype,
                              void* const* d_out_all, void* const* d_ws_all, size_t ws_bytes) {
    if (nranks != NR || (unsigned)rank >= (unsigned)NR || n_in != 4 || ws_bytes < WS_NEED || out_size != DIM * DIM ||
        in_sizes[0] != DIM * KS || in_sizes[1] != DIM * KS) {
        fprintf(stderr, "kernel_launch: unexpected problem shape\n");
        return;
    }
    static bool attr_set[NR] = {};
    if (!attr_set[rank]) {
        HIP_CHECK(hipFuncSetAttribute((const void*)fused_allgather_gemm, hipFuncAttributeMaxDynamicSharedMemorySize, LDS_BYTES));
        attr_set[rank] = true;
    }
    Args a;
    a.x = (const unsigned char*)d_in[0]; a.w = (const unsigned char*)d_in[1];
    a.sx = (const float*)d_in[2]; a.sw = (const float*)d_in[3];
    a.out = (float*)d_out;
    for (int p = 0; p < NR; ++p) a.ws[p] = (char*)d_ws_all[p];
    a.me = a.ws[rank]; a.rank = rank; a.pad = 0;
    fused_allgather_gemm<<<256, 512, LDS_BYTES, stream>>>(a);
    HIP_CHECK(hipGetLastError());
}
```

```cpp
#include <hip/hip_runtime.h>
#include <rccl/rccl.h>
#include <cstdio>
#include <type_traits>

#define HIP_CHECK(expr) do { hipError_t _e = (expr); if (_e != hipSuccess) { \
    fprintf(stderr, "HIP error %s at %s:%d\n", hipGetErrorString(_e), __FILE__, __LINE__); \
    return; } } while (0)

#define LAS __attribute__((address_space(3)))
#define GAS __attribute__((address_space(1)))
typedef unsigned u32;
typedef unsigned long long u64;
typedef int v2i __attribute__((ext_vector_type(2)));
typedef int v4i __attribute__((ext_vector_type(4)));
typedef int v8i __attribute__((ext_vector_type(8)));
typedef float v2f __attribute__((ext_vector_type(2)));
typedef float v4f __attribute__((ext_vector_type(4)));
typedef unsigned u32x2 __attribute__((ext_vector_type(2)));
typedef unsigned u32x4 __attribute__((ext_vector_type(4)));

namespace {
constexpr int NR = 8;
constexpr int DIM = 4096;
constexpr int KS = DIM / NR;
constexpr int NT = 32;
constexpr u32 NUNITS = NT * 64, NBLOCKS = NUNITS / 2;
constexpr size_t STAGE_OFF = 0;
constexpr size_t SRC_BYTES = (size_t)NUNITS * 2048;
constexpr size_t OUTBOX_OFF = STAGE_OFF + NR * SRC_BYTES;
constexpr size_t VBOX_OFF = OUTBOX_OFF + 2 * SRC_BYTES;
constexpr size_t VBOX_HALF = (size_t)NBLOCKS * 1024;
constexpr size_t RDY_OFF = VBOX_OFF + 2 * VBOX_HALF + 4096;
constexpr u32 RDY_SRC = 65536 + 2304, RDY_DENSE = NR * RDY_SRC, RDY_DENSE_SRC = 1280;
constexpr size_t RDY_HALF = RDY_DENSE + NR * RDY_DENSE_SRC;
#define DONE_SHIFT 7
constexpr size_t DONE_OFF = RDY_OFF + 2 * RDY_HALF;
constexpr size_t CLAIM_OFF = DONE_OFF + ((size_t)NUNITS << DONE_SHIFT);
constexpr size_t CNT_OFF = CLAIM_OFF + NR * 128;
constexpr size_t TMO_OFF = CNT_OFF + 256 * 4;
constexpr size_t DBG_OFF = TMO_OFF + 128;
constexpr size_t WS_NEED = DBG_OFF + 256 * 64 + 256 * 8 * 80 * 8;
#define OUT_FLAGS "sc0 sc1"
#define POLL_SLEEP 4
constexpr u32 VMID = 448, VWIN = 192;
constexpr u32 H_RAW = 0x80000000u, H_BIG = 0x40000000u;
constexpr u64 DEADLINE = 150000000ull;
constexpr int BM = 256, BN = 320, XS = BM * 16, WS = BN * 16, XB = 8 * XS, WB = 8 * WS, STG = XB + WB;
constexpr int LDS_BYTES = 2 * STG + 512;
constexpr int GEMM_SLOTS = 26, PULL_SLOTS = 6;

#define STAG_A 0
#define RAW_BLOCKS 32
struct Order { unsigned short blk[NBLOCKS]; unsigned char raw[256]; };
constexpr int lag16(int p) { return STAG_A * 16 * p / 15; }
constexpr Order make_order() {
    Order r = {};
    int n = 0;
    for (int key = 0; n < (int)NBLOCKS; ++key)
        for (int p = 0; p < 16; ++p) {
            const int d = key - lag16(p);
            if (d >= 0 && d % 16 == 0 && d / 16 < NT)
                for (int o = 0; o < 2; ++o) {
                    const int blk = (d / 16) * 32 + o * 16 + p;
                    if (n < RAW_BLOCKS) r.raw[(blk >> 5) * 8 + ((blk & 15) >> 1)] = 1;
                    r.blk[n++] = (unsigned short)blk;
                }
        }
    for (int i = 64; i < 256; ++i) r.raw[i] = 0;
    for (int i = 0; i < (int)NBLOCKS; ++i) if (r.raw[(r.blk[i] >> 5) * 8 + ((r.blk[i] & 15) >> 1)]) r.blk[i] += NBLOCKS;
    return r;
}
__device__ constexpr Order ORDER = make_order();
constexpr u64 raw_mask() { const Order o = make_order(); u64 m = 0; for (int i = 0; i < 64; ++i) if (o.raw[i]) m |= 1ull << i; return m; }
constexpr u64 RAW_TASKS = raw_mask();

struct Args {
    const unsigned char* x; const unsigned char* w; const float* sx; const float* sw; float* out;
    char* me; char* ws[NR]; int rank; int pad;
};

__device__ __forceinline__ void st16_sys(void* p, u32x4 v) {
    asm volatile("global_store_dwordx4 %0, %1, off sc0 sc1\n\ts_nop 1" ::"v"(p), "v"(v) : "memory");
}
__device__ __forceinline__ void st16_agent(void* p, u32x4 v) {
    asm volatile("global_store_dwordx4 %0, %1, off sc1\n\ts_nop 1" ::"v"(p), "v"(v) : "memory");
}
__device__ __forceinline__ u32 ld32_agent(const void* p) {
    u32 v;
    asm volatile("global_load_dword %0, %1, off sc1\n\ts_waitcnt vmcnt(0)" : "=&v"(v) : "v"(p) : "memory");
    return v;
}
__device__ __forceinline__ void st32_agent_lane0(void* p, u32 v) {
    asm volatile("s_mov_b64 exec, 1\n\tglobal_store_dword %0, %1, off sc1\n\ts_mov_b64 exec, -1" ::"v"(p), "v"(v) : "memory");
}
__device__ __forceinline__ bool give_up(u32* tmo, u32 call, u64 t0, u32& spins) {
    if ((++spins & 15u) != 0u) return false;
    if ((u32)__builtin_amdgcn_readfirstlane((int)ld32_agent(tmo)) == call || __builtin_amdgcn_s_memrealtime() - t0 > DEADLINE) {
        st32_agent_lane0(tmo, call);
        return true;
    }
    return false;
}
template <class T> __device__ __forceinline__ T* uniform_ptr(T* p) {
    const u64 v = (u64)p;
    return (T*)(((u64)(u32)__builtin_amdgcn_readfirstlane((int)(u32)(v >> 32)) << 32) | (u32)__builtin_amdgcn_readfirstlane((int)(u32)v));
}
__device__ __forceinline__ v8i ldfrag(LAS const char* p, int off, int stride) {
    v4i a = *(LAS const v4i*)(p + off), b = *(LAS const v4i*)(p + off + stride);
    return __builtin_shufflevector(a, b, 0, 1, 2, 3, 4, 5, 6, 7);
}
#define SCHED __builtin_amdgcn_sched_barrier(0)
#define STAMP(i) do { } while (0)
#define WSTAMP(i) do { } while (0)

__device__ __forceinline__ u32 wave_incl_scan(u32 x) {
    x += (u32)__builtin_amdgcn_update_dpp(0, (int)x, 0x111, 0xf, 0xf, false);
    x += (u32)__builtin_amdgcn_update_dpp(0, (int)x, 0x112, 0xf, 0xf, false);
    x += (u32)__builtin_amdgcn_update_dpp(0, (int)x, 0x114, 0xf, 0xf, false);
    x += (u32)__builtin_amdgcn_update_dpp(0, (int)x, 0x118, 0xf, 0xf, false);
    x += (u32)__builtin_amdgcn_update_dpp(0, (int)x, 0x142, 0xa, 0xf, false);
    x += (u32)__builtin_amdgcn_update_dpp(0, (int)x, 0x143, 0xc, 0xf, false);
    return x;
}
__device__ __forceinline__ u32 rotr(u32 x, u32 n) { return __builtin_amdgcn_alignbit(x, x, n); }
__device__ __forceinline__ u32 bfi(u32 m, u32 a, u32 b) { return __builtin_amdgcn_bitop3_b32(m, a, b, 0xca); }
__device__ __forceinline__ u32 and3(u32 a, u32 b, u32 c) { return __builtin_amdgcn_bitop3_b32(a, b, c, 0x80); }
__device__ __forceinline__ u32 lds32(u32 addr) { return *(LAS const u32*)(size_t)addr; }
__device__ __forceinline__ u32x4 make_rsrc(const void* p, u32 bytes) {
    const u64 v = (u64)p;
    return (u32x4){(u32)__builtin_amdgcn_readfirstlane((int)(u32)v), (u32)__builtin_amdgcn_readfirstlane((int)(u32)(v >> 32)) & 0xffffu, bytes, 0x00020000u};
}

constexpr int P0_VAR = 2560, P0_LUT = P0_VAR + 1536, P0_WAVE = P0_LUT + 128;

template <int O>
__device__ __forceinline__ void pack_unit(const u32 (&d)[8], char* slot, char* vslot, u32 h, LAS char* img, u32 lut, int lane) {
    constexpr int SH = O ? 2 : 3;
    constexpr u32 EM = O ? 0x1f1f1f1fu : 0x0f0f0f0fu;
    u32 ex[8], sum = 0;
#pragma unroll
    for (int j = 0; j < 8; ++j) ex[j] = (d[j] >> SH) & EM;
#pragma unroll
    for (int j = 0; j < 8; j += 2) sum = __builtin_amdgcn_udot4(ex[j], 0x01010101u, sum, false);
    const u32 tot = (u32)__builtin_amdgcn_readlane((int)wave_incl_scan(sum), 63);
    const u32 lo = tot > 50u ? (tot - 50u) >> 10 : 0u;
    const u32 bias = (0x80u - lo) * 0x01010101u;
    u32 P[2] = {0u, 0u}, X = 0, n2 = 0, n3 = 0, b3 = 0, q[8], n2j[8];
#pragma unroll
    for (int j = 0; j < 8; ++j) {
        const u32 a7 = ex[j] + bias;
        const u32 t = ((~a7 | (a7 - 0x03030303u)) & 0x80808080u) >> 7;
        P[j >> 2] |= ((a7 | t | (t << 1)) & 0x03030303u) << (2 * (j & 3));
        const u32 sc = a7 & ((a7 | 0x80808080u) >> 5) & 0x07070707u;
        X = (X >> 1) | ((~(a7 + 0x07070707u) | (a7 - 0x04040404u)) & 0x80808080u);
        const u32 cz = __builtin_amdgcn_perm(0x07070707u, sc, lds32(__builtin_amdgcn_udot4(t, 0x20100804u, lut, false)));
        q[j] = __builtin_amdgcn_udot4(__builtin_amdgcn_perm(0x00010303u, 0x03030302u, cz), 0x40100401u, 0u, false);
        const u32 f = __builtin_amdgcn_perm(0x00000101u, 0x01010100u, cz);
        const u32 c3 = __builtin_amdgcn_perm(0u, cz, lds32(__builtin_amdgcn_udot4(f, 0x20100804u, lut + 64u, false)));
        b3 |= __builtin_amdgcn_udot4(c3, 0x003e0801u, c3 >> 15, false) << (n3 & 31u);
        n3 = __builtin_amdgcn_udot4(f, 0x03030303u, n3, false);
        n2j[j] = n2;
        n2 = __builtin_amdgcn_udot4(t, 0x02020202u, n2, false);
    }
    const u32 qa = q[0] | (q[1] << n2j[1]) | (q[2] << n2j[2]) | (q[3] << n2j[3]);
    const u32 qb = q[4] | (q[5] << (n2j[5] - n2j[4])) | (q[6] << (n2j[6] - n2j[4])) | (q[7] << (n2j[7] - n2j[4]));
    const u64 b2 = (u64)qa | ((u64)qb << n2j[4]);
    const u32 both = n2 | (n3 << 16), incl = wave_incl_scan(both), pre = incl - both;
    const u32 total = (u32)__builtin_amdgcn_readlane((int)incl, 63);
    const u32 len2 = (((total & 0xffffu) + 31u) >> 5) << 2, len3 = (((total >> 16) + 31u) >> 5) << 2;
    const u32 used = 4u + len2 + len3;
    bool raw = __any(n3 > 30u) || used > VMID - 16u;
    u32 nexc = 0;
    if (!raw) {
        const u32 p2 = pre & 0xffffu, p3 = pre >> 16;
        LAS u32* w2 = (LAS u32*)(img + (h ? VMID + 4u : VMID - 4u - len2) + ((p2 >> 5) << 2));
        LAS u32* w3 = (LAS u32*)(img + (h ? VMID + 4u + len2 : VMID - used) + ((p3 >> 5) << 2));
        const u64 u2 = b2 << (p2 & 31u), u3 = (u64)b3 << (p3 & 31u);
        __hip_atomic_fetch_or(w2, (u32)u2, __ATOMIC_RELAXED, __HIP_MEMORY_SCOPE_WAVEFRONT);
        __hip_atomic_fetch_or(w2 + 1, (u32)(u2 >> 32), __ATOMIC_RELAXED, __HIP_MEMORY_SCOPE_WAVEFRONT);
        __hip_atomic_fetch_or(w2 + 2, (u32)(b2 >> 32) >> 1 >> (31u - (p2 & 31u)), __ATOMIC_RELAXED, __HIP_MEMORY_SCOPE_WAVEFRONT);
        __hip_atomic_fetch_or(w3, (u32)u3, __ATOMIC_RELAXED, __HIP_MEMORY_SCOPE_WAVEFRONT);
        __hip_atomic_fetch_or(w3 + 1, (u32)(u3 >> 32), __ATOMIC_RELAXED, __HIP_MEMORY_SCOPE_WAVEFRONT);
        u64 bal;
        while ((bal = __ballot(X != 0u)) != 0ull) {
            const u32 slotn = nexc + __builtin_amdgcn_mbcnt_hi((u32)(bal >> 32), __builtin_amdgcn_mbcnt_lo((u32)bal, 0u));
            nexc += (u32)__builtin_popcountll(bal);
            if (nexc > 64u || used + 2u * nexc > VMID - 16u) { raw = true; break; }
            if (X != 0u) {
                const u32 k = (u32)__builtin_ctz(X), j = k & 7u, i8 = k & 24u;
                const u32 e01 = (j & 1) ? ex[1] : ex[0], e23 = (j & 1) ? ex[3] : ex[2], e45 = (j & 1) ? ex[5] : ex[4], e67 = (j & 1) ? ex[7] : ex[6];
                const u32 e03 = (j & 2) ? e23 : e01, e47 = (j & 2) ? e67 : e45, ej = (j & 4) ? e47 : e03;
                const u32 n01 = (j & 1) ? n2j[1] : n2j[0], n23 = (j & 1) ? n2j[3] : n2j[2], n45 = (j & 1) ? n2j[5] : n2j[4], n67 = (j & 1) ? n2j[7] : n2j[6];
                const u32 n03 = (j & 2) ? n23 : n01, n47 = (j & 2) ? n67 : n45, nj = (j & 4) ? n47 : n03;
                const u32 Pj = ((j & 4) ? P[1] : P[0]) >> (2u * (j & 3u));
                const u32 pos = ((p2 + nj) >> 1) + (u32)__builtin_popcount(Pj & (Pj >> 1) & 0x01010101u & ((1u << i8) - 1u));
                *(LAS unsigned short*)(img + (h ? VMID + used + 2u * slotn : VMID - used - 2u - 2u * slotn)) = (unsigned short)(pos | (((ej >> i8) & 31u) << 11));
                X &= X - 1u;
            }
        }
    }
    if (raw) {
        st16_sys(slot + lane * 16, (u32x4){d[0], d[1], d[2], d[3]});
        st16_sys(slot + 1024 + lane * 16, (u32x4){d[4], d[5], d[6], d[7]});
        asm volatile("s_mov_b64 exec, 1\n\tglobal_store_dword %0, %1, off sc0 sc1\n\ts_mov_b64 exec, -1" ::"v"(vslot + (h ? VMID : VMID - 4u)), "v"(H_RAW) : "memory");
        return;
    }
    const u32 nvar = (used + 2u * nexc + 15u) >> 4;
    if (lane == 0) *(LAS u32*)(img + (h ? VMID : VMID - 4u)) = lo | (nexc << 5) | (len3 << 12) | (nvar * 16u > VWIN ? H_BIG : 0u);
    const u32 first = h ? VMID : VMID - nvar * 16u;
    const u32x4 v = *(LAS const u32x4*)(img + first + lane * 16);
    if (O == 0) {
        u32x4 sm;
#pragma unroll
        for (int k = 0; k < 4; ++k) sm[k] = bfi(0x87878787u, d[2 * k], rotr(d[2 * k + 1], 28));
        st16_sys(slot + lane * 16, sm);
        asm volatile("global_store_dwordx2 %0, %1, off offset:1024 sc0 sc1\n\ts_nop 1" ::"v"(slot + lane * 8), "v"((u32x2){P[0], P[1]}) : "memory");
    } else {
        u32x4 sm;
        sm[0] = bfi(0x60606060u, d[6] << 5, bfi(0x83838383u, d[0], rotr(d[1], 29)));
        sm[1] = bfi(0x60606060u, d[7] << 5, bfi(0x83838383u, d[2], rotr(d[3], 29)));
        sm[2] = bfi(0x60606060u, bfi(0x40404040u, d[6] >> 1, d[7] >> 2), bfi(0x83838383u, d[4], rotr(d[5], 29)));
        sm[3] = P[0];
        st16_sys(slot + lane * 16, sm);
        const u32 up = __builtin_amdgcn_permlane32_swap(P[1], P[1], false, false)[1];
        if (lane < 32) asm volatile("global_store_dwordx2 %0, %1, off offset:1024 sc0 sc1\n\ts_nop 1" ::"v"(slot + lane * 8), "v"((u32x2){P[1], up}) : "memory");
    }
    if ((u32)lane < nvar) st16_sys(vslot + first + lane * 16, v);
}

__device__ __forceinline__ u32 spread3(u32 v) {
    const u32 x = bfi(0x3fu, v, v << 10);
    return bfi(0x00070007u, x, x << 5) & 0x07070707u;
}
__device__ __forceinline__ u32 spread2(u32 v) {
    const u32 x = bfi(0xfu, v, v << 12);
    return bfi(0x00030003u, x, x << 6) & 0x03030303u;
}

template <int O, class F>
__device__ __forceinline__ void unpack_unit(const u32x4 A, const u32x2 B, const u32 hdr, const bool H, u32 (&d)[8], u32 mid, u32 exp, u32 lut, int lane, F&& midway) {
    constexpr int SH = O ? 2 : 3;
    const u32 P[2] = {O ? A[3] : B[0], O ? __builtin_amdgcn_permlane32_swap(B[0], B[1], false, false)[0] : B[1]};
    const u32 lo = hdr & 31u, nexc = (hdr >> 5) & 127u, len3 = min((hdr >> 12) & 0x1fcu, VMID - 16u);
    const u32 E[2] = {P[0] & (P[0] >> 1) & 0x55555555u, P[1] & (P[1] >> 1) & 0x55555555u};
    const u32 cnt = (u32)__builtin_popcount(E[0]) + (u32)__builtin_popcount(E[1]), incl = wave_incl_scan(cnt);
    const u32 n = min((u32)__builtin_amdgcn_readlane((int)incl, 63), 1712u);
    const u32 prim = ((lo * 0x01010101u + 0x00020100u) << SH);
    const u32 b8 = ((lo + 24u) & 31u) * 0x01010101u;
    const u32 t3lo = ((b8 + 0x03020100u) & 0x1f1f1f1fu) << SH, t3hi = ((b8 + 0x07060504u) & 0x1f1f1f1fu) << SH;
    const u32 t2 = ((b8 + 0x000b0607u) & 0x001f1f1fu) << SH;
    const u32 len2 = ((n + 15u) >> 4) << 2;
    const u32 s2 = H ? mid + 4u : mid - 4u - len2, s3 = H ? s2 + len2 : s2 - len3;
    u32 m3 = 0;
    for (u32 base = 0; base < n; base += 512u) {
        u32 c = *(LAS const unsigned short*)(size_t)(s2 + (base >> 2) + (u32)lane * 2u);
        if (base + (u32)lane * 8u >= n) c = 0u;
        const u32 c0 = spread2(c), c1 = spread2(c >> 8);
        const u32 f0 = and3(c0, c0 >> 1, 0x01010101u), f1 = and3(c1, c1 >> 1, 0x01010101u);
        const u32 k0 = __builtin_amdgcn_udot4(f0, 0x03030303u, 0u, false), k = __builtin_amdgcn_udot4(f1, 0x03030303u, k0, false);
        const u32 in3 = wave_incl_scan(k), bit = min(m3 + in3 - k, 3583u);
        LAS const volatile u32* wp = (LAS const volatile u32*)(size_t)(s3 + ((bit >> 5) << 2));
        const u32 g0 = lds32(__builtin_amdgcn_udot4(f0, 0x20100804u, lut, false)), g1 = lds32(__builtin_amdgcn_udot4(f1, 0x20100804u, lut, false));
        const u32 w0 = wp[0], w1 = wp[1];
        const u32 w = __builtin_amdgcn_alignbit(w1, w0, bit);
        const u32 e0 = __builtin_amdgcn_perm(t3hi, t3lo, spread3(w)), e1 = __builtin_amdgcn_perm(t3hi, t3lo, spread3(w >> k0));
        *(LAS u32x2*)(size_t)(exp + base + (u32)lane * 8u) = (u32x2){__builtin_amdgcn_perm(e0, t2, c0 ^ g0), __builtin_amdgcn_perm(e1, t2, c1 ^ g1)};
        m3 += (u32)__builtin_amdgcn_readlane((int)in3, 63);
    }
    if (nexc) {
        if ((u32)lane < nexc) {
            const u32 e = *(LAS const unsigned short*)(size_t)(H ? s3 + len3 + 2u * (u32)lane : s3 - 2u - 2u * (u32)lane);
            *(LAS unsigned char*)(size_t)(exp + (e & 0x7ffu)) = (unsigned char)((e >> 11) << SH);
        }
    }
    midway();
    u32 ptr[8], dep[8], ex[8];
    u32 at = exp + incl - cnt;
#pragma unroll
    for (int j = 0; j < 8; ++j) {
        const int g = j & 3;
        const u32 Eh = g < 2 ? E[j >> 2] : E[j >> 2] >> 4;
        const u32 m = Eh & ((g & 1) ? 0x04040404u : 0x01010101u);
        dep[j] = lds32(__builtin_amdgcn_udot4(m, (g & 1) ? 0x08040201u : 0x20100804u, lut, false));
        ptr[j] = at;
        at += (u32)__builtin_popcount(m);
    }
    u32x2 wn[8];
    asm volatile("ds_read2_b32 %0, %8 offset1:1\n\tds_read2_b32 %1, %9 offset1:1\n\tds_read2_b32 %2, %10 offset1:1\n\tds_read2_b32 %3, %11 offset1:1\n\t"
                 "ds_read2_b32 %4, %12 offset1:1\n\tds_read2_b32 %5, %13 offset1:1\n\tds_read2_b32 %6, %14 offset1:1\n\tds_read2_b32 %7, %15 offset1:1\n\t"
                 "s_waitcnt lgkmcnt(0)"
                 : "=&v"(wn[0]), "=&v"(wn[1]), "=&v"(wn[2]), "=&v"(wn[3]), "=&v"(wn[4]), "=&v"(wn[5]), "=&v"(wn[6]), "=&v"(wn[7])
                 : "v"(ptr[0] & ~3u), "v"(ptr[1] & ~3u), "v"(ptr[2] & ~3u), "v"(ptr[3] & ~3u), "v"(ptr[4] & ~3u), "v"(ptr[5] & ~3u), "v"(ptr[6] & ~3u), "v"(ptr[7] & ~3u)
                 : "memory");
#pragma unroll
    for (int j = 0; j < 8; ++j)
        ex[j] = __builtin_amdgcn_perm(__builtin_amdgcn_alignbyte(wn[j][1], wn[j][0], ptr[j]), prim, ((P[j >> 2] >> (2 * (j & 3))) & 0x03030303u) ^ dep[j]);
    if (O == 0) {
#pragma unroll
        for (int k = 0; k < 4; ++k) { d[2 * k] = (A[k] & 0x87878787u) | ex[2 * k]; d[2 * k + 1] = (rotr(A[k], 4) & 0x87878787u) | ex[2 * k + 1]; }
    } else {
#pragma unroll
        for (int k = 0; k < 3; ++k) { d[2 * k] = (A[k] & 0x83838383u) | ex[2 * k]; d[2 * k + 1] = (rotr(A[k], 3) & 0x83838383u) | ex[2 * k + 1]; }
        d[6] = (bfi(0x80808080u, A[2] << 1, A[0] >> 5) & 0x83838383u) | ex[6];
        d[7] = (bfi(0x80808080u, A[2] << 2, A[1] >> 5) & 0x83838383u) | ex[7];
    }
}

__host__ __device__ constexpr u32 task_unit(u32 b, u32 wid) { return (b >> 3) * 64u + (wid >> 2) * 32u + (b & 7u) * 4u + (wid & 3u); }
__host__ __device__ constexpr u32 unit_task(u32 k) { return (k >> 6) * 8u + ((k & 31u) >> 2); }
__device__ __forceinline__ u32 stage_off(u32 k) { return ((k >> 5) << 19) | ((k & 31u) << 11); }

__device__ __forceinline__ void load_unit(const Args& a, u32 k, int lane, u32x4& r0, u32x4& r1) {
    const u32 t = k >> 6, rows = (k & 31u) * 128u;
    const unsigned char* src = (k & 32u) ? a.w + (size_t)(16u * t + (lane >> 3)) * DIM + rows + (lane & 7) * 16
                                         : a.x + (size_t)(rows + lane) * KS + 16u * t;
    r0 = *(const u32x4*)src; r1 = *(const u32x4*)(src + 32768);
}
__device__ __forceinline__ u32 stage_out(const Args& a, LAS char* lds, const u32 b, const bool plain, const u32 prev, const u32x4 r0, const u32x4 r1) {
    const int tid = threadIdx.x, wid = __builtin_amdgcn_readfirstlane(tid >> 6), lane = tid & 63;
    const u32 k = task_unit(b, (u32)wid), o = (k >> 5) & 1u;
    char* myws = a.me;
    LAS char* wl = lds + wid * P0_WAVE;
    {
        u32 sel = 0x04040404u; int n = 0;
#pragma unroll
        for (int i = 0; i < 4; ++i) if ((lane >> i) & 1) { sel = (sel & ~(0xffu << (8 * n))) | ((u32)i << (8 * n)); ++n; }
        if (lane & 16) sel |= (sel & 0x04040404u) << 1;
        if (lane < 32) *(LAS u32*)(wl + P0_LUT + lane * 4) = sel;
        *(LAS u32x4*)(wl + P0_VAR + lane * 16) = (u32x4){0u, 0u, 0u, 0u};
    }
    SCHED;
    u32 d[8];
    if (o == 0) {
#pragma unroll
        for (int j = 0; j < 4; ++j) { d[j] = r0[j]; d[4 + j] = r1[j]; }
    } else {
        *(LAS u32x4*)(wl + (lane >> 3) * 160 + (lane & 7) * 16) = r0;
        *(LAS u32x4*)(wl + (8 + (lane >> 3)) * 160 + (lane & 7) * 16) = r1;
        LAS char* tb = wl + ((lane & 15) >> 1) * 160 + (lane >> 4) * 16 + (lane & 1) * 8;
        asm volatile("" ::: "memory");
        const v2i c00 = __builtin_amdgcn_ds_read_tr8_b64_v2i32((LAS v2i*)tb);
        const v2i c01 = __builtin_amdgcn_ds_read_tr8_b64_v2i32((LAS v2i*)(tb + 8 * 160));
        const v2i c10 = __builtin_amdgcn_ds_read_tr8_b64_v2i32((LAS v2i*)(tb + 64));
        const v2i c11 = __builtin_amdgcn_ds_read_tr8_b64_v2i32((LAS v2i*)(tb + 8 * 160 + 64));
        const u32 g0 = (k & 31u) * 2u, g1 = g0 + 1u;
        const int mixed = ((lane & 15) * 4 + (lane >> 4)) * 4;
        const int s0 = g0 % 5u == 4u ? lane * 4 : mixed, s1 = (g1 % 5u == 4u || g1 == 63u) ? lane * 4 : mixed;
        d[0] = __builtin_amdgcn_ds_bpermute(s0, c00[0]); d[1] = __builtin_amdgcn_ds_bpermute(s0, c00[1]);
        d[2] = __builtin_amdgcn_ds_bpermute(s0, c01[0]); d[3] = __builtin_amdgcn_ds_bpermute(s0, c01[1]);
        d[4] = __builtin_amdgcn_ds_bpermute(s1, c10[0]); d[5] = __builtin_amdgcn_ds_bpermute(s1, c10[1]);
        d[6] = __builtin_amdgcn_ds_bpermute(s1, c11[0]); d[7] = __builtin_amdgcn_ds_bpermute(s1, c11[1]);
    }
    const u32 call = (u32)__builtin_amdgcn_readfirstlane((int)prev) + 1u;
    const u32 par = call & 1u;
    char* slot = myws + OUTBOX_OFF + par * SRC_BYTES + (size_t)k * 2048;
    char* own = myws + STAGE_OFF + (size_t)a.rank * 65536 + stage_off(k) + lane * 16;
    const u32x4 w0 = {d[0], d[1], d[2], d[3]}, w1 = {d[4], d[5], d[6], d[7]};
    if (plain) { st16_sys(slot + lane * 16, w0); st16_sys(slot + 1024 + lane * 16, w1); }
    st16_agent(own, w0); st16_agent(own + 1024, w1);
    if (!plain) {
        char* vslot = myws + VBOX_OFF + par * VBOX_HALF + (size_t)(k >> 1) * 1024;
        if (o == 0) pack_unit<0>(d, slot, vslot, k & 1u, wl + P0_VAR, (u32)(size_t)(wl + P0_LUT), lane);
        else pack_unit<1>(d, slot, vslot, k & 1u, wl + P0_VAR, (u32)(size_t)(wl + P0_LUT), lane);
    }
    char* w[NR];
#pragma unroll
    for (int p = 0; p < NR; ++p) w[p] = a.ws[p];
    asm volatile("" : "+s"(w[0]), "+s"(w[1]), "+s"(w[2]), "+s"(w[3]), "+s"(w[4]), "+s"(w[5]), "+s"(w[6]), "+s"(w[7]));
    const u32 off = (u32)(RDY_OFF + par * RDY_HALF) + (lane == 0 ? (u32)a.rank * RDY_SRC + b * 256u : RDY_DENSE + (u32)a.rank * RDY_DENSE_SRC + b * 4u);
    asm volatile("s_waitcnt vmcnt(0)" ::: "memory");
    __syncthreads();
    if (wid == 0)
        asm volatile("s_mov_b64 exec, 3\n\tglobal_store_dword %0, %1, %2 sc0 sc1\n\tglobal_store_dword %0, %1, %3 sc0 sc1\n\tglobal_store_dword %0, %1, %4 sc0 sc1\n\t"
                     "global_store_dword %0, %1, %5 sc0 sc1\n\tglobal_store_dword %0, %1, %6 sc0 sc1\n\tglobal_store_dword %0, %1, %7 sc0 sc1\n\t"
                     "global_store_dword %0, %1, %8 sc0 sc1\n\tglobal_store_dword %0, %1, %9 sc0 sc1\n\ts_mov_b64 exec, -1"
                     ::"v"(off), "v"(call), "s"(w[0]), "s"(w[1]), "s"(w[2]), "s"(w[3]), "s"(w[4]), "s"(w[5]), "s"(w[6]), "s"(w[7]) : "memory");
    st32_agent_lane0((u32*)(myws + DONE_OFF + ((size_t)k << DONE_SHIFT)) + a.rank, call);
    return call;
}

constexpr int PL_WAVE = 512 + 1536 + 2560 + 64;
#define PULL_WPW 7
constexpr u32 PULL_WAVES = PULL_SLOTS * 8 * PULL_WPW;

__device__ __forceinline__ void pull(const Args& a, u32* cnt, int c, LAS char* lds) {
    const int tid = threadIdx.x, wid = __builtin_amdgcn_readfirstlane(tid >> 6), lane = tid & 63;
    const u32 ord = ((const u32*)ORDER.blk)[tid];
    const u32 call = (u32)__builtin_amdgcn_readfirstlane((int)ld32_agent(cnt)) + 1u;
    const u32 gw = (u32)c * PULL_WPW + (u32)wid, rk = gw / 7u, link = PULL_WPW == 7 ? ((u32)wid + (u32)c) % 7u : gw % 7u;
    const u32 nw = (PULL_WAVES + 6u - link) / 7u;
    const u32 s = ((u32)a.rank + 1u + link) & 7u, par = call & 1u;
    char* myws = a.me;
    u32* tmo = (u32*)(myws + TMO_OFF);
    const u32 img = (u32)(size_t)(lds + 2048 + wid * PL_WAVE + 512), exp = img + 1536u, lut = exp + 2560u;
    *(LAS u32*)(lds + tid * 4) = ord;
    if (lane < 16) {
        u32 dep = 0; u32 n = 0;
#pragma unroll
        for (int i = 0; i < 4; ++i) if ((lane >> i) & 1) { dep |= (3u ^ (4u + n)) << (8 * i); ++n; }
        *(LAS u32*)(size_t)(lut + lane * 4) = dep;
    }
    __builtin_amdgcn_s_waitcnt(0x0F70);
    __syncthreads();
    if (wid == 0) st32_agent_lane0(cnt, call);
    if (wid >= PULL_WPW) return;
    STAMP(1);
    const u32x4 rbox = make_rsrc(a.ws[s] + OUTBOX_OFF + par * SRC_BYTES, (u32)(RDY_OFF - OUTBOX_OFF - par * SRC_BYTES));
    const u32x4 rstage = make_rsrc(myws + STAGE_OFF + s * 65536u, (u32)(NR * SRC_BYTES) - s * 65536u);
    const u32x4 rdone = make_rsrc(myws + DONE_OFF + s * 4u, NUNITS << DONE_SHIFT);
    const u32 vdelta = (u32)(VBOX_OFF - OUTBOX_OFF) + par * (u32)VBOX_HALF - par * (u32)SRC_BYTES;
    const char* rdy = myws + RDY_OFF + par * RDY_HALF;
    u32* claim = (u32*)(myws + CLAIM_OFF) + link * 32u;
    const u32 tbase = nw - (call - 1u) * NBLOCKS;
    const u32 l16 = (u32)lane * 16u, l8 = (u32)lane * 8u;
    u32 vcall = call; asm volatile("" : "+v"(vcall));
    u32 known = 0;
    auto describe = [&](u32 ticket, u32& kb, bool refresh) -> bool {
        kb = (u32)__builtin_amdgcn_readfirstlane(*(LAS const unsigned short*)(lds + ticket * 2u)) * 2u;
        const u32 b = unit_task(kb & 2047u);
        if (__builtin_expect(b >= known, 0)) {
            if (refresh) {
                u32x4 f;
                asm volatile("global_load_dwordx4 %0, %1, off sc0 sc1\n\ts_waitcnt vmcnt(0)" : "=&v"(f) : "v"(rdy + RDY_DENSE + s * RDY_DENSE_SRC + l16) : "memory");
                const u64 bad = __ballot(f[0] != call || f[1] != call || f[2] != call || f[3] != call);
                known = bad ? (u32)__builtin_ctzll(bad) * 4u : 256u;
                if (b < known) return true;
            }
            const u64 t0 = __builtin_amdgcn_s_memrealtime();
            for (u32 spins = 0;;) {
                u32 f;
                asm volatile("s_mov_b64 exec, 1\n\tglobal_load_dword %0, %1, off sc0 sc1\n\ts_mov_b64 exec, -1\n\ts_waitcnt vmcnt(0)" : "=&v"(f) : "v"(rdy + s * RDY_SRC + b * 256u) : "memory");
                const u32 seq = (u32)__builtin_amdgcn_readfirstlane((int)f);
                if (seq == call) break;
                bool out = (int)(seq - call) > 0;
                if (out) st32_agent_lane0(tmo, call); else out = give_up(tmo, call, t0, spins);
                if (out) {
                    while (ticket < NBLOCKS) {
                        u32 r;
                        asm volatile("s_mov_b64 exec, 1\n\tglobal_atomic_add %0, %1, %2, off sc0\n\ts_mov_b64 exec, -1\n\ts_waitcnt vmcnt(0)" : "=&v"(r) : "v"(claim), "v"(1u) : "memory");
                        ticket = (u32)__builtin_amdgcn_readfirstlane((int)r) + tbase;
                    }
                    return false;
                }
                __builtin_amdgcn_s_sleep(POLL_SLEEP);
            }
        }
        return true;
    };
#define ZONE "v240", "v241", "v242", "v243", "v244", "v245", "v246", "v247", "v248", "v249", "v250"
#define ISSUE0(kb) \
        asm volatile("s_nop 4\n\tbuffer_load_dwordx4 v[240:243], %0, %2, %3 offen sc0 sc1\n\ts_mov_b64 exec, %5\n\tbuffer_load_dwordx2 v[248:249], %1, %2, %3 offen offset:1024 sc0 sc1\n\t" \
                     "s_mov_b64 exec, 0xffffff\n\tbuffer_load_dwordx4 v[244:247], %0, %2, %4 offen offset:256 sc0 sc1\n\ts_mov_b64 exec, -1" \
                     :: "v"(l16), "v"(l8), "s"(rbox), "s"((kb) << 11), "s"(vdelta + ((kb) << 9)), "s"(((kb) & 32u) ? 0xffffffffull : ~0ull) : "memory", ZONE)
#define HEADERS(kb) do { \
        asm volatile("v_readlane_b32 %0, v247, 11\n\tv_readlane_b32 %1, v244, 12" : "=&s"(hdr0), "=&s"(hdr1) :: "memory", ZONE); \
        vat = img + 256u + l16; \
        if (__builtin_expect(((hdr0 | hdr1) & H_BIG) != 0u, 0)) { \
            asm volatile("s_nop 4\n\tbuffer_load_dwordx4 v[244:247], %0, %1, %2 offen sc0 sc1\n\ts_waitcnt vmcnt(0)" :: "v"(l16), "s"(rbox), "s"(vdelta + ((kb) << 9)) : "memory", ZONE); \
            vat = img + l16; \
        } } while (0)
#define TAKE(n) asm volatile("s_waitcnt vmcnt(" #n ")\n\tv_mov_b64 %0, v[240:241]\n\tv_mov_b64 %1, v[242:243]\n\tv_mov_b64 %2, v[248:249]" : "=v"(A0), "=v"(A1), "=v"(B) :: "memory", ZONE)
    auto FLAG = [=](u32 k) { asm volatile("s_mov_b64 exec, 1\n\tbuffer_store_dword %0, off, %1, %2 sc1\n\ts_mov_b64 exec, -1" ::"v"(vcall), "s"(rdone), "s"(k << DONE_SHIFT) : "memory"); };
#define STORE2(d, so, o0, o1) asm volatile("buffer_store_dwordx4 %0, %2, %3, %4 offen offset:" #o0 " sc1\n\tbuffer_store_dwordx4 %1, %2, %3, %4 offen offset:" #o1 " sc1\n\ts_nop 1" \
                     ::"v"((u32x4){d[0], d[1], d[2], d[3]}), "v"((u32x4){d[4], d[5], d[6], d[7]}), "v"(l16), "s"(rstage), "s"(so) : "memory")
    static_assert(VMID - VWIN == 256 && 2 * VWIN == 24 * 16, "the asm above and below has these built in");
    u32 kb, hdr0 = 0, hdr1 = 0, vat = 0;
    if (!describe(rk, kb, false)) return;
    STAMP(2); WSTAMP(33);
    u32x2 A0, A1, B;
    u32 prv = NUNITS, ticket = 0;
#define PST() do { } while (0)
    auto packed = [&](auto oc) -> bool {
        constexpr int O = decltype(oc)::value;
        u32 d[8];
        PST();
        asm volatile("ds_write_b128 %0, v[244:247]\n\t"
                     "s_mov_b64 exec, 1\n\tglobal_atomic_add v250, %1, %2, off sc0\n\ts_mov_b64 exec, -1\n\t"
                     "buffer_load_dwordx4 v[240:243], %3, %5, %6 offen offset:2048 sc0 sc1\n\ts_mov_b64 exec, %7\n\t"
                     "buffer_load_dwordx2 v[248:249], %4, %5, %6 offen offset:3072 sc0 sc1\n\ts_mov_b64 exec, -1"
                     :: "v"(vat), "v"(claim), "v"(1u), "v"(l16), "v"(l8), "s"(rbox), "s"(kb << 11), "s"(O ? 0xffffffffull : ~0ull) : "memory", ZONE);
        unpack_unit<O>((u32x4){A0[0], A0[1], A1[0], A1[1]}, B, hdr0, false, d, img + VMID, exp, lut, lane,
                       [&]() { if (prv < NUNITS) { asm volatile("s_waitcnt vmcnt(3)" ::: "memory"); FLAG(prv); } });
        WSTAMP(kb >> 6); PST();
        STORE2(d, stage_off(kb), 0, 1024);
        TAKE(2);
        PST();
        asm volatile("v_readfirstlane_b32 %0, v250" : "=s"(ticket) :: "memory", ZONE);
        ticket += tbase;
        u32 nkb = NUNITS;
        if (ticket < NBLOCKS && !describe(ticket, nkb, true)) { ticket = NBLOCKS; return false; }
        const bool go = nkb < NUNITS;
        PST();
        if (__builtin_expect(go, 1)) ISSUE0(nkb);
        unpack_unit<O>((u32x4){A0[0], A0[1], A1[0], A1[1]}, B, hdr1, true, d, img + VMID, exp, lut, lane, [&]() {
            if (__builtin_expect(go, 1)) asm volatile("s_waitcnt vmcnt(3)" ::: "memory"); else asm volatile("s_waitcnt vmcnt(0)" ::: "memory");
            FLAG(kb);
        });
        PST();
        STORE2(d, stage_off(kb), 2048, 3072);
        prv = kb + 1u; kb = nkb;
        if (__builtin_expect(!go, 0)) return false;
        TAKE(2);
        HEADERS(kb);
        return ((hdr0 | hdr1) & H_RAW) == 0u;
    };
    if (kb < NUNITS) { ISSUE0(kb); TAKE(0); HEADERS(kb); }
    for (;;) {
        if (kb < NUNITS && ((hdr0 | hdr1) & H_RAW) == 0u) {
            while ((kb & 32u) ? packed(std::integral_constant<int, 1>{}) : packed(std::integral_constant<int, 0>{})) { }
            asm volatile("s_waitcnt vmcnt(0)" ::: "memory");
            if (prv < NUNITS) FLAG(prv);
            prv = NUNITS;
            if (ticket >= NBLOCKS) break;
        }
        if (kb >= NUNITS) { kb &= NUNITS - 1u; hdr0 = hdr1 = H_RAW; }
        u32x4 R[4], V; u32x2 C0, C1; u32 tk;
        asm volatile("s_nop 4\n\ts_mov_b64 exec, 1\n\tglobal_atomic_add %4, %5, %6, off sc0\n\ts_mov_b64 exec, -1\n\t"
                     "buffer_load_dwordx4 %0, %7, %8, %9 offen sc0 sc1\n\tbuffer_load_dwordx4 %1, %7, %8, %9 offen offset:1024 sc0 sc1\n\t"
                     "buffer_load_dwordx4 %2, %7, %8, %9 offen offset:2048 sc0 sc1\n\tbuffer_load_dwordx4 %3, %7, %8, %9 offen offset:3072 sc0 sc1\n\ts_waitcnt vmcnt(0)"
                     : "=&v"(R[0]), "=&v"(R[1]), "=&v"(R[2]), "=&v"(R[3]), "=&v"(tk) : "v"(claim), "v"(1u), "v"(l16), "s"(rbox), "s"(kb << 11) : "memory");
        if (!(hdr0 & hdr1 & H_RAW)) {
            asm volatile("s_nop 4\n\tbuffer_load_dwordx2 %1, %4, %5, %6 offen offset:1024 sc0 sc1\n\tbuffer_load_dwordx2 %2, %4, %5, %6 offen offset:3072 sc0 sc1\n\t"
                         "buffer_load_dwordx4 %0, %3, %5, %7 offen sc0 sc1\n\ts_waitcnt vmcnt(0)"
                         : "=&v"(V), "=&v"(C0), "=&v"(C1) : "v"(l16), "v"(l8), "s"(rbox), "s"(kb << 11), "s"(vdelta + (kb << 9)) : "memory");
            *(LAS u32x4*)(size_t)(img + l16) = V;
#pragma nounroll
            for (int h = 0; h < 2; ++h) {
                const u32 hdr = h ? hdr1 : hdr0;
                if (hdr & H_RAW) continue;
                u32 d[8];
                if (kb & 32u) unpack_unit<1>(h ? R[2] : R[0], h ? C1 : C0, hdr, h != 0, d, img + VMID, exp, lut, lane, []() {});
                else unpack_unit<0>(h ? R[2] : R[0], h ? C1 : C0, hdr, h != 0, d, img + VMID, exp, lut, lane, []() {});
                const u32x4 lo4 = {d[0], d[1], d[2], d[3]}, hi4 = {d[4], d[5], d[6], d[7]};
                if (h) { R[2] = lo4; R[3] = hi4; } else { R[0] = lo4; R[1] = hi4; }
            }
        }
        asm volatile("buffer_store_dwordx4 %0, %4, %5, %6 offen sc1\n\tbuffer_store_dwordx4 %1, %4, %5, %6 offen offset:1024 sc1\n\t"
                     "buffer_store_dwordx4 %2, %4, %5, %6 offen offset:2048 sc1\n\tbuffer_store_dwordx4 %3, %4, %5, %6 offen offset:3072 sc1\n\ts_nop 1"
                     :: "v"(R[0]), "v"(R[1]), "v"(R[2]), "v"(R[3]), "v"(l16), "s"(rstage), "s"(stage_off(kb)) : "memory");
        WSTAMP(kb >> 6);
        ticket = (u32)__builtin_amdgcn_readfirstlane((int)tk) + tbase;
        const u32 done = kb;
        const bool next = ticket < NBLOCKS && describe(ticket, kb, false);
        if (next && kb < NUNITS) {
            ISSUE0(kb);
            asm volatile("s_waitcnt vmcnt(3)" ::: "memory");
            FLAG(done); FLAG(done + 1u);
            TAKE(2);
            HEADERS(kb);
        } else {
            asm volatile("s_waitcnt vmcnt(0)" ::: "memory");
            FLAG(done); FLAG(done + 1u);
            if (!next) break;
        }
    }
#undef ISSUE0
#undef TAKE
#undef HEADERS
#undef PST
#undef ZONE
#undef STORE2
    WSTAMP(32);
    STAMP(3);
}

__device__ __forceinline__ void gemm(const Args& a, u32 call, int xcd, int slot, LAS char* lds) {
    const int tid = threadIdx.x, wid = __builtin_amdgcn_readfirstlane(tid >> 6), lane = tid & 63;
    const int li = (xcd & 1) * GEMM_SLOTS + slot, ti = (xcd >> 1) * 4 + (li & 3), tj = li >> 2;
    const int row0 = ti * BM, col0 = tj * BN > DIM - BN ? DIM - BN : tj * BN;
    const int wr = wid >> 2, wc = wid & 3, fr = lane & 15, fq = lane >> 4;
    char* myws = a.me;
    u32* tmo = (u32*)(myws + TMO_OFF);
    const __amdgpu_buffer_rsrc_t rf = __builtin_amdgcn_make_buffer_rsrc(myws + DONE_OFF, 0, (int)(DBG_OFF - DONE_OFF), 0x00020000);
    const int vx = wid * 65536 + (row0 + lane) * 16, vw = (8 + wid) * 65536 + (col0 + lane) * 16;
    const int xoff = 2 * fq * XS + (wr * 128 + fr) * 16, woff = XB + 2 * fq * WS + (wc * 64 + fr) * 16, woff4 = woff + (256 - wc * 48) * 16;
    const int fu = lane < 16 ? ti * 2 + (lane >> 3) : lane < 40 ? 30 + (col0 >> 7) + (lane >> 3) : ti * 2;
    const int fw = (fu << (DONE_SHIFT - 2)) + (lane & 7);
    const u32* fl = (const u32*)(myws + DONE_OFF) + fw;
    bool dead = false;
    LAS u32* snap = (LAS u32*)(lds + 2 * STG);
    auto wait_tile = [&](int t) {
        if (dead) return;
        const u64 t0 = __builtin_amdgcn_s_memrealtime();
        for (u32 spins = 0;;) {
            const u32 seen = ld32_agent(fl + (t << (DONE_SHIFT + 4)));
            if (__all(seen == call)) break;
            if (give_up(tmo, call, t0, spins)) { dead = true; break; }
            __builtin_amdgcn_s_sleep(4);
        }
    };
#define RS(t) __builtin_amdgcn_make_buffer_rsrc(myws + STAGE_OFF, 0, (t) < NT ? (int)(NR * SRC_BYTES) : 0, 0x00020000)
#define DMA_X(t, b, i) __builtin_amdgcn_raw_ptr_buffer_load_lds(RS(t), (LAS void*)(lds + (b) * STG + wid * XS), 16, vx, (t) << 20, (i) * 1024, 17)
#define DMA_W(t, b, i) __builtin_amdgcn_raw_ptr_buffer_load_lds(RS(t), (LAS void*)(lds + (b) * STG + XB + wid * WS + ((i) == 4 ? 4096 : 0)), 16, vw, ((t) << 20) + ((i) == 4 ? 4096 : 0), ((i) & 3) * 1024, 17)
#define DMA_ALL(t, b) do { DMA_W(t, b, 0); DMA_W(t, b, 1); DMA_W(t, b, 2); DMA_W(t, b, 3); DMA_W(t, b, 4); DMA_X(t, b, 0); DMA_X(t, b, 1); DMA_X(t, b, 2); DMA_X(t, b, 3); } while (0)
    v4f acc[8][5];
#pragma unroll
    for (int m = 0; m < 8; ++m)
#pragma unroll
        for (int n = 0; n < 5; ++n) acc[m][n] = (v4f){0.f, 0.f, 0.f, 0.f};
    const float scale = __builtin_bit_cast(float, __builtin_amdgcn_readfirstlane(__builtin_bit_cast(int, a.sx[0] * a.sw[0])));
    __builtin_amdgcn_s_waitcnt(0x0F70);
    if (wid == 0) { wait_tile(0); wait_tile(1); }
    asm volatile("s_waitcnt lgkmcnt(0)" ::: "memory");
    __builtin_amdgcn_s_barrier(); asm volatile("" ::: "memory");
    STAMP(2);
    DMA_ALL(0, 0); DMA_W(1, 1, 0); DMA_W(1, 1, 1); DMA_W(1, 1, 2);
    asm volatile("s_waitcnt vmcnt(3)" ::: "memory");
    __builtin_amdgcn_s_barrier(); asm volatile("" ::: "memory");
    v8i wf[5], xa, xb;
    xa = ldfrag(lds, xoff, XS); SCHED;
#pragma unroll
    for (int n = 0; n < 5; ++n) { wf[n] = ldfrag(lds, n < 4 ? woff + n * 256 : woff4, WS); SCHED; }
#define MF(m, n, xf) acc[m][n] = __builtin_amdgcn_mfma_scale_f32_16x16x128_f8f6f4(xf, wf[n], acc[m][n], 0, 1, 0, 0, 0, 0)
#define MM(m, xf) do { _Pragma("unroll") for (int n = 0; n < 5; ++n) MF(m, n, xf); } while (0)
    for (int t = 0; t < NT - 1; ++t) {
        const int b = t & 1;
        LAS const char* cur = lds + b * STG; LAS const char* nxt = lds + (b ^ 1) * STG;
        xb = ldfrag(cur, xoff + 1 * 256, XS); DMA_W(t + 1, b ^ 1, 3); DMA_W(t + 1, b ^ 1, 4); DMA_X(t + 1, b ^ 1, 0); SCHED; MM(0, xa); SCHED;
        xa = ldfrag(cur, xoff + 2 * 256, XS); DMA_X(t + 1, b ^ 1, 1); DMA_X(t + 1, b ^ 1, 2); DMA_X(t + 1, b ^ 1, 3); SCHED; MM(1, xb); SCHED;
        xb = ldfrag(cur, xoff + 3 * 256, XS); SCHED; MM(2, xa); SCHED;
        xa = ldfrag(cur, xoff + 4 * 256, XS); SCHED; MM(3, xb); SCHED;
        xb = ldfrag(cur, xoff + 5 * 256, XS);
        if (wid == 0) {
            __builtin_amdgcn_raw_ptr_buffer_load_lds(rf, (LAS void*)snap, 4, fw * 4, ((t + 2) & 31) << (DONE_SHIFT + 6), 0, 16);
            __builtin_amdgcn_raw_ptr_buffer_load_lds(rf, (LAS void*)(snap + 64), 4, 0, (int)(TMO_OFF - DONE_OFF), 0, 16);
        }
        SCHED; MM(4, xa); SCHED;
        xa = ldfrag(cur, xoff + 6 * 256, XS); SCHED; MM(5, xb); SCHED;
        xb = ldfrag(cur, xoff + 7 * 256, XS); SCHED; MM(6, xa); SCHED;
        asm volatile("s_waitcnt vmcnt(0) lgkmcnt(0)" ::: "memory");
        if (wid == 0) {
            WSTAMP(t);
            if (t + 2 < NT && !__all(*(LAS volatile u32*)(snap + lane) == call)) wait_tile(t + 2);
            WSTAMP(32 + t);
            __builtin_amdgcn_s_waitcnt(0xC07F);
        }
        __builtin_amdgcn_s_barrier(); asm volatile("" ::: "memory"); SCHED;
        DMA_W(t + 2, b, 0); DMA_W(t + 2, b, 1); DMA_W(t + 2, b, 2);
        xa = ldfrag(nxt, xoff, XS); SCHED;
#pragma unroll
        for (int n = 0; n < 5; ++n) { MF(7, n, xb); SCHED; wf[n] = ldfrag(nxt, n < 4 ? woff + n * 256 : woff4, WS); SCHED; }
    }
    STAMP(3);
    const float sc = *(LAS const volatile u32*)(snap + 64) == call ? __builtin_nanf("") : scale;
    const float nsl = scale * -1.44269504089f;
    const u32x4 ro = make_rsrc(a.out + (size_t)(row0 + wr * 128) * DIM + col0, (u32)(128 * DIM - col0) * 4u);
    const int v4 = (tj == 12 && wc == 0) ? 0x40000000 : (fq * 4 * DIM + wc * 64 + fr * 4) * 4, v1 = (fq * 4 * DIM + 256 + wc * 16 + fr) * 4;
    auto silu2 = [&](float a0, float a1, float& r0, float& r1) {
        const v2f v = {a0, a1}, y = v * sc, u = v * nsl;
        v2f e = {__builtin_amdgcn_exp2f(__builtin_amdgcn_fmed3f(u[0], -86.5617024533f, 86.5617024533f)),
                 __builtin_amdgcn_exp2f(__builtin_amdgcn_fmed3f(u[1], -86.5617024533f, 86.5617024533f))};
        e = e + 1.f;
        r0 = __builtin_amdgcn_rcpf(e[0]) * y[0]; r1 = __builtin_amdgcn_rcpf(e[1]) * y[1];
    };
#define EPI(m) do { _Pragma("unroll") for (int j = 0; j < 4; j += 2) { \
        float r0[5], r1[5]; \
        _Pragma("unroll") for (int n = 0; n < 5; ++n) silu2(acc[m][n][j], acc[m][n][j + 1], r0[n], r1[n]); \
        asm volatile("buffer_store_dwordx4 %0, %4, %6, %7 offen " OUT_FLAGS "\n\tbuffer_store_dwordx4 %1, %4, %6, %8 offen " OUT_FLAGS "\n\t" \
                     "buffer_store_dword %2, %5, %6, %7 offen " OUT_FLAGS "\n\tbuffer_store_dword %3, %5, %6, %8 offen " OUT_FLAGS \
                     :: "v"((v4f){r0[0], r0[1], r0[2], r0[3]}), "v"((v4f){r1[0], r1[1], r1[2], r1[3]}), "v"(r0[4]), "v"(r1[4]), "v"(v4), "v"(v1), \
                        "s"(ro), "s"(((m) * 16 + j) * DIM * 4), "s"(((m) * 16 + j + 1) * DIM * 4) : "memory"); \
    } } while (0)
    {
        LAS const char* cur = lds + ((NT - 1) & 1) * STG;
        xb = ldfrag(cur, xoff + 1 * 256, XS); SCHED; MM(0, xa); SCHED;
        xa = ldfrag(cur, xoff + 2 * 256, XS); SCHED; MM(1, xb); SCHED; EPI(0); SCHED;
        xb = ldfrag(cur, xoff + 3 * 256, XS); SCHED; MM(2, xa); SCHED; EPI(1); SCHED;
        xa = ldfrag(cur, xoff + 4 * 256, XS); SCHED; MM(3, xb); SCHED; EPI(2); SCHED;
        xb = ldfrag(cur, xoff + 5 * 256, XS); SCHED; MM(4, xa); SCHED; EPI(3); SCHED;
        xa = ldfrag(cur, xoff + 6 * 256, XS); SCHED; MM(5, xb); SCHED; EPI(4); SCHED;
        xb = ldfrag(cur, xoff + 7 * 256, XS); SCHED; MM(6, xa); SCHED; EPI(5); SCHED;
        MM(7, xb); SCHED; EPI(6); SCHED; EPI(7);
    }
    STAMP(4);
}
}

__global__ void __launch_bounds__(512, 2) fused_allgather_gemm(Args a) {
    extern __shared__ __attribute__((aligned(16))) char lds_dyn[];
    LAS char* lds = (LAS char*)lds_dyn;
    asm volatile("" :: "s"(a.x), "s"(a.w), "s"(a.me), "s"(a.rank));
    STAMP(0);
    const int xcd = blockIdx.x & 7, slot = blockIdx.x >> 3;
    u32* cnt = (u32*)(a.me + CNT_OFF) + blockIdx.x;
    if (__builtin_expect(slot < GEMM_SLOTS, 1)) {
        const u32 k0 = task_unit(blockIdx.x, (u32)__builtin_amdgcn_readfirstlane(threadIdx.x >> 6));
        u32x4 r0, r1;
        load_unit(a, k0, threadIdx.x & 63, r0, r1);
        const u32 prev = __hip_atomic_load(cnt, __ATOMIC_RELAXED, __HIP_MEMORY_SCOPE_AGENT);
        SCHED;
        const u32 call = stage_out(a, lds, blockIdx.x, blockIdx.x < 64 && ((RAW_TASKS >> blockIdx.x) & 1ull), prev, r0, r1);
        if (slot < PULL_SLOTS) {
            load_unit(a, k0 + GEMM_SLOTS * 64u, threadIdx.x & 63, r0, r1);
            stage_out(a, lds, blockIdx.x + GEMM_SLOTS * 8, false, prev, r0, r1);
        }
        if (__builtin_amdgcn_readfirstlane(threadIdx.x) == 0) st32_agent_lane0(cnt, call);
        STAMP(1);
        gemm(a, call, xcd, slot, lds);
    } else {
        pull(a, cnt, (slot - GEMM_SLOTS) * 8 + xcd, lds);
    }
}

extern "C" void kernel_launch(int rank, int nranks, ncclComm_t comm, hipStream_t stream,
                              void* const* d_in, const int* in_sizes, int n_in,
                              void* d_out, int out_size, const int* in_dtypes, int out_dtype,
                              void* const* d_out_all, void* const* d_ws_all, size_t ws_bytes) {
    if (nranks != NR || (unsigned)rank >= (unsigned)NR || n_in != 4 || ws_bytes < WS_NEED || out_size != DIM * DIM ||
        in_sizes[0] != DIM * KS || in_sizes[1] != DIM * KS) {
        fprintf(stderr, "kernel_launch: unexpected problem shape\n");
        return;
    }
    static bool attr_set[NR] = {};
    if (!attr_set[rank]) {
        HIP_CHECK(hipFuncSetAttribute((const void*)fused_allgather_gemm, hipFuncAttributeMaxDynamicSharedMemorySize, LDS_BYTES));
        attr_set[rank] = true;
    }
    Args a;
    a.x = (const unsigned char*)d_in[0]; a.w = (const unsigned char*)d_in[1];
    a.sx = (const float*)d_in[2]; a.sw = (const float*)d_in[3];
    a.out = (float*)d_out;
    for (int p = 0; p < NR; ++p) a.ws[p] = (char*)d_ws_all[p];
    a.me = a.ws[rank]; a.rank = rank; a.pad = 0;
    fused_allgather_gemm<<<256, 512, LDS_BYTES, stream>>>(a);
    HIP_CHECK(hipGetLastError());
}
```

```cpp
#include <hip/hip_runtime.h>
#include <rccl/rccl.h>
#include <cstdio>
#include <type_traits>

#define HIP_CHECK(expr) do { hipError_t _e = (expr); if (_e != hipSuccess) { \
    fprintf(stderr, "HIP error %s at %s:%d\n", hipGetErrorString(_e), __FILE__, __LINE__); \
    return; } } while (0)

#define LAS __attribute__((address_space(3)))
#define GAS __attribute__((address_space(1)))
typedef unsigned u32;
typedef unsigned long long u64;
typedef int v2i __attribute__((ext_vector_type(2)));
typedef int v4i __attribute__((ext_vector_type(4)));
typedef int v8i __attribute__((ext_vector_type(8)));
typedef float v2f __attribute__((ext_vector_type(2)));
typedef float v4f __attribute__((ext_vector_type(4)));
typedef unsigned u32x2 __attribute__((ext_vector_type(2)));
typedef unsigned u32x4 __attribute__((ext_vector_type(4)));

namespace {
constexpr int NR = 8;
constexpr int DIM = 4096;
constexpr int KS = DIM / NR;
constexpr int NT = 32;
constexpr u32 NUNITS = NT * 64, NBLOCKS = NUNITS / 2;
constexpr size_t STAGE_OFF = 0;
constexpr size_t SRC_BYTES = (size_t)NUNITS * 2048;
constexpr size_t OUTBOX_OFF = STAGE_OFF + NR * SRC_BYTES;
constexpr size_t VBOX_OFF = OUTBOX_OFF + 2 * SRC_BYTES;
constexpr size_t VBOX_HALF = (size_t)NBLOCKS * 1024;
constexpr size_t RDY_OFF = VBOX_OFF + 2 * VBOX_HALF + 4096;
constexpr u32 RDY_SRC = 65536 + 2304, RDY_DENSE = NR * RDY_SRC, RDY_DENSE_SRC = 1280;
constexpr size_t RDY_HALF = RDY_DENSE + NR * RDY_DENSE_SRC;
#define DONE_SHIFT 7
constexpr size_t DONE_OFF = RDY_OFF + 2 * RDY_HALF;
constexpr size_t CLAIM_OFF = DONE_OFF + ((size_t)(NUNITS + 512) << DONE_SHIFT);
constexpr size_t CNT_OFF = CLAIM_OFF + NR * 128;
constexpr size_t TMO_OFF = CNT_OFF + 256 * 4;
constexpr size_t DBG_OFF = TMO_OFF + 128;
constexpr size_t WS_NEED = DBG_OFF + 256 * 64 + 256 * 8 * 80 * 8;
#define OUT_FLAGS "sc0 sc1"
#define DMA_EARLY 0
#define LATE_TASKS 32
#define LATE_SLEEP 32
#define POLL_SLEEP 4
constexpr u32 VMID = 448, VWIN = 192;
constexpr u32 H_RAW = 0x80000000u, H_BIG = 0x40000000u;
constexpr u64 DEADLINE = 150000000ull;
constexpr int BM = 256, BN = 320, XS = BM * 16, WS = BN * 16, XB = 8 * XS, WB = 8 * WS, STG = XB + WB;
constexpr int LDS_BYTES = 2 * STG + 512;
constexpr int GEMM_SLOTS = 26, PULL_SLOTS = 6;

#define STAG_A 0
#define RAW_BLOCKS 32
#define SPLIT_HEAD 64
#define SPLIT_TAIL 32
constexpr u32 NITEMS = NBLOCKS + 256;
constexpr u32 I_RAW = 0x1000u, I_SOLO = 0x2000u;
struct Order { unsigned short item[NITEMS]; unsigned char raw[256]; u32 n; };
constexpr int lag16(int p) { return STAG_A * 16 * p / 15; }
constexpr Order make_order() {
    Order r = {};
    unsigned short blks[NBLOCKS] = {};
    int n = 0;
    for (int key = 0; n < (int)NBLOCKS; ++key)
        for (int p = 0; p < 16; ++p) {
            const int d = key - lag16(p);
            if (d >= 0 && d % 16 == 0 && d / 16 < NT)
                for (int o = 0; o < 2; ++o) {
                    const int blk = (d / 16) * 32 + o * 16 + p, task = (blk >> 5) * 8 + ((blk & 15) >> 1);
                    if (n < RAW_BLOCKS && task < 64) r.raw[task] = 1;
                    blks[n++] = (unsigned short)blk;
                }
        }
    for (int i = 0; i < (int)NBLOCKS; ++i) {
        const u32 k = blks[i] * 2u;
        if (r.raw[(blks[i] >> 5) * 8 + ((blks[i] & 15) >> 1)]) { r.item[r.n++] = (unsigned short)(k | I_RAW); r.item[r.n++] = (unsigned short)((k + 1) | I_RAW); }
        else if (i < SPLIT_HEAD || i >= (int)NBLOCKS - SPLIT_TAIL) { r.item[r.n++] = (unsigned short)(k | I_SOLO); r.item[r.n++] = (unsigned short)((k + 1) | I_SOLO); }
        else r.item[r.n++] = (unsigned short)k;
    }
    return r;
}
__device__ constexpr Order ORDER = make_order();
constexpr u64 raw_mask() { const Order o = make_order(); u64 m = 0; for (int i = 0; i < 64; ++i) if (o.raw[i]) m |= 1ull << i; return m; }
constexpr u64 RAW_TASKS = raw_mask();
constexpr u32 NTICKETS = make_order().n;
static_assert(NTICKETS <= NITEMS, "order table too small");

struct Args {
    const unsigned char* x; const unsigned char* w; const float* sx; const float* sw; float* out;
    char* me; char* ws[NR]; int rank; int pad;
};

__device__ __forceinline__ void st16_sys(void* p, u32x4 v) {
    asm volatile("global_store_dwordx4 %0, %1, off sc0 sc1\n\ts_nop 1" ::"v"(p), "v"(v) : "memory");
}
__device__ __forceinline__ void st16_agent(void* p, u32x4 v) {
    asm volatile("global_store_dwordx4 %0, %1, off sc1\n\ts_nop 1" ::"v"(p), "v"(v) : "memory");
}
__device__ __forceinline__ u32 ld32_agent(const void* p) {
    u32 v;
    asm volatile("global_load_dword %0, %1, off sc1\n\ts_waitcnt vmcnt(0)" : "=&v"(v) : "v"(p) : "memory");
    return v;
}
__device__ __forceinline__ void st32_agent_lane0(void* p, u32 v) {
    asm volatile("s_mov_b64 exec, 1\n\tglobal_store_dword %0, %1, off sc1\n\ts_mov_b64 exec, -1" ::"v"(p), "v"(v) : "memory");
}
__device__ __forceinline__ bool give_up(u32* tmo, u32 call, u64 t0, u32& spins) {
    if ((++spins & 15u) != 0u) return false;
    if ((u32)__builtin_amdgcn_readfirstlane((int)ld32_agent(tmo)) == call || __builtin_amdgcn_s_memrealtime() - t0 > DEADLINE) {
        st32_agent_lane0(tmo, call);
        return true;
    }
    return false;
}
template <class T> __device__ __forceinline__ T* uniform_ptr(T* p) {
    const u64 v = (u64)p;
    return (T*)(((u64)(u32)__builtin_amdgcn_readfirstlane((int)(u32)(v >> 32)) << 32) | (u32)__builtin_amdgcn_readfirstlane((int)(u32)v));
}
__device__ __forceinline__ v8i ldfrag(LAS const char* p, int off, int stride) {
    v4i a = *(LAS const v4i*)(p + off), b = *(LAS const v4i*)(p + off + stride);
    return __builtin_shufflevector(a, b, 0, 1, 2, 3, 4, 5, 6, 7);
}
#define SCHED __builtin_amdgcn_sched_barrier(0)
#define STAMP(i) do { } while (0)
#define GSTAMP(i) do { } while (0)
#define WSTAMP(i) do { } while (0)

__device__ __forceinline__ u32 wave_incl_scan(u32 x) {
    x += (u32)__builtin_amdgcn_update_dpp(0, (int)x, 0x111, 0xf, 0xf, false);
    x += (u32)__builtin_amdgcn_update_dpp(0, (int)x, 0x112, 0xf, 0xf, false);
    x += (u32)__builtin_amdgcn_update_dpp(0, (int)x, 0x114, 0xf, 0xf, false);
    x += (u32)__builtin_amdgcn_update_dpp(0, (int)x, 0x118, 0xf, 0xf, false);
    x += (u32)__builtin_amdgcn_update_dpp(0, (int)x, 0x142, 0xa, 0xf, false);
    x += (u32)__builtin_amdgcn_update_dpp(0, (int)x, 0x143, 0xc, 0xf, false);
    return x;
}
__device__ __forceinline__ u32 rotr(u32 x, u32 n) { return __builtin_amdgcn_alignbit(x, x, n); }
__device__ __forceinline__ u32 bfi(u32 m, u32 a, u32 b) { return __builtin_amdgcn_bitop3_b32(m, a, b, 0xca); }
__device__ __forceinline__ u32 and3(u32 a, u32 b, u32 c) { return __builtin_amdgcn_bitop3_b32(a, b, c, 0x80); }
__device__ __forceinline__ u32 lds32(u32 addr) { return *(LAS const u32*)(size_t)addr; }
__device__ __forceinline__ u32x4 make_rsrc(const void* p, u32 bytes) {
    const u64 v = (u64)p;
    return (u32x4){(u32)__builtin_amdgcn_readfirstlane((int)(u32)v), (u32)__builtin_amdgcn_readfirstlane((int)(u32)(v >> 32)) & 0xffffu, bytes, 0x00020000u};
}

constexpr int P0_VAR = 2560, P0_LUT = P0_VAR + 1536, P0_WAVE = P0_LUT + 128;

template <int O>
__device__ __forceinline__ void pack_unit(const u32 (&d)[8], char* slot, char* vslot, u32 h, LAS char* img, u32 lut, int lane) {
    constexpr int SH = O ? 2 : 3;
    constexpr u32 EM = O ? 0x1f1f1f1fu : 0x0f0f0f0fu;
    u32 ex[8], sum = 0;
#pragma unroll
    for (int j = 0; j < 8; ++j) ex[j] = (d[j] >> SH) & EM;
#pragma unroll
    for (int j = 0; j < 8; j += 2) sum = __builtin_amdgcn_udot4(ex[j], 0x01010101u, sum, false);
    const u32 tot = (u32)__builtin_amdgcn_readlane((int)wave_incl_scan(sum), 63);
    const u32 lo = tot > 50u ? (tot - 50u) >> 10 : 0u;
    const u32 bias = (0x80u - lo) * 0x01010101u;
    u32 P[2] = {0u, 0u}, X = 0, n2 = 0, n3 = 0, b3 = 0, q[8], n2j[8];
#pragma unroll
    for (int j = 0; j < 8; ++j) {
        const u32 a7 = ex[j] + bias;
        const u32 t = ((~a7 | (a7 - 0x03030303u)) & 0x80808080u) >> 7;
        P[j >> 2] |= ((a7 | t | (t << 1)) & 0x03030303u) << (2 * (j & 3));
        const u32 sc = a7 & ((a7 | 0x80808080u) >> 5) & 0x07070707u;
        X = (X >> 1) | ((~(a7 + 0x07070707u) | (a7 - 0x04040404u)) & 0x80808080u);
        const u32 cz = __builtin_amdgcn_perm(0x07070707u, sc, lds32(__builtin_amdgcn_udot4(t, 0x20100804u, lut, false)));
        q[j] = __builtin_amdgcn_udot4(__builtin_amdgcn_perm(0x00010303u, 0x03030302u, cz), 0x40100401u, 0u, false);
        const u32 f = __builtin_amdgcn_perm(0x00000101u, 0x01010100u, cz);
        const u32 c3 = __builtin_amdgcn_perm(0u, cz, lds32(__builtin_amdgcn_udot4(f, 0x20100804u, lut + 64u, false)));
        b3 |= __builtin_amdgcn_udot4(c3, 0x003e0801u, c3 >> 15, false) << (n3 & 31u);
        n3 = __builtin_amdgcn_udot4(f, 0x03030303u, n3, false);
        n2j[j] = n2;
        n2 = __builtin_amdgcn_udot4(t, 0x02020202u, n2, false);
    }
    const u32 qa = q[0] | (q[1] << n2j[1]) | (q[2] << n2j[2]) | (q[3] << n2j[3]);
    const u32 qb = q[4] | (q[5] << (n2j[5] - n2j[4])) | (q[6] << (n2j[6] - n2j[4])) | (q[7] << (n2j[7] - n2j[4]));
    const u64 b2 = (u64)qa | ((u64)qb << n2j[4]);
    const u32 both = n2 | (n3 << 16), incl = wave_incl_scan(both), pre = incl - both;
    const u32 total = (u32)__builtin_amdgcn_readlane((int)incl, 63);
    const u32 len2 = (((total & 0xffffu) + 31u) >> 5) << 2, len3 = (((total >> 16) + 31u) >> 5) << 2;
    const u32 used = 4u + len2 + len3;
    bool raw = __any(n3 > 30u) || used > VMID - 16u;
    u32 nexc = 0;
    if (!raw) {
        const u32 p2 = pre & 0xffffu, p3 = pre >> 16;
        LAS u32* w2 = (LAS u32*)(img + (h ? VMID + 4u : VMID - 4u - len2) + ((p2 >> 5) << 2));
        LAS u32* w3 = (LAS u32*)(img + (h ? VMID + 4u + len2 : VMID - used) + ((p3 >> 5) << 2));
        const u64 u2 = b2 << (p2 & 31u), u3 = (u64)b3 << (p3 & 31u);
        __hip_atomic_fetch_or(w2, (u32)u2, __ATOMIC_RELAXED, __HIP_MEMORY_SCOPE_WAVEFRONT);
        __hip_atomic_fetch_or(w2 + 1, (u32)(u2 >> 32), __ATOMIC_RELAXED, __HIP_MEMORY_SCOPE_WAVEFRONT);
        __hip_atomic_fetch_or(w2 + 2, (u32)(b2 >> 32) >> 1 >> (31u - (p2 & 31u)), __ATOMIC_RELAXED, __HIP_MEMORY_SCOPE_WAVEFRONT);
        __hip_atomic_fetch_or(w3, (u32)u3, __ATOMIC_RELAXED, __HIP_MEMORY_SCOPE_WAVEFRONT);
        __hip_atomic_fetch_or(w3 + 1, (u32)(u3 >> 32), __ATOMIC_RELAXED, __HIP_MEMORY_SCOPE_WAVEFRONT);
        u64 bal;
        while ((bal = __ballot(X != 0u)) != 0ull) {
            const u32 slotn = nexc + __builtin_amdgcn_mbcnt_hi((u32)(bal >> 32), __builtin_amdgcn_mbcnt_lo((u32)bal, 0u));
            nexc += (u32)__builtin_popcountll(bal);
            if (nexc > 64u || used + 2u * nexc > VMID - 16u) { raw = true; break; }
            if (X != 0u) {
                const u32 k = (u32)__builtin_ctz(X), j = k & 7u, i8 = k & 24u;
                const u32 e01 = (j & 1) ? ex[1] : ex[0], e23 = (j & 1) ? ex[3] : ex[2], e45 = (j & 1) ? ex[5] : ex[4], e67 = (j & 1) ? ex[7] : ex[6];
                const u32 e03 = (j & 2) ? e23 : e01, e47 = (j & 2) ? e67 : e45, ej = (j & 4) ? e47 : e03;
                const u32 n01 = (j & 1) ? n2j[1] : n2j[0], n23 = (j & 1) ? n2j[3] : n2j[2], n45 = (j & 1) ? n2j[5] : n2j[4], n67 = (j & 1) ? n2j[7] : n2j[6];
                const u32 n03 = (j & 2) ? n23 : n01, n47 = (j & 2) ? n67 : n45, nj = (j & 4) ? n47 : n03;
                const u32 Pj = ((j & 4) ? P[1] : P[0]) >> (2u * (j & 3u));
                const u32 pos = ((p2 + nj) >> 1) + (u32)__builtin_popcount(Pj & (Pj >> 1) & 0x01010101u & ((1u << i8) - 1u));
                *(LAS unsigned short*)(img + (h ? VMID + used + 2u * slotn : VMID - used - 2u - 2u * slotn)) = (unsigned short)(pos | (((ej >> i8) & 31u) << 11));
                X &= X - 1u;
            }
        }
    }
    if (raw) {
        st16_sys(slot + lane * 16, (u32x4){d[0], d[1], d[2], d[3]});
        st16_sys(slot + 1024 + lane * 16, (u32x4){d[4], d[5], d[6], d[7]});
        asm volatile("s_mov_b64 exec, 1\n\tglobal_store_dword %0, %1, off sc0 sc1\n\ts_mov_b64 exec, -1" ::"v"(vslot + (h ? VMID : VMID - 4u)), "v"(H_RAW) : "memory");
        return;
    }
    const u32 nvar = (used + 2u * nexc + 15u) >> 4;
    if (lane == 0) *(LAS u32*)(img + (h ? VMID : VMID - 4u)) = lo | (nexc << 5) | (len3 << 12) | (nvar * 16u > VWIN ? H_BIG : 0u);
    const u32 first = h ? VMID : VMID - nvar * 16u;
    const u32x4 v = *(LAS const u32x4*)(img + first + lane * 16);
    if (O == 0) {
        u32x4 sm;
#pragma unroll
        for (int k = 0; k < 4; ++k) sm[k] = bfi(0x87878787u, d[2 * k], rotr(d[2 * k + 1], 28));
        st16_sys(slot + lane * 16, sm);
        asm volatile("global_store_dwordx2 %0, %1, off offset:1024 sc0 sc1\n\ts_nop 1" ::"v"(slot + lane * 8), "v"((u32x2){P[0], P[1]}) : "memory");
    } else {
        u32x4 sm;
        sm[0] = bfi(0x60606060u, d[6] << 5, bfi(0x83838383u, d[0], rotr(d[1], 29)));
        sm[1] = bfi(0x60606060u, d[7] << 5, bfi(0x83838383u, d[2], rotr(d[3], 29)));
        sm[2] = bfi(0x60606060u, bfi(0x40404040u, d[6] >> 1, d[7] >> 2), bfi(0x83838383u, d[4], rotr(d[5], 29)));
        sm[3] = P[0];
        st16_sys(slot + lane * 16, sm);
        const u32 up = __builtin_amdgcn_permlane32_swap(P[1], P[1], false, false)[1];
        if (lane < 32) asm volatile("global_store_dwordx2 %0, %1, off offset:1024 sc0 sc1\n\ts_nop 1" ::"v"(slot + lane * 8), "v"((u32x2){P[1], up}) : "memory");
    }
    if ((u32)lane < nvar) st16_sys(vslot + first + lane * 16, v);
}

__device__ __forceinline__ u32 spread3(u32 v) {
    const u32 x = bfi(0x3fu, v, v << 10);
    return bfi(0x00070007u, x, x << 5) & 0x07070707u;
}
__device__ __forceinline__ u32 spread2(u32 v) {
    const u32 x = bfi(0xfu, v, v << 12);
    return bfi(0x00030003u, x, x << 6) & 0x03030303u;
}

template <int O, class F>
__device__ __forceinline__ void unpack_unit(const u32x4 A, const u32x2 B, const u32 hdr, const bool H, u32 (&d)[8], u32 mid, u32 exp, u32 lut, int lane, F&& midway) {
    constexpr int SH = O ? 2 : 3;
    const u32 P[2] = {O ? A[3] : B[0], O ? __builtin_amdgcn_permlane32_swap(B[0], B[1], false, false)[0] : B[1]};
    const u32 lo = hdr & 31u, nexc = (hdr >> 5) & 127u, len3 = min((hdr >> 12) & 0x1fcu, VMID - 16u);
    const u32 E[2] = {P[0] & (P[0] >> 1) & 0x55555555u, P[1] & (P[1] >> 1) & 0x55555555u};
    const u32 cnt = (u32)__builtin_popcount(E[0]) + (u32)__builtin_popcount(E[1]), incl = wave_incl_scan(cnt);
    const u32 n = min((u32)__builtin_amdgcn_readlane((int)incl, 63), 1712u);
    const u32 prim = ((lo * 0x01010101u + 0x00020100u) << SH);
    const u32 b8 = ((lo + 24u) & 31u) * 0x01010101u;
    const u32 t3lo = ((b8 + 0x03020100u) & 0x1f1f1f1fu) << SH, t3hi = ((b8 + 0x07060504u) & 0x1f1f1f1fu) << SH;
    const u32 t2 = ((b8 + 0x000b0607u) & 0x001f1f1fu) << SH;
    const u32 len2 = ((n + 15u) >> 4) << 2;
    const u32 s2 = H ? mid + 4u : mid - 4u - len2, s3 = H ? s2 + len2 : s2 - len3;
    u32 m3 = 0;
    for (u32 base = 0; base < n; base += 512u) {
        u32 c = *(LAS const unsigned short*)(size_t)(s2 + (base >> 2) + (u32)lane * 2u);
        if (base + (u32)lane * 8u >= n) c = 0u;
        const u32 c0 = spread2(c), c1 = spread2(c >> 8);
        const u32 f0 = and3(c0, c0 >> 1, 0x01010101u), f1 = and3(c1, c1 >> 1, 0x01010101u);
        const u32 k0 = __builtin_amdgcn_udot4(f0, 0x03030303u, 0u, false), k = __builtin_amdgcn_udot4(f1, 0x03030303u, k0, false);
        const u32 in3 = wave_incl_scan(k), bit = min(m3 + in3 - k, 3583u);
        LAS const volatile u32* wp = (LAS const volatile u32*)(size_t)(s3 + ((bit >> 5) << 2));
        const u32 g0 = lds32(__builtin_amdgcn_udot4(f0, 0x20100804u, lut, false)), g1 = lds32(__builtin_amdgcn_udot4(f1, 0x20100804u, lut, false));
        const u32 w0 = wp[0], w1 = wp[1];
        const u32 w = __builtin_amdgcn_alignbit(w1, w0, bit);
        const u32 e0 = __builtin_amdgcn_perm(t3hi, t3lo, spread3(w)), e1 = __builtin_amdgcn_perm(t3hi, t3lo, spread3(w >> k0));
        *(LAS u32x2*)(size_t)(exp + base + (u32)lane * 8u) = (u32x2){__builtin_amdgcn_perm(e0, t2, c0 ^ g0), __builtin_amdgcn_perm(e1, t2, c1 ^ g1)};
        m3 += (u32)__builtin_amdgcn_readlane((int)in3, 63);
    }
    if (nexc) {
        if ((u32)lane < nexc) {
            const u32 e = *(LAS const unsigned short*)(size_t)(H ? s3 + len3 + 2u * (u32)lane : s3 - 2u - 2u * (u32)lane);
            *(LAS unsigned char*)(size_t)(exp + (e & 0x7ffu)) = (unsigned char)((e >> 11) << SH);
        }
    }
    midway();
    u32 ptr[8], dep[8], ex[8];
    u32 at = exp + incl - cnt;
#pragma unroll
    for (int j = 0; j < 8; ++j) {
        const int g = j & 3;
        const u32 Eh = g < 2 ? E[j >> 2] : E[j >> 2] >> 4;
        const u32 m = Eh & ((g & 1) ? 0x04040404u : 0x01010101u);
        dep[j] = lds32(__builtin_amdgcn_udot4(m, (g & 1) ? 0x08040201u : 0x20100804u, lut, false));
        ptr[j] = at;
        at += (u32)__builtin_popcount(m);
    }
    u32x2 wn[8];
    asm volatile("ds_read2_b32 %0, %8 offset1:1\n\tds_read2_b32 %1, %9 offset1:1\n\tds_read2_b32 %2, %10 offset1:1\n\tds_read2_b32 %3, %11 offset1:1\n\t"
                 "ds_read2_b32 %4, %12 offset1:1\n\tds_read2_b32 %5, %13 offset1:1\n\tds_read2_b32 %6, %14 offset1:1\n\tds_read2_b32 %7, %15 offset1:1\n\t"
                 "s_waitcnt lgkmcnt(0)"
                 : "=&v"(wn[0]), "=&v"(wn[1]), "=&v"(wn[2]), "=&v"(wn[3]), "=&v"(wn[4]), "=&v"(wn[5]), "=&v"(wn[6]), "=&v"(wn[7])
                 : "v"(ptr[0] & ~3u), "v"(ptr[1] & ~3u), "v"(ptr[2] & ~3u), "v"(ptr[3] & ~3u), "v"(ptr[4] & ~3u), "v"(ptr[5] & ~3u), "v"(ptr[6] & ~3u), "v"(ptr[7] & ~3u)
                 : "memory");
#pragma unroll
    for (int j = 0; j < 8; ++j)
        ex[j] = __builtin_amdgcn_perm(__builtin_amdgcn_alignbyte(wn[j][1], wn[j][0], ptr[j]), prim, ((P[j >> 2] >> (2 * (j & 3))) & 0x03030303u) ^ dep[j]);
    if (O == 0) {
#pragma unroll
        for (int k = 0; k < 4; ++k) { d[2 * k] = (A[k] & 0x87878787u) | ex[2 * k]; d[2 * k + 1] = (rotr(A[k], 4) & 0x87878787u) | ex[2 * k + 1]; }
    } else {
#pragma unroll
        for (int k = 0; k < 3; ++k) { d[2 * k] = (A[k] & 0x83838383u) | ex[2 * k]; d[2 * k + 1] = (rotr(A[k], 3) & 0x83838383u) | ex[2 * k + 1]; }
        d[6] = (bfi(0x80808080u, A[2] << 1, A[0] >> 5) & 0x83838383u) | ex[6];
        d[7] = (bfi(0x80808080u, A[2] << 2, A[1] >> 5) & 0x83838383u) | ex[7];
    }
}

__host__ __device__ constexpr u32 task_unit(u32 b, u32 wid) { return (b >> 3) * 64u + (wid >> 2) * 32u + (b & 7u) * 4u + (wid & 3u); }
__host__ __device__ constexpr u32 unit_task(u32 k) { return (k >> 6) * 8u + ((k & 31u) >> 2); }
__device__ __forceinline__ u32 stage_off(u32 k) { return ((k >> 5) << 19) | ((k & 31u) << 11); }

__device__ __forceinline__ void load_unit(const Args& a, u32 k, int lane, u32x4& r0, u32x4& r1) {
    const u32 t = k >> 6, rows = (k & 31u) * 128u;
    const unsigned char* src = (k & 32u) ? a.w + (size_t)(16u * t + (lane >> 3)) * DIM + rows + (lane & 7) * 16
                                         : a.x + (size_t)(rows + lane) * KS + 16u * t;
    r0 = *(const u32x4*)src; r1 = *(const u32x4*)(src + 32768);
}
__device__ __forceinline__ u32 stage_out(const Args& a, LAS char* lds, const u32 b, const bool plain, const u32 prev, const u32x4 r0, const u32x4 r1) {
    const int tid = threadIdx.x, wid = __builtin_amdgcn_readfirstlane(tid >> 6), lane = tid & 63;
    const u32 k = task_unit(b, (u32)wid), o = (k >> 5) & 1u;
    char* myws = a.me;
    LAS char* wl = lds + wid * P0_WAVE;
    {
        u32 sel = 0x04040404u; int n = 0;
#pragma unroll
        for (int i = 0; i < 4; ++i) if ((lane >> i) & 1) { sel = (sel & ~(0xffu << (8 * n))) | ((u32)i << (8 * n)); ++n; }
        if (lane & 16) sel |= (sel & 0x04040404u) << 1;
        if (lane < 32) *(LAS u32*)(wl + P0_LUT + lane * 4) = sel;
        *(LAS u32x4*)(wl + P0_VAR + lane * 16) = (u32x4){0u, 0u, 0u, 0u};
    }
    SCHED;
    u32 d[8];
    if (o == 0) {
#pragma unroll
        for (int j = 0; j < 4; ++j) { d[j] = r0[j]; d[4 + j] = r1[j]; }
    } else {
        *(LAS u32x4*)(wl + (lane >> 3) * 160 + (lane & 7) * 16) = r0;
        *(LAS u32x4*)(wl + (8 + (lane >> 3)) * 160 + (lane & 7) * 16) = r1;
        LAS char* tb = wl + ((lane & 15) >> 1) * 160 + (lane >> 4) * 16 + (lane & 1) * 8;
        asm volatile("" ::: "memory");
        const v2i c00 = __builtin_amdgcn_ds_read_tr8_b64_v2i32((LAS v2i*)tb);
        const v2i c01 = __builtin_amdgcn_ds_read_tr8_b64_v2i32((LAS v2i*)(tb + 8 * 160));
        const v2i c10 = __builtin_amdgcn_ds_read_tr8_b64_v2i32((LAS v2i*)(tb + 64));
        const v2i c11 = __builtin_amdgcn_ds_read_tr8_b64_v2i32((LAS v2i*)(tb + 8 * 160 + 64));
        const u32 g0 = (k & 31u) * 2u, g1 = g0 + 1u;
        const int mixed = ((lane & 15) * 4 + (lane >> 4)) * 4;
        const int s0 = g0 % 5u == 4u ? lane * 4 : mixed, s1 = (g1 % 5u == 4u || g1 == 63u) ? lane * 4 : mixed;
        d[0] = __builtin_amdgcn_ds_bpermute(s0, c00[0]); d[1] = __builtin_amdgcn_ds_bpermute(s0, c00[1]);
        d[2] = __builtin_amdgcn_ds_bpermute(s0, c01[0]); d[3] = __builtin_amdgcn_ds_bpermute(s0, c01[1]);
        d[4] = __builtin_amdgcn_ds_bpermute(s1, c10[0]); d[5] = __builtin_amdgcn_ds_bpermute(s1, c10[1]);
        d[6] = __builtin_amdgcn_ds_bpermute(s1, c11[0]); d[7] = __builtin_amdgcn_ds_bpermute(s1, c11[1]);
    }
    const u32 call = (u32)__builtin_amdgcn_readfirstlane((int)prev) + 1u;
    const u32 par = call & 1u;
    char* slot = myws + OUTBOX_OFF + par * SRC_BYTES + (size_t)k * 2048;
    char* own = myws + STAGE_OFF + (size_t)a.rank * 65536 + stage_off(k) + lane * 16;
    const u32x4 w0 = {d[0], d[1], d[2], d[3]}, w1 = {d[4], d[5], d[6], d[7]};
    if (plain) { st16_sys(slot + lane * 16, w0); st16_sys(slot + 1024 + lane * 16, w1); }
    st16_agent(own, w0); st16_agent(own + 1024, w1);
    if (!plain) {
        char* vslot = myws + VBOX_OFF + par * VBOX_HALF + (size_t)(k >> 1) * 1024;
        if (o == 0) pack_unit<0>(d, slot, vslot, k & 1u, wl + P0_VAR, (u32)(size_t)(wl + P0_LUT), lane);
        else pack_unit<1>(d, slot, vslot, k & 1u, wl + P0_VAR, (u32)(size_t)(wl + P0_LUT), lane);
    }
    char* w[NR];
#pragma unroll
    for (int p = 0; p < NR; ++p) w[p] = a.ws[p];
    asm volatile("" : "+s"(w[0]), "+s"(w[1]), "+s"(w[2]), "+s"(w[3]), "+s"(w[4]), "+s"(w[5]), "+s"(w[6]), "+s"(w[7]));
    const u32 off = (u32)(RDY_OFF + par * RDY_HALF) + (lane == 0 ? (u32)a.rank * RDY_SRC + b * 256u : RDY_DENSE + (u32)a.rank * RDY_DENSE_SRC + b * 4u);
    asm volatile("s_waitcnt vmcnt(0)" ::: "memory");
    __syncthreads();
    if (wid == 0)
        asm volatile("s_mov_b64 exec, 3\n\tglobal_store_dword %0, %1, %2 sc0 sc1\n\tglobal_store_dword %0, %1, %3 sc0 sc1\n\tglobal_store_dword %0, %1, %4 sc0 sc1\n\t"
                     "global_store_dword %0, %1, %5 sc0 sc1\n\tglobal_store_dword %0, %1, %6 sc0 sc1\n\tglobal_store_dword %0, %1, %7 sc0 sc1\n\t"
                     "global_store_dword %0, %1, %8 sc0 sc1\n\tglobal_store_dword %0, %1, %9 sc0 sc1\n\ts_mov_b64 exec, -1"
                     ::"v"(off), "v"(call), "s"(w[0]), "s"(w[1]), "s"(w[2]), "s"(w[3]), "s"(w[4]), "s"(w[5]), "s"(w[6]), "s"(w[7]) : "memory");
    st32_agent_lane0((u32*)(myws + DONE_OFF + ((size_t)k << DONE_SHIFT)) + a.rank, call);
    return call;
}

constexpr int PL_WAVE = 512 + 1536 + 2560 + 64;
#define PULL_WPW 7
constexpr u32 PULL_WAVES = PULL_SLOTS * 8 * PULL_WPW;

__device__ __forceinline__ void pull(const Args& a, u32* cnt, int c, LAS char* lds) {
    const int tid = threadIdx.x, wid = __builtin_amdgcn_readfirstlane(tid >> 6), lane = tid & 63;
    const u32 ord = ((const u32*)ORDER.item)[tid], ord2 = ((const u32*)ORDER.item)[512 + (tid & 127)];
    const u32 call = (u32)__builtin_amdgcn_readfirstlane((int)ld32_agent(cnt)) + 1u;
    const u32 gw = (u32)c * PULL_WPW + (u32)wid, rk = gw / 7u, link = PULL_WPW == 7 ? ((u32)wid + (u32)c) % 7u : gw % 7u;
    const u32 nw = (PULL_WAVES + 6u - link) / 7u;
    const u32 s = ((u32)a.rank + 1u + link) & 7u, par = call & 1u;
    char* myws = a.me;
    u32* tmo = (u32*)(myws + TMO_OFF);
    const u32 img = (u32)(size_t)(lds + 4096 + wid * PL_WAVE + 512), exp = img + 1536u, lut = exp + 2560u;
    *(LAS u32*)(lds + tid * 4) = ord; *(LAS u32*)(lds + 2048 + (tid & 127) * 4) = ord2;
    if (lane < 16) {
        u32 dep = 0; u32 n = 0;
#pragma unroll
        for (int i = 0; i < 4; ++i) if ((lane >> i) & 1) { dep |= (3u ^ (4u + n)) << (8 * i); ++n; }
        *(LAS u32*)(size_t)(lut + lane * 4) = dep;
    }
    __builtin_amdgcn_s_waitcnt(0x0F70);
    __syncthreads();
    if (wid == 0) st32_agent_lane0(cnt, call);
    if (wid >= PULL_WPW) return;
    STAMP(1);
    const u32x4 rbox = make_rsrc(a.ws[s] + OUTBOX_OFF + par * SRC_BYTES, (u32)(RDY_OFF - OUTBOX_OFF - par * SRC_BYTES));
    const u32x4 rstage = make_rsrc(myws + STAGE_OFF + s * 65536u, (u32)(NR * SRC_BYTES) - s * 65536u);
    const u32x4 rdone = make_rsrc(myws + DONE_OFF + s * 4u, (NUNITS + 511u) << DONE_SHIFT);
    const u32 vdelta = (u32)(VBOX_OFF - OUTBOX_OFF) + par * (u32)VBOX_HALF - par * (u32)SRC_BYTES;
    const char* rdy = myws + RDY_OFF + par * RDY_HALF;
    u32* claim = (u32*)(myws + CLAIM_OFF) + link * 32u;
    const u32 tbase = nw - (call - 1u) * NTICKETS;
    const u32 l16 = (u32)lane * 16u, l8 = (u32)lane * 8u;
    u32 vcall = call; asm volatile("" : "+v"(vcall));
    u32 known = 0;
    auto lookup = [&](u32 ticket) -> u32 { return (u32)__builtin_amdgcn_readfirstlane(*(LAS const unsigned short*)(lds + ticket * 2u)); };
    auto await = [&](u32 ticket, u32 item, bool refresh) -> bool {
        const u32 b = unit_task(item & 2047u);
        {
            if (refresh) {
                u32x4 f;
                asm volatile("global_load_dwordx4 %0, %1, off sc0 sc1\n\ts_waitcnt vmcnt(0)" : "=&v"(f) : "v"(rdy + RDY_DENSE + s * RDY_DENSE_SRC + l16) : "memory");
                const u64 bad = __ballot(f[0] != call || f[1] != call || f[2] != call || f[3] != call);
                known = bad ? (u32)__builtin_ctzll(bad) * 4u : 256u;
                if (b < known) return true;
            }
            const u64 t0 = __builtin_amdgcn_s_memrealtime();
            for (u32 spins = 0;;) {
                u32 f;
                asm volatile("s_mov_b64 exec, 1\n\tglobal_load_dword %0, %1, off sc0 sc1\n\ts_mov_b64 exec, -1\n\ts_waitcnt vmcnt(0)" : "=&v"(f) : "v"(rdy + s * RDY_SRC + b * 256u) : "memory");
                const u32 seq = (u32)__builtin_amdgcn_readfirstlane((int)f);
                if (seq == call) break;
                bool out = (int)(seq - call) > 0;
                if (out) st32_agent_lane0(tmo, call); else out = give_up(tmo, call, t0, spins);
                if (out) {
                    while (ticket < NTICKETS) {
                        u32 r;
                        asm volatile("s_mov_b64 exec, 1\n\tglobal_atomic_add %0, %1, %2, off sc0\n\ts_mov_b64 exec, -1\n\ts_waitcnt vmcnt(0)" : "=&v"(r) : "v"(claim), "v"(1u) : "memory");
                        ticket = (u32)__builtin_amdgcn_readfirstlane((int)r) + tbase;
                    }
                    return false;
                }
                __builtin_amdgcn_s_sleep(POLL_SLEEP);
            }
        }
        return true;
    };
#define ZONE "v240", "v241", "v242", "v243", "v244", "v245", "v246", "v247", "v248", "v249", "v250", "v252", "v253", "v254", "v255"
#define ISSUE(it) do { const u32 k_ = (it) & 2047u; const bool r_ = ((it) & I_RAW) != 0u; \
        asm volatile("buffer_load_dwordx4 v[240:243], %0, %2, %3 offen sc0 sc1\n\ts_mov_b64 exec, %5\n\tbuffer_load_dwordx2 v[248:249], %1, %2, %3 offen offset:1024 sc0 sc1\n\t" \
                     "s_mov_b64 exec, %6\n\tbuffer_load_dwordx4 v[244:247], %0, %2, %4 offen sc0 sc1\n\t" \
                     "s_mov_b64 exec, 1\n\tglobal_atomic_add v250, %7, %8, off sc0\n\ts_mov_b64 exec, -1" \
                     :: "v"(l16), "v"(l8), "s"(rbox), "s"(k_ << 11), "s"(r_ ? (k_ << 11) + 1024u : vdelta + 256u + ((k_ >> 1) << 10)), \
                        "s"(r_ ? 1ull : (k_ & 32u) ? 0xffffffffull : ~0ull), "s"(r_ ? ~0ull : 0xffffffull), "v"(claim), "v"(1u) : "memory", ZONE); } while (0)
#define TAKE(n) asm volatile("s_waitcnt vmcnt(" #n ")\n\tv_mov_b64 %0, v[240:241]\n\tv_mov_b64 %1, v[242:243]\n\tv_mov_b64 %2, v[248:249]" : "=v"(A0), "=v"(A1), "=v"(B) :: "memory", ZONE)
    auto FLAG = [=](u32 k) { asm volatile("s_mov_b64 exec, 1\n\tbuffer_store_dword %0, off, %1, %2 sc1\n\ts_mov_b64 exec, -1" ::"v"(vcall), "s"(rdone), "s"(k << DONE_SHIFT) : "memory"); };
#define STORE2(d, so) asm volatile("buffer_store_dwordx4 %0, %2, %3, %4 offen sc1\n\tbuffer_store_dwordx4 %1, %2, %3, %4 offen offset:1024 sc1\n\ts_nop 1" \
                     ::"v"((u32x4){d[0], d[1], d[2], d[3]}), "v"((u32x4){d[4], d[5], d[6], d[7]}), "v"(l16), "s"(rstage), "s"(so) : "memory")
    static_assert(VMID - VWIN == 256 && 2 * VWIN == 24 * 16, "the asm above and below has these built in");
    u32 it = lookup(rk);
    if (!await(rk, it, false)) return;
    STAMP(2); GSTAMP(33);
    u32x2 A0, A1, B;
    const u32 none = NUNITS + gw;
    u32 prv = none;
#define PST() do { } while (0)
    const char* dense = rdy + RDY_DENSE + s * RDY_DENSE_SRC + l16;
#define GLANCE() asm volatile("global_load_dwordx4 v[252:255], %0, off sc0 sc1" :: "v"(dense) : "memory", ZONE)
    auto noted = [&]() {
        u32 t; u64 bad;
        asm volatile("v_xor_b32 v252, %2, v252\n\tv_xor_b32 v253, %2, v253\n\tv_xor_b32 v254, %2, v254\n\tv_xor_b32 v255, %2, v255\n\t"
                     "v_or3_b32 %0, v252, v253, v254\n\tv_or_b32 %0, %0, v255\n\tv_cmp_ne_u32 %1, 0, %0\n\ts_nop 1" : "=&v"(t), "=s"(bad) : "s"(call) : "memory", ZONE);
        known = bad ? (u32)__builtin_ctzll(bad) * 4u : 256u;
    };
    GLANCE();
    ISSUE(it);
    TAKE(0);
    noted();
    for (;;) {
        const u32 k = it & 2047u;
        u32 ticket = 0, hdr0, hdr1, nit = ~0u, d[8];
        asm volatile("v_readlane_b32 %0, v247, 11\n\tv_readlane_b32 %1, v244, 12" : "=&s"(hdr0), "=&s"(hdr1) :: "memory", ZONE);
        PST();
        bool glanced = false;
        bool ahead = false;
        auto advance = [&]() {
            asm volatile("s_waitcnt vmcnt(3)\n\tv_readfirstlane_b32 %0, v250" : "=s"(ticket) :: "memory", ZONE);
            ticket += tbase;
            if (ticket < NTICKETS) {
                nit = lookup(ticket);
                const u32 b = unit_task(nit & 2047u);
                if (__builtin_expect(b < known, 1)) {
                    if (__builtin_expect(b + 48u >= known && known < 256u, 0)) { GLANCE(); glanced = true; }
                    ISSUE(nit); ahead = true;
                }
            }
        };
        auto publish = [&](u32 u) {
            if (__builtin_expect(ahead, 1)) asm volatile("s_waitcnt vmcnt(4)" ::: "memory"); else asm volatile("s_waitcnt vmcnt(0)" ::: "memory");
            FLAG(u);
        };
        u32 last = k;
        if (it & I_RAW) {
            u32x2 V0, V1;
            asm volatile("v_mov_b64 %0, v[244:245]\n\tv_mov_b64 %1, v[246:247]" : "=v"(V0), "=v"(V1) :: "memory", ZONE);
            advance();
            d[0] = A0[0]; d[1] = A0[1]; d[2] = A1[0]; d[3] = A1[1]; d[4] = V0[0]; d[5] = V0[1]; d[6] = V1[0]; d[7] = V1[1];
            publish(prv);
        } else {
            const bool solo = (it & I_SOLO) != 0u;
            const u32 hdr = (k & 1u) ? hdr1 : hdr0, hdrs = solo ? hdr : hdr0 | hdr1;
            if (__builtin_expect((hdrs & H_RAW) != 0u, 0)) {
                last = solo ? k : k + 1u;
                asm volatile("s_nop 4\n\tbuffer_load_dwordx4 v[244:247], %0, %1, %2 offen sc0 sc1\n\ts_waitcnt vmcnt(0)\n\tds_write_b128 %3, v[244:247]\n\ts_waitcnt lgkmcnt(0)"
                             :: "v"(l16), "s"(rbox), "s"(vdelta + ((k >> 1) << 10)), "v"(img + l16) : "memory", ZONE);
                FLAG(prv);
#pragma nounroll
                for (u32 u = k;; ++u) {
                    u32x4 R0, R1; u32x2 C;
                    asm volatile("s_nop 4\n\tbuffer_load_dwordx4 %0, %3, %5, %6 offen sc0 sc1\n\tbuffer_load_dwordx4 %1, %3, %5, %6 offen offset:1024 sc0 sc1\n\t"
                                 "buffer_load_dwordx2 %2, %4, %5, %6 offen offset:1024 sc0 sc1\n\ts_waitcnt vmcnt(0)"
                                 : "=&v"(R0), "=&v"(R1), "=&v"(C) : "v"(l16), "v"(l8), "s"(rbox), "s"(u << 11) : "memory");
                    const u32 hu = (u & 1u) ? hdr1 : hdr0;
                    if (hu & H_RAW) {
#pragma unroll
                        for (int j = 0; j < 4; ++j) { d[j] = R0[j]; d[4 + j] = R1[j]; }
                    } else if (u & 32u) unpack_unit<1>(R0, C, hu, (u & 1u) != 0u, d, img + VMID, exp, lut, lane, []() {});
                    else unpack_unit<0>(R0, C, hu, (u & 1u) != 0u, d, img + VMID, exp, lut, lane, []() {});
                    if (u == last) break;
                    STORE2(d, stage_off(u));
                    asm volatile("s_waitcnt vmcnt(0)" ::: "memory");
                    FLAG(u);
                }
                advance();
                FLAG(none);
            } else {
                u32 vat = img + 256u + l16;
                if (__builtin_expect((hdrs & H_BIG) != 0u, 0)) {
                    asm volatile("s_nop 4\n\tbuffer_load_dwordx4 v[244:247], %0, %1, %2 offen sc0 sc1\n\ts_waitcnt vmcnt(0)" :: "v"(l16), "s"(rbox), "s"(vdelta + ((k >> 1) << 10)) : "memory", ZONE);
                    vat = img + l16;
                }
                asm volatile("ds_write_b128 %0, v[244:247]" :: "v"(vat) : "memory", ZONE);
                auto body = [&](auto oc) {
                    constexpr int O = decltype(oc)::value;
                    if (__builtin_expect(!solo, 1)) {
                        asm volatile("buffer_load_dwordx4 v[240:243], %0, %2, %3 offen offset:2048 sc0 sc1\n\ts_mov_b64 exec, %4\n\t"
                                     "buffer_load_dwordx2 v[248:249], %1, %2, %3 offen offset:3072 sc0 sc1\n\ts_mov_b64 exec, -1"
                                     :: "v"(l16), "v"(l8), "s"(rbox), "s"(k << 11), "s"(O ? 0xffffffffull : ~0ull) : "memory", ZONE);
                        unpack_unit<O>((u32x4){A0[0], A0[1], A1[0], A1[1]}, B, hdr0, false, d, img + VMID, exp, lut, lane,
                                       [&]() { asm volatile("s_waitcnt vmcnt(2)" ::: "memory"); FLAG(prv); });
                        WSTAMP(k >> 6); PST();
                        STORE2(d, stage_off(k));
                        TAKE(3);
                        PST();
                        prv = k; last = k + 1u;
                    }
                    advance();
                    PST();
                    unpack_unit<O>((u32x4){A0[0], A0[1], A1[0], A1[1]}, B, (last & 1u) ? hdr1 : hdr0, (last & 1u) != 0u, d, img + VMID, exp, lut, lane, [&]() { publish(prv); });
                };
                if (k & 32u) body(std::integral_constant<int, 1>{}); else body(std::integral_constant<int, 0>{});
            }
        }
        PST();
        STORE2(d, stage_off(last));
        if (__builtin_expect(ahead, 1)) {
            TAKE(4);
            if (__builtin_expect(glanced, 0)) noted();
            prv = last;
        } else {
            asm volatile("s_waitcnt vmcnt(0)" ::: "memory");
            FLAG(last);
            prv = none;
            if (nit == ~0u || !await(ticket, nit, false)) break;
            GLANCE();
            ISSUE(nit);
            TAKE(0);
            noted();
        }
        it = nit;
    }
#undef ISSUE
#undef TAKE
#undef GLANCE
#undef PST
#undef ZONE
#undef STORE2
    GSTAMP(32);
    STAMP(3);
}

__device__ __forceinline__ void gemm(const Args& a, u32 call, int xcd, int slot, LAS char* lds) {
    const int tid = threadIdx.x, wid = __builtin_amdgcn_readfirstlane(tid >> 6), lane = tid & 63;
    const int li = (xcd & 1) * GEMM_SLOTS + slot, ti = (xcd >> 1) * 4 + (li & 3), tj = li >> 2;
    const int row0 = ti * BM, col0 = tj * BN > DIM - BN ? DIM - BN : tj * BN;
    const int wr = wid >> 2, wc = wid & 3, fr = lane & 15, fq = lane >> 4;
    char* myws = a.me;
    u32* tmo = (u32*)(myws + TMO_OFF);
    const __amdgpu_buffer_rsrc_t rf = __builtin_amdgcn_make_buffer_rsrc(myws + DONE_OFF, 0, (int)(DBG_OFF - DONE_OFF), 0x00020000);
    const int vx = wid * 65536 + (row0 + lane) * 16, vw = (8 + wid) * 65536 + (col0 + lane) * 16;
    const int xoff = 2 * fq * XS + (wr * 128 + fr) * 16, woff = XB + 2 * fq * WS + (wc * 64 + fr) * 16, woff4 = woff + (256 - wc * 48) * 16;
    const int fu = lane < 16 ? ti * 2 + (lane >> 3) : lane < 40 ? 30 + (col0 >> 7) + (lane >> 3) : ti * 2;
    const int fw = (fu << (DONE_SHIFT - 2)) + (lane & 7);
    const u32* fl = (const u32*)(myws + DONE_OFF) + fw;
    bool dead = false;
    LAS u32* snap = (LAS u32*)(lds + 2 * STG);
    auto wait_tile = [&](int t) {
        if (dead) return;
        const u64 t0 = __builtin_amdgcn_s_memrealtime();
        for (u32 spins = 0;;) {
            const u32 seen = ld32_agent(fl + (t << (DONE_SHIFT + 4)));
            if (__all(seen == call)) break;
            if (give_up(tmo, call, t0, spins)) { dead = true; break; }
            __builtin_amdgcn_s_sleep(4);
        }
    };
#define RS(t) __builtin_amdgcn_make_buffer_rsrc(myws + STAGE_OFF, 0, (t) < NT ? (int)(NR * SRC_BYTES) : 0, 0x00020000)
#define DMA_X(t, b, i) __builtin_amdgcn_raw_ptr_buffer_load_lds(RS(t), (LAS void*)(lds + (b) * STG + wid * XS), 16, vx, (t) << 20, (i) * 1024, 17)
#define DMA_W(t, b, i) __builtin_amdgcn_raw_ptr_buffer_load_lds(RS(t), (LAS void*)(lds + (b) * STG + XB + wid * WS + ((i) == 4 ? 4096 : 0)), 16, vw, ((t) << 20) + ((i) == 4 ? 4096 : 0), ((i) & 3) * 1024, 17)
#define DMA_ALL(t, b) do { DMA_W(t, b, 0); DMA_W(t, b, 1); DMA_W(t, b, 2); DMA_W(t, b, 3); DMA_W(t, b, 4); DMA_X(t, b, 0); DMA_X(t, b, 1); DMA_X(t, b, 2); DMA_X(t, b, 3); } while (0)
    v4f acc[8][5];
#pragma unroll
    for (int m = 0; m < 8; ++m)
#pragma unroll
        for (int n = 0; n < 5; ++n) acc[m][n] = (v4f){0.f, 0.f, 0.f, 0.f};
    const float scale = __builtin_bit_cast(float, __builtin_amdgcn_readfirstlane(__builtin_bit_cast(int, a.sx[0] * a.sw[0])));
    __builtin_amdgcn_s_waitcnt(0x0F70);
    if (wid == 0) wait_tile(DMA_EARLY ? 1 : 0);
    if (wid == 0 && DMA_EARLY) wait_tile(0);
    asm volatile("s_waitcnt lgkmcnt(0)" ::: "memory");
    __builtin_amdgcn_s_barrier(); asm volatile("" ::: "memory");
    STAMP(2);
    DMA_ALL(0, 0);
    if (DMA_EARLY) { DMA_W(1, 1, 0); DMA_W(1, 1, 1); DMA_W(1, 1, 2); DMA_W(1, 1, 3); DMA_W(1, 1, 4); asm volatile("s_waitcnt vmcnt(5)" ::: "memory"); }
    else { if (wid == 0) wait_tile(1); asm volatile("s_waitcnt vmcnt(0)" ::: "memory"); }
    __builtin_amdgcn_s_barrier(); asm volatile("" ::: "memory");
    v8i wf[5], xa, xb, xc;
    xc = ldfrag(lds, xoff, XS); SCHED;
#pragma unroll
    for (int n = 0; n < 5; ++n) { wf[n] = ldfrag(lds, n < 4 ? woff + n * 256 : woff4, WS); SCHED; }
#define MF(m, n, xf) acc[m][n] = __builtin_amdgcn_mfma_scale_f32_16x16x128_f8f6f4(xf, wf[n], acc[m][n], 0, 1, 0, 0, 0, 0)
#define MM(m, xf) do { _Pragma("unroll") for (int n = 0; n < 5; ++n) MF(m, n, xf); } while (0)
#define PRIO(p) __builtin_amdgcn_s_setprio(p)
    for (int t = 0; t < NT - 1; ++t) {
        const int b = t & 1;
        LAS const char* cur = lds + b * STG; LAS const char* nxt = lds + (b ^ 1) * STG;
        xb = ldfrag(cur, xoff + 1 * 256, XS); SCHED;
        if (DMA_EARLY) {
            MF(0, 0, xc); SCHED; DMA_X(t + 1, b ^ 1, 0); SCHED; MF(0, 1, xc); SCHED; DMA_X(t + 1, b ^ 1, 1); SCHED;
            MF(0, 2, xc); SCHED; DMA_X(t + 1, b ^ 1, 2); SCHED; MF(0, 3, xc); SCHED; DMA_X(t + 1, b ^ 1, 3); SCHED; MF(0, 4, xc); SCHED;
            PRIO(2);
            xa = ldfrag(cur, xoff + 2 * 256, XS); SCHED; MM(1, xb); SCHED;
        } else {
            MF(0, 0, xc); SCHED; DMA_W(t + 1, b ^ 1, 0); SCHED; MF(0, 1, xc); SCHED; DMA_W(t + 1, b ^ 1, 1); SCHED; MF(0, 2, xc); SCHED; DMA_W(t + 1, b ^ 1, 2); SCHED;
            MF(0, 3, xc); SCHED; DMA_W(t + 1, b ^ 1, 3); SCHED; MF(0, 4, xc); SCHED; DMA_W(t + 1, b ^ 1, 4); SCHED;
            PRIO(2);
            xa = ldfrag(cur, xoff + 2 * 256, XS); SCHED;
            MF(1, 0, xb); SCHED; DMA_X(t + 1, b ^ 1, 0); SCHED; MF(1, 1, xb); SCHED; DMA_X(t + 1, b ^ 1, 1); SCHED;
            MF(1, 2, xb); SCHED; DMA_X(t + 1, b ^ 1, 2); SCHED; MF(1, 3, xb); SCHED; DMA_X(t + 1, b ^ 1, 3); SCHED; MF(1, 4, xb); SCHED;
        }
        xb = ldfrag(cur, xoff + 3 * 256, XS);
        if (wid == 0) {
            __builtin_amdgcn_raw_ptr_buffer_load_lds(rf, (LAS void*)snap, 4, fw * 4, ((t + 2) & 31) << (DONE_SHIFT + 6), 0, 16);
            __builtin_amdgcn_raw_ptr_buffer_load_lds(rf, (LAS void*)(snap + 64), 4, 0, (int)(TMO_OFF - DONE_OFF), 0, 16);
        }
        SCHED; MM(2, xa); SCHED;
        PRIO(1);
        xa = ldfrag(cur, xoff + 4 * 256, XS); SCHED; MM(3, xb); SCHED;
        xc = ldfrag(cur, xoff + 5 * 256, XS); SCHED; MM(4, xa); SCHED;
        PRIO(0);
        xa = ldfrag(cur, xoff + 6 * 256, XS); xb = ldfrag(cur, xoff + 7 * 256, XS); SCHED; MM(5, xc); SCHED;
        asm volatile("s_waitcnt vmcnt(0) lgkmcnt(0)" ::: "memory");
        if (wid == 0) {
            GSTAMP(t);
            if (t + 2 < NT && !__all(*(LAS volatile u32*)(snap + lane) == call)) wait_tile(t + 2);
            GSTAMP(32 + t);
            __builtin_amdgcn_s_waitcnt(0xC07F);
        }
        __builtin_amdgcn_s_barrier(); asm volatile("" ::: "memory"); SCHED;
        PRIO(3);
#pragma unroll
        for (int n = 0; n < 5; ++n) {
            MF(6, n, xa); SCHED;
            if (n == 0) xc = ldfrag(nxt, xoff, XS);
            if (DMA_EARLY) { if (n == 0) DMA_W(t + 2, b, 0); if (n == 1) DMA_W(t + 2, b, 1); if (n == 2) DMA_W(t + 2, b, 2); if (n == 3) DMA_W(t + 2, b, 3); if (n == 4) DMA_W(t + 2, b, 4); }
            SCHED; MF(7, n, xb); SCHED; wf[n] = ldfrag(nxt, n < 4 ? woff + n * 256 : woff4, WS); SCHED;
        }
    }
    STAMP(3);
    const float sc = *(LAS const volatile u32*)(snap + 64) == call ? __builtin_nanf("") : scale;
    const float nsl = scale * -1.44269504089f;
    const u32x4 ro = make_rsrc(a.out + (size_t)(row0 + wr * 128) * DIM + col0, (u32)(128 * DIM - col0) * 4u);
    const int v4 = (tj == 12 && wc == 0) ? 0x40000000 : (fq * 4 * DIM + wc * 64 + fr * 4) * 4, v1 = (fq * 4 * DIM + 256 + wc * 16 + fr) * 4;
    auto silu2 = [&](float a0, float a1, float& r0, float& r1) {
        const v2f v = {a0, a1}, y = v * sc, u = v * nsl;
        v2f e = {__builtin_amdgcn_exp2f(__builtin_amdgcn_fmed3f(u[0], -86.5617024533f, 86.5617024533f)),
                 __builtin_amdgcn_exp2f(__builtin_amdgcn_fmed3f(u[1], -86.5617024533f, 86.5617024533f))};
        e = e + 1.f;
        r0 = __builtin_amdgcn_rcpf(e[0]) * y[0]; r1 = __builtin_amdgcn_rcpf(e[1]) * y[1];
    };
#define EPI(m) do { _Pragma("unroll") for (int j = 0; j < 4; j += 2) { \
        float r0[5], r1[5]; \
        _Pragma("unroll") for (int n = 0; n < 5; ++n) silu2(acc[m][n][j], acc[m][n][j + 1], r0[n], r1[n]); \
        asm volatile("buffer_store_dwordx4 %0, %4, %6, %7 offen " OUT_FLAGS "\n\tbuffer_store_dwordx4 %1, %4, %6, %8 offen " OUT_FLAGS "\n\t" \
                     "buffer_store_dword %2, %5, %6, %7 offen " OUT_FLAGS "\n\tbuffer_store_dword %3, %5, %6, %8 offen " OUT_FLAGS \
                     :: "v"((v4f){r0[0], r0[1], r0[2], r0[3]}), "v"((v4f){r1[0], r1[1], r1[2], r1[3]}), "v"(r0[4]), "v"(r1[4]), "v"(v4), "v"(v1), \
                        "s"(ro), "s"(((m) * 16 + j) * DIM * 4), "s"(((m) * 16 + j + 1) * DIM * 4) : "memory"); \
    } } while (0)
    {
        LAS const char* cur = lds + ((NT - 1) & 1) * STG;
        xb = ldfrag(cur, xoff + 1 * 256, XS); SCHED; MM(0, xc); SCHED;
        xa = ldfrag(cur, xoff + 2 * 256, XS); SCHED; MM(1, xb); SCHED; EPI(0); SCHED; PRIO(2);
        xb = ldfrag(cur, xoff + 3 * 256, XS); SCHED; MM(2, xa); SCHED; EPI(1); SCHED;
        xa = ldfrag(cur, xoff + 4 * 256, XS); SCHED; MM(3, xb); SCHED; EPI(2); SCHED; PRIO(1);
        xb = ldfrag(cur, xoff + 5 * 256, XS); SCHED; MM(4, xa); SCHED; EPI(3); SCHED;
        xa = ldfrag(cur, xoff + 6 * 256, XS); SCHED; MM(5, xb); SCHED; EPI(4); SCHED; PRIO(0);
        xb = ldfrag(cur, xoff + 7 * 256, XS); SCHED; MM(6, xa); SCHED; EPI(5); SCHED;
        MM(7, xb); SCHED; EPI(6); SCHED; EPI(7);
    }
    STAMP(4);
}
}

__global__ void __launch_bounds__(512, 2) fused_allgather_gemm(Args a) {
    extern __shared__ __attribute__((aligned(16))) char lds_dyn[];
    LAS char* lds = (LAS char*)lds_dyn;
    asm volatile("" :: "s"(a.x), "s"(a.w), "s"(a.me), "s"(a.rank));
    STAMP(0);
    const int xcd = blockIdx.x & 7, slot = blockIdx.x >> 3;
    u32* cnt = (u32*)(a.me + CNT_OFF) + blockIdx.x;
    if (__builtin_expect(slot < GEMM_SLOTS, 1)) {
        const u32 k0 = task_unit(blockIdx.x, (u32)__builtin_amdgcn_readfirstlane(threadIdx.x >> 6));
        u32x4 r0, r1;
        if (blockIdx.x >= LATE_TASKS) __builtin_amdgcn_s_sleep(LATE_SLEEP);
        load_unit(a, k0, threadIdx.x & 63, r0, r1);
        const u32 prev = __hip_atomic_load(cnt, __ATOMIC_RELAXED, __HIP_MEMORY_SCOPE_AGENT);
        SCHED;
        const u32 call = stage_out(a, lds, blockIdx.x, blockIdx.x < 64 && ((RAW_TASKS >> blockIdx.x) & 1ull), prev, r0, r1);
        if (slot < PULL_SLOTS) {
            load_unit(a, k0 + GEMM_SLOTS * 64u, threadIdx.x & 63, r0, r1);
            stage_out(a, lds, blockIdx.x + GEMM_SLOTS * 8, false, prev, r0, r1);
        }
        if (__builtin_amdgcn_readfirstlane(threadIdx.x) == 0) st32_agent_lane0(cnt, call);
        STAMP(1);
        gemm(a, call, xcd, slot, lds);
    } else {
        pull(a, cnt, (slot - GEMM_SLOTS) * 8 + xcd, lds);
    }
}

extern "C" void kernel_launch(int rank, int nranks, ncclComm_t comm, hipStream_t stream,
                              void* const* d_in, const int* in_sizes, int n_in,
                              void* d_out, int out_size, const int* in_dtypes, int out_dtype,
                              void* const* d_out_all, void* const* d_ws_all, size_t ws_bytes) {
    if (nranks != NR || (unsigned)rank >= (unsigned)NR || n_in != 4 || ws_bytes < WS_NEED || out_size != DIM * DIM ||
        in_sizes[0] != DIM * KS || in_sizes[1] != DIM * KS) {
        fprintf(stderr, "kernel_launch: unexpected problem shape\n");
        return;
    }
    static bool attr_set[NR] = {};
    if (!attr_set[rank]) {
        HIP_CHECK(hipFuncSetAttribute((const void*)fused_allgather_gemm, hipFuncAttributeMaxDynamicSharedMemorySize, LDS_BYTES));
        attr_set[rank] = true;
    }
    Args a;
    a.x = (const unsigned char*)d_in[0]; a.w = (const unsigned char*)d_in[1];
    a.sx = (const float*)d_in[2]; a.sw = (const float*)d_in[3];
    a.out = (float*)d_out;
    for (int p = 0; p < NR; ++p) a.ws[p] = (char*)d_ws_all[p];
    a.me = a.ws[rank]; a.rank = rank; a.pad = 0;
    fused_allgather_gemm<<<256, 512, LDS_BYTES, stream>>>(a);
    HIP_CHECK(hipGetLastError());
}
```

```cpp
#include <hip/hip_runtime.h>
#include <rccl/rccl.h>
#include <cstdio>
#include <type_traits>

#define HIP_CHECK(expr) do { hipError_t _e = (expr); if (_e != hipSuccess) { \
    fprintf(stderr, "HIP error %s at %s:%d\n", hipGetErrorString(_e), __FILE__, __LINE__); \
    return; } } while (0)

#define LAS __attribute__((address_space(3)))
#define GAS __attribute__((address_space(1)))
typedef unsigned u32;
typedef unsigned long long u64;
typedef int v2i __attribute__((ext_vector_type(2)));
typedef int v4i __attribute__((ext_vector_type(4)));
typedef int v8i __attribute__((ext_vector_type(8)));
typedef float v2f __attribute__((ext_vector_type(2)));
typedef float v4f __attribute__((ext_vector_type(4)));
typedef unsigned u32x2 __attribute__((ext_vector_type(2)));
typedef unsigned u32x4 __attribute__((ext_vector_type(4)));

namespace {
constexpr int NR = 8;
constexpr int DIM = 4096;
constexpr int KS = DIM / NR;
constexpr int NT = 32;
constexpr u32 NUNITS = NT * 64, NBLOCKS = NUNITS / 2;
constexpr size_t STAGE_OFF = 0;
constexpr size_t STAGE_BYTES = (size_t)(NT + 2) << 20;
constexpr size_t SRC_BYTES = (size_t)NUNITS * 2048;
constexpr size_t OUTBOX_OFF = STAGE_OFF + STAGE_BYTES;
constexpr size_t VBOX_OFF = OUTBOX_OFF + 2 * SRC_BYTES;
constexpr size_t VBOX_HALF = (size_t)NBLOCKS * 1024;
constexpr size_t RDY_OFF = VBOX_OFF + 2 * VBOX_HALF + 4096;
constexpr u32 RDY_SRC = 65536 + 2304, RDY_DENSE = NR * RDY_SRC, RDY_DENSE_SRC = 1280;
constexpr size_t RDY_HALF = RDY_DENSE + NR * RDY_DENSE_SRC;
#define DONE_SHIFT 7
constexpr size_t DONE_OFF = RDY_OFF + 2 * RDY_HALF;
constexpr size_t CLAIM_OFF = DONE_OFF + ((size_t)(NUNITS + 128 + 512) << DONE_SHIFT);
constexpr size_t CNT_OFF = CLAIM_OFF + NR * 128;
constexpr size_t TMO_OFF = CNT_OFF + 256 * 4;
constexpr size_t DBG_OFF = TMO_OFF + 128;
constexpr size_t WS_NEED = DBG_OFF + 256 * 64 + 256 * 8 * 80 * 8;
#define OUT_FLAGS "sc0 sc1"
#define LATE_TASKS 32
#define OWN_FIRST 3
#define OWN_LAST 18
#define GEMM_SLEEP 4
#define LATE2_SLEEP 20
#define LATE_SLEEP 32
#define POLL_SLEEP 4
constexpr u32 VMID = 448, VWIN = 192;
constexpr u32 H_RAW = 0x80000000u, H_BIG = 0x40000000u;
constexpr u64 DEADLINE = 150000000ull;
constexpr int BM = 256, BN = 320, XS = BM * 16, WS = BN * 16, XB = 8 * XS, WB = 8 * WS, STG = XB + WB;
constexpr int LDS_BYTES = 2 * STG + 2048 + 64;
constexpr int GEMM_SLOTS = 26, PULL_SLOTS = 6;

#define STAG_A 0
#define PUSH_TASKS 12
#define SPLIT_HEAD 0
#define SPLIT_TAIL 0
constexpr u32 NITEMS = NBLOCKS + 256;
constexpr u32 I_SOLO = 0x2000u;
constexpr u32 NPUSH = PUSH_TASKS * 8;
static_assert(NPUSH <= 128, "pushed units stay inside K-tiles 0 and 1");
struct Order { unsigned short item[NITEMS]; u32 n; };
constexpr int lag16(int p) { return STAG_A * 16 * p / 15; }
constexpr Order make_order() {
    Order r = {};
    int n = 0;
    for (int key = 0; n < (int)NBLOCKS; ++key)
        for (int p = 0; p < 16; ++p) {
            const int d = key - lag16(p);
            if (d >= 0 && d % 16 == 0 && d / 16 < NT)
                for (int o = 0; o < 2; ++o, ++n) {
                    const u32 k = ((d / 16) * 32 + o * 16 + p) * 2;
                    if ((d / 16) * 8 + (p >> 1) < PUSH_TASKS) continue;
                    if (n < SPLIT_HEAD || n >= (int)NBLOCKS - SPLIT_TAIL) { r.item[r.n++] = (unsigned short)(k | I_SOLO); r.item[r.n++] = (unsigned short)((k + 1) | I_SOLO); }
                    else r.item[r.n++] = (unsigned short)k;
                }
        }
    return r;
}
__device__ constexpr Order ORDER = make_order();
constexpr u32 NTICKETS = make_order().n;
static_assert(NTICKETS <= NITEMS, "order table too small");

struct Args {
    const unsigned char* x; const unsigned char* w; const float* sx; const float* sw; float* out;
    char* me; char* ws[NR]; int rank; int pad;
};

__device__ __forceinline__ void st16_sys(void* p, u32x4 v) {
    asm volatile("global_store_dwordx4 %0, %1, off sc0 sc1\n\ts_nop 1" ::"v"(p), "v"(v) : "memory");
}
__device__ __forceinline__ void st16_agent(void* p, u32x4 v) {
    asm volatile("global_store_dwordx4 %0, %1, off sc1\n\ts_nop 1" ::"v"(p), "v"(v) : "memory");
}
__device__ __forceinline__ u32 ld32_agent(const void* p) {
    u32 v;
    asm volatile("global_load_dword %0, %1, off sc1\n\ts_waitcnt vmcnt(0)" : "=&v"(v) : "v"(p) : "memory");
    return v;
}
__device__ __forceinline__ void st32_agent_lane0(void* p, u32 v) {
    asm volatile("s_mov_b64 exec, 1\n\tglobal_store_dword %0, %1, off sc1\n\ts_mov_b64 exec, -1" ::"v"(p), "v"(v) : "memory");
}
__device__ __forceinline__ bool give_up(u32* tmo, u32 call, u64 t0, u32& spins) {
    if ((++spins & 15u) != 0u) return false;
    if ((u32)__builtin_amdgcn_readfirstlane((int)ld32_agent(tmo)) == call || __builtin_amdgcn_s_memrealtime() - t0 > DEADLINE) {
        st32_agent_lane0(tmo, call);
        return true;
    }
    return false;
}
template <class T> __device__ __forceinline__ T* uniform_ptr(T* p) {
    const u64 v = (u64)p;
    return (T*)(((u64)(u32)__builtin_amdgcn_readfirstlane((int)(u32)(v >> 32)) << 32) | (u32)__builtin_amdgcn_readfirstlane((int)(u32)v));
}
__device__ __forceinline__ v8i ldfrag(LAS const char* p, int off, int stride) {
    v4i a = *(LAS const v4i*)(p + off), b = *(LAS const v4i*)(p + off + stride);
    return __builtin_shufflevector(a, b, 0, 1, 2, 3, 4, 5, 6, 7);
}
#define SCHED __builtin_amdgcn_sched_barrier(0)
#define STAMP(i) do { } while (0)
#define GSTAMP(i) do { } while (0)
#define WSTAMP(i) do { } while (0)

__device__ __forceinline__ u32 wave_incl_scan(u32 x) {
    x += (u32)__builtin_amdgcn_update_dpp(0, (int)x, 0x111, 0xf, 0xf, false);
    x += (u32)__builtin_amdgcn_update_dpp(0, (int)x, 0x112, 0xf, 0xf, false);
    x += (u32)__builtin_amdgcn_update_dpp(0, (int)x, 0x114, 0xf, 0xf, false);
    x += (u32)__builtin_amdgcn_update_dpp(0, (int)x, 0x118, 0xf, 0xf, false);
    x += (u32)__builtin_amdgcn_update_dpp(0, (int)x, 0x142, 0xa, 0xf, false);
    x += (u32)__builtin_amdgcn_update_dpp(0, (int)x, 0x143, 0xc, 0xf, false);
    return x;
}
__device__ __forceinline__ u32 rotr(u32 x, u32 n) { return __builtin_amdgcn_alignbit(x, x, n); }
__device__ __forceinline__ u32 bfi(u32 m, u32 a, u32 b) { return __builtin_amdgcn_bitop3_b32(m, a, b, 0xca); }
__device__ __forceinline__ u32 and3(u32 a, u32 b, u32 c) { return __builtin_amdgcn_bitop3_b32(a, b, c, 0x80); }
__device__ __forceinline__ u32 lds32(u32 addr) { return *(LAS const u32*)(size_t)addr; }
__device__ __forceinline__ u32x4 make_rsrc(const void* p, u32 bytes) {
    const u64 v = (u64)p;
    return (u32x4){(u32)__builtin_amdgcn_readfirstlane((int)(u32)v), (u32)__builtin_amdgcn_readfirstlane((int)(u32)(v >> 32)) & 0xffffu, bytes, 0x00020000u};
}

constexpr int P0_VAR = 2560, P0_LUT = P0_VAR + 1536, P0_WAVE = P0_LUT + 128;

template <int O>
__device__ __forceinline__ void pack_unit(const u32 (&d)[8], char* slot, char* vslot, u32 h, LAS char* img, u32 lut, int lane) {
    constexpr int SH = O ? 2 : 3;
    constexpr u32 EM = O ? 0x1f1f1f1fu : 0x0f0f0f0fu;
    u32 ex[8], sum = 0;
#pragma unroll
    for (int j = 0; j < 8; ++j) ex[j] = (d[j] >> SH) & EM;
#pragma unroll
    for (int j = 0; j < 8; j += 2) sum = __builtin_amdgcn_udot4(ex[j], 0x01010101u, sum, false);
    const u32 tot = (u32)__builtin_amdgcn_readlane((int)wave_incl_scan(sum), 63);
    const u32 lo = tot > 50u ? (tot - 50u) >> 10 : 0u;
    const u32 bias = (0x80u - lo) * 0x01010101u;
    u32 P[2] = {0u, 0u}, X = 0, n2 = 0, n3 = 0, b3 = 0, q[8], n2j[8];
#pragma unroll
    for (int j = 0; j < 8; ++j) {
        const u32 a7 = ex[j] + bias;
        const u32 t = ((~a7 | (a7 - 0x03030303u)) & 0x80808080u) >> 7;
        P[j >> 2] |= ((a7 | t | (t << 1)) & 0x03030303u) << (2 * (j & 3));
        const u32 sc = a7 & ((a7 | 0x80808080u) >> 5) & 0x07070707u;
        X = (X >> 1) | ((~(a7 + 0x07070707u) | (a7 - 0x04040404u)) & 0x80808080u);
        const u32 cz = __builtin_amdgcn_perm(0x07070707u, sc, lds32(__builtin_amdgcn_udot4(t, 0x20100804u, lut, false)));
        q[j] = __builtin_amdgcn_udot4(__builtin_amdgcn_perm(0x00010303u, 0x03030302u, cz), 0x40100401u, 0u, false);
        const u32 f = __builtin_amdgcn_perm(0x00000101u, 0x01010100u, cz);
        const u32 c3 = __builtin_amdgcn_perm(0u, cz, lds32(__builtin_amdgcn_udot4(f, 0x20100804u, lut + 64u, false)));
        b3 |= __builtin_amdgcn_udot4(c3, 0x003e0801u, c3 >> 15, false) << (n3 & 31u);
        n3 = __builtin_amdgcn_udot4(f, 0x03030303u, n3, false);
        n2j[j] = n2;
        n2 = __builtin_amdgcn_udot4(t, 0x02020202u, n2, false);
    }
    const u32 qa = q[0] | (q[1] << n2j[1]) | (q[2] << n2j[2]) | (q[3] << n2j[3]);
    const u32 qb = q[4] | (q[5] << (n2j[5] - n2j[4])) | (q[6] << (n2j[6] - n2j[4])) | (q[7] << (n2j[7] - n2j[4]));
    const u64 b2 = (u64)qa | ((u64)qb << n2j[4]);
    const u32 both = n2 | (n3 << 16), incl = wave_incl_scan(both), pre = incl - both;
    const u32 total = (u32)__builtin_amdgcn_readlane((int)incl, 63);
    const u32 len2 = (((total & 0xffffu) + 31u) >> 5) << 2, len3 = (((total >> 16) + 31u) >> 5) << 2;
    const u32 used = 4u + len2 + len3;
    bool raw = __any(n3 > 30u) || used > VMID - 16u;
    u32 nexc = 0;
    if (!raw) {
        const u32 p2 = pre & 0xffffu, p3 = pre >> 16;
        LAS u32* w2 = (LAS u32*)(img + (h ? VMID + 4u : VMID - 4u - len2) + ((p2 >> 5) << 2));
        LAS u32* w3 = (LAS u32*)(img + (h ? VMID + 4u + len2 : VMID - used) + ((p3 >> 5) << 2));
        const u64 u2 = b2 << (p2 & 31u), u3 = (u64)b3 << (p3 & 31u);
        __hip_atomic_fetch_or(w2, (u32)u2, __ATOMIC_RELAXED, __HIP_MEMORY_SCOPE_WAVEFRONT);
        __hip_atomic_fetch_or(w2 + 1, (u32)(u2 >> 32), __ATOMIC_RELAXED, __HIP_MEMORY_SCOPE_WAVEFRONT);
        __hip_atomic_fetch_or(w2 + 2, (u32)(b2 >> 32) >> 1 >> (31u - (p2 & 31u)), __ATOMIC_RELAXED, __HIP_MEMORY_SCOPE_WAVEFRONT);
        __hip_atomic_fetch_or(w3, (u32)u3, __ATOMIC_RELAXED, __HIP_MEMORY_SCOPE_WAVEFRONT);
        __hip_atomic_fetch_or(w3 + 1, (u32)(u3 >> 32), __ATOMIC_RELAXED, __HIP_MEMORY_SCOPE_WAVEFRONT);
        u64 bal;
        while ((bal = __ballot(X != 0u)) != 0ull) {
            const u32 slotn = nexc + __builtin_amdgcn_mbcnt_hi((u32)(bal >> 32), __builtin_amdgcn_mbcnt_lo((u32)bal, 0u));
            nexc += (u32)__builtin_popcountll(bal);
            if (nexc > 64u || used + 2u * nexc > VMID - 16u) { raw = true; break; }
            if (X != 0u) {
                const u32 k = (u32)__builtin_ctz(X), j = k & 7u, i8 = k & 24u;
                const u32 e01 = (j & 1) ? ex[1] : ex[0], e23 = (j & 1) ? ex[3] : ex[2], e45 = (j & 1) ? ex[5] : ex[4], e67 = (j & 1) ? ex[7] : ex[6];
                const u32 e03 = (j & 2) ? e23 : e01, e47 = (j & 2) ? e67 : e45, ej = (j & 4) ? e47 : e03;
                const u32 n01 = (j & 1) ? n2j[1] : n2j[0], n23 = (j & 1) ? n2j[3] : n2j[2], n45 = (j & 1) ? n2j[5] : n2j[4], n67 = (j & 1) ? n2j[7] : n2j[6];
                const u32 n03 = (j & 2) ? n23 : n01, n47 = (j & 2) ? n67 : n45, nj = (j & 4) ? n47 : n03;
                const u32 Pj = ((j & 4) ? P[1] : P[0]) >> (2u * (j & 3u));
                const u32 pos = ((p2 + nj) >> 1) + (u32)__builtin_popcount(Pj & (Pj >> 1) & 0x01010101u & ((1u << i8) - 1u));
                *(LAS unsigned short*)(img + (h ? VMID + used + 2u * slotn : VMID - used - 2u - 2u * slotn)) = (unsigned short)(pos | (((ej >> i8) & 31u) << 11));
                X &= X - 1u;
            }
        }
    }
    if (raw) {
        st16_sys(slot + lane * 16, (u32x4){d[0], d[1], d[2], d[3]});
        st16_sys(slot + 1024 + lane * 16, (u32x4){d[4], d[5], d[6], d[7]});
        asm volatile("s_mov_b64 exec, 1\n\tglobal_store_dword %0, %1, off sc0 sc1\n\ts_mov_b64 exec, -1" ::"v"(vslot + (h ? VMID : VMID - 4u)), "v"(H_RAW) : "memory");
        return;
    }
    const u32 nvar = (used + 2u * nexc + 15u) >> 4;
    if (lane == 0) *(LAS u32*)(img + (h ? VMID : VMID - 4u)) = lo | (nexc << 5) | (len3 << 10) | (len2 << 17) | (nvar * 16u > VWIN ? H_BIG : 0u);
    const u32 first = h ? VMID : VMID - nvar * 16u;
    const u32x4 v = *(LAS const u32x4*)(img + first + lane * 16);
    if (O == 0) {
        u32x4 sm;
#pragma unroll
        for (int k = 0; k < 4; ++k) sm[k] = bfi(0x87878787u, d[2 * k], rotr(d[2 * k + 1], 28));
        st16_sys(slot + lane * 16, sm);
        asm volatile("global_store_dwordx2 %0, %1, off offset:1024 sc0 sc1\n\ts_nop 1" ::"v"(slot + lane * 8), "v"((u32x2){P[0], P[1]}) : "memory");
    } else {
        u32x4 sm;
        sm[0] = bfi(0x60606060u, d[6] << 5, bfi(0x83838383u, d[0], rotr(d[1], 29)));
        sm[1] = bfi(0x60606060u, d[7] << 5, bfi(0x83838383u, d[2], rotr(d[3], 29)));
        sm[2] = bfi(0x60606060u, bfi(0x40404040u, d[6] >> 1, d[7] >> 2), bfi(0x83838383u, d[4], rotr(d[5], 29)));
        sm[3] = P[0];
        st16_sys(slot + lane * 16, sm);
        const u32 up = __builtin_amdgcn_permlane32_swap(P[1], P[1], false, false)[1];
        if (lane < 32) asm volatile("global_store_dwordx2 %0, %1, off offset:1024 sc0 sc1\n\ts_nop 1" ::"v"(slot + lane * 8), "v"((u32x2){P[1], up}) : "memory");
    }
    if ((u32)lane < nvar) st16_sys(vslot + first + lane * 16, v);
}

__device__ __forceinline__ u32 spread3(u32 v) {
    const u32 x = bfi(0x3fu, v, v << 10);
    return bfi(0x00070007u, x, x << 5) & 0x07070707u;
}
__device__ __forceinline__ u32 spread2(u32 v) {
    const u32 x = bfi(0xfu, v, v << 12);
    return bfi(0x00030003u, x, x << 6);
}

template <int O, class F>
__device__ __forceinline__ void unpack_unit(const u32x4 A, const u32x2 B, const u32 hdr, const bool H, u32 (&d)[8], u32 mid, u32 exp, u32 lut, int lane, F&& midway) {
    constexpr int SH = O ? 2 : 3;
    const u32 P[2] = {O ? A[3] : B[0], O ? __builtin_amdgcn_permlane32_swap(B[0], B[1], false, false)[0] : B[1]};
    const u32 lo = hdr & 31u, nexc = (hdr >> 5) & 127u, len3 = (hdr >> 10) & 0x1fcu, len2 = (hdr >> 17) & 0x1fcu;
    const u32 s2 = H ? mid + 4u : mid - 4u - len2, s3 = H ? s2 + len2 : s2 - len3;
    u32 c = *(LAS const unsigned short*)(size_t)(s2 + (u32)lane * 2u);
    const u32 E[2] = {and3(P[0], P[0] >> 1, 0x55555555u), and3(P[1], P[1] >> 1, 0x55555555u)};
    const u32 cnt = (u32)__builtin_popcount(E[0]) + (u32)__builtin_popcount(E[1]);
    const u32 prim = ((lo * 0x01010101u + 0x00020100u) << SH);
    const u32 b8 = ((lo + 24u) & 31u) * 0x01010101u;
    const u32 t3lo = ((b8 + 0x03020100u) & 0x1f1f1f1fu) << SH, t3hi = ((b8 + 0x07060504u) & 0x1f1f1f1fu) << SH;
    const u32 t2 = ((b8 + 0x000b0607u) & 0x001f1f1fu) << SH;
    u32 at = 0, m3 = 0;
    for (u32 base = 0;;) {
        const u32 c0 = spread2(c), c1 = spread2(c >> 8);
        const u32 f0 = and3(c0, c0 >> 1, 0x01010101u), f1 = and3(c1, c1 >> 1, 0x01010101u);
        const u32 k0 = __builtin_amdgcn_udot4(f0, 0x03030303u, 0u, false), k = __builtin_amdgcn_udot4(f1, 0x03030303u, k0, false);
        const u32 g0 = lds32(__builtin_amdgcn_udot4(f0, 0x20100804u, lut, false)), g1 = lds32(__builtin_amdgcn_udot4(f1, 0x20100804u, lut, false));
        const u32 both = k | (cnt << 16), incl = wave_incl_scan(both), pre = incl - both;
        if (base == 0) at = exp + (pre >> 16);
        const u32 bit = m3 + (pre & 0xffffu);
        LAS const volatile u32* wp = (LAS const volatile u32*)(size_t)(s3 + ((bit >> 5) << 2));
        const u32 w0 = wp[0], w1 = wp[1];
        const u32 w = __builtin_amdgcn_alignbit(w1, w0, bit);
        const u32 e0 = __builtin_amdgcn_perm(t3hi, t3lo, spread3(w)), e1 = __builtin_amdgcn_perm(t3hi, t3lo, spread3(w >> k0));
        *(LAS u32x2*)(size_t)(exp + base * 4u + (u32)lane * 8u) = (u32x2){__builtin_amdgcn_perm(e0, t2, (c0 & 0x03030303u) ^ g0), __builtin_amdgcn_perm(e1, t2, (c1 & 0x03030303u) ^ g1)};
        base += 128u;
        if (__builtin_expect(base >= len2, 1)) break;
        m3 += (u32)__builtin_amdgcn_readlane((int)incl, 63) & 0xffffu;
        c = *(LAS const unsigned short*)(size_t)(s2 + base + (u32)lane * 2u);
    }
    if (nexc) {
        if ((u32)lane < nexc) {
            const u32 e = *(LAS const unsigned short*)(size_t)(H ? s3 + len3 + 2u * (u32)lane : s3 - 2u - 2u * (u32)lane);
            *(LAS unsigned char*)(size_t)(exp + (e & 0x7ffu)) = (unsigned char)((e >> 11) << SH);
        }
    }
    midway();
    u32 ptr[8], dep[8], ex[8];
#pragma unroll
    for (int j = 0; j < 8; ++j) {
        const int g = j & 3;
        const u32 Eh = g < 2 ? E[j >> 2] : E[j >> 2] >> 4;
        const u32 m = Eh & ((g & 1) ? 0x04040404u : 0x01010101u);
        dep[j] = lds32(__builtin_amdgcn_udot4(m, (g & 1) ? 0x08040201u : 0x20100804u, lut, false));
        ptr[j] = at;
        at += (u32)__builtin_popcount(m);
    }
    u32x2 wn[8];
    asm volatile("ds_read2_b32 %0, %8 offset1:1\n\tds_read2_b32 %1, %9 offset1:1\n\tds_read2_b32 %2, %10 offset1:1\n\tds_read2_b32 %3, %11 offset1:1\n\t"
                 "ds_read2_b32 %4, %12 offset1:1\n\tds_read2_b32 %5, %13 offset1:1\n\tds_read2_b32 %6, %14 offset1:1\n\tds_read2_b32 %7, %15 offset1:1\n\t"
                 "s_waitcnt lgkmcnt(0)"
                 : "=&v"(wn[0]), "=&v"(wn[1]), "=&v"(wn[2]), "=&v"(wn[3]), "=&v"(wn[4]), "=&v"(wn[5]), "=&v"(wn[6]), "=&v"(wn[7])
                 : "v"(ptr[0] & ~3u), "v"(ptr[1] & ~3u), "v"(ptr[2] & ~3u), "v"(ptr[3] & ~3u), "v"(ptr[4] & ~3u), "v"(ptr[5] & ~3u), "v"(ptr[6] & ~3u), "v"(ptr[7] & ~3u)
                 : "memory");
#pragma unroll
    for (int j = 0; j < 8; ++j)
        ex[j] = __builtin_amdgcn_perm(__builtin_amdgcn_alignbyte(wn[j][1], wn[j][0], ptr[j]), prim, ((P[j >> 2] >> (2 * (j & 3))) & 0x03030303u) ^ dep[j]);
    if (O == 0) {
#pragma unroll
        for (int k = 0; k < 4; ++k) { d[2 * k] = (A[k] & 0x87878787u) | ex[2 * k]; d[2 * k + 1] = (rotr(A[k], 4) & 0x87878787u) | ex[2 * k + 1]; }
    } else {
#pragma unroll
        for (int k = 0; k < 3; ++k) { d[2 * k] = (A[k] & 0x83838383u) | ex[2 * k]; d[2 * k + 1] = (rotr(A[k], 3) & 0x83838383u) | ex[2 * k + 1]; }
        d[6] = (bfi(0x80808080u, A[2] << 1, A[0] >> 5) & 0x83838383u) | ex[6];
        d[7] = (bfi(0x80808080u, A[2] << 2, A[1] >> 5) & 0x83838383u) | ex[7];
    }
}

__host__ __device__ constexpr u32 task_unit(u32 b, u32 wid) { return (b >> 3) * 64u + (wid >> 2) * 32u + (b & 7u) * 4u + (wid & 3u); }
__host__ __device__ constexpr u32 unit_task(u32 k) { return (k >> 6) * 8u + ((k & 31u) >> 2); }
__device__ __forceinline__ u32 stage_off(u32 k) { return ((k >> 5) << 19) | ((k & 31u) << 11); }

__device__ __forceinline__ void load_unit(const Args& a, u32 k, int lane, u32x4& r0, u32x4& r1) {
    const u32 t = k >> 6, rows = (k & 31u) * 128u;
    const unsigned char* src = (k & 32u) ? a.w + (size_t)(16u * t + (lane >> 3)) * DIM + rows + (lane & 7) * 16
                                         : a.x + (size_t)(rows + lane) * KS + 16u * t;
    r0 = *(const u32x4*)src; r1 = *(const u32x4*)(src + 32768);
}
__device__ __forceinline__ void retile(u32 k, const u32x4 r0, const u32x4 r1, u32 (&d)[8], LAS char* wl, int lane) {
    if ((k & 32u) == 0u) {
#pragma unroll
        for (int j = 0; j < 4; ++j) { d[j] = r0[j]; d[4 + j] = r1[j]; }
    } else {
        *(LAS u32x4*)(wl + (lane >> 3) * 160 + (lane & 7) * 16) = r0;
        *(LAS u32x4*)(wl + (8 + (lane >> 3)) * 160 + (lane & 7) * 16) = r1;
        LAS char* tb = wl + ((lane & 15) >> 1) * 160 + (lane >> 4) * 16 + (lane & 1) * 8;
        asm volatile("" ::: "memory");
        const v2i c00 = __builtin_amdgcn_ds_read_tr8_b64_v2i32((LAS v2i*)tb);
        const v2i c01 = __builtin_amdgcn_ds_read_tr8_b64_v2i32((LAS v2i*)(tb + 8 * 160));
        const v2i c10 = __builtin_amdgcn_ds_read_tr8_b64_v2i32((LAS v2i*)(tb + 64));
        const v2i c11 = __builtin_amdgcn_ds_read_tr8_b64_v2i32((LAS v2i*)(tb + 8 * 160 + 64));
        const u32 g0 = (k & 31u) * 2u, g1 = g0 + 1u;
        const int mixed = ((lane & 15) * 4 + (lane >> 4)) * 4;
        const int s0 = g0 % 5u == 4u ? lane * 4 : mixed, s1 = (g1 % 5u == 4u || g1 == 63u) ? lane * 4 : mixed;
        d[0] = __builtin_amdgcn_ds_bpermute(s0, c00[0]); d[1] = __builtin_amdgcn_ds_bpermute(s0, c00[1]);
        d[2] = __builtin_amdgcn_ds_bpermute(s0, c01[0]); d[3] = __builtin_amdgcn_ds_bpermute(s0, c01[1]);
        d[4] = __builtin_amdgcn_ds_bpermute(s1, c10[0]); d[5] = __builtin_amdgcn_ds_bpermute(s1, c10[1]);
        d[6] = __builtin_amdgcn_ds_bpermute(s1, c11[0]); d[7] = __builtin_amdgcn_ds_bpermute(s1, c11[1]);
    }
}
__device__ __forceinline__ u32 slot_unit(u32 k, u32 par) { return k + (k < 128u ? par * NUNITS : 0u); }
__device__ __forceinline__ void pack_tables(LAS char* wl, int lane) {
    u32 sel = 0x04040404u; int n = 0;
#pragma unroll
    for (int i = 0; i < 4; ++i) if ((lane >> i) & 1) { sel = (sel & ~(0xffu << (8 * n))) | ((u32)i << (8 * n)); ++n; }
    if (lane & 16) sel |= (sel & 0x04040404u) << 1;
    if (lane < 32) *(LAS u32*)(wl + P0_LUT + lane * 4) = sel;
}
__device__ __forceinline__ void stage_unit(const Args& a, LAS char* wl, u32 k, bool plain, u32 par, const u32 (&d)[8], int lane) {
    char* own = a.me + STAGE_OFF + (size_t)a.rank * 65536 + stage_off(slot_unit(k, par)) + lane * 16;
    st16_agent(own, (u32x4){d[0], d[1], d[2], d[3]}); st16_agent(own + 1024, (u32x4){d[4], d[5], d[6], d[7]});
    if (!plain) {
        char* slot = a.me + OUTBOX_OFF + par * SRC_BYTES + (size_t)k * 2048;
        char* vslot = a.me + VBOX_OFF + par * VBOX_HALF + (size_t)(k >> 1) * 1024;
        if (k & 32u) pack_unit<1>(d, slot, vslot, k & 1u, wl + P0_VAR, (u32)(size_t)(wl + P0_LUT), lane);
        else pack_unit<0>(d, slot, vslot, k & 1u, wl + P0_VAR, (u32)(size_t)(wl + P0_LUT), lane);
    }
}
__device__ __forceinline__ void publish_task(char* const (&w)[NR], u32 rank, u32 b, u32 call, int lane) {
    const u32 off = (u32)RDY_OFF + (call & 1u) * (u32)RDY_HALF + (lane == 0 ? rank * RDY_SRC + b * 256u : RDY_DENSE + rank * RDY_DENSE_SRC + b * 4u);
    asm volatile("s_mov_b64 exec, 3\n\tglobal_store_dword %0, %1, %2 sc0 sc1\n\tglobal_store_dword %0, %1, %3 sc0 sc1\n\tglobal_store_dword %0, %1, %4 sc0 sc1\n\t"
                 "global_store_dword %0, %1, %5 sc0 sc1\n\tglobal_store_dword %0, %1, %6 sc0 sc1\n\tglobal_store_dword %0, %1, %7 sc0 sc1\n\t"
                 "global_store_dword %0, %1, %8 sc0 sc1\n\tglobal_store_dword %0, %1, %9 sc0 sc1\n\ts_mov_b64 exec, -1"
                 ::"v"(off), "v"(call), "s"(w[0]), "s"(w[1]), "s"(w[2]), "s"(w[3]), "s"(w[4]), "s"(w[5]), "s"(w[6]), "s"(w[7]) : "memory");
}
__device__ __forceinline__ u32 stage_out(const Args& a, LAS char* lds, const u32 prev, const u32x4 r0, const u32x4 r1) {
    const int tid = threadIdx.x, wid = __builtin_amdgcn_readfirstlane(tid >> 6), lane = tid & 63;
    const u32 b = blockIdx.x, k = task_unit(b, (u32)wid);
    LAS char* wl = lds + wid * P0_WAVE;
    pack_tables(wl, lane);
    *(LAS u32x4*)(wl + P0_VAR + lane * 16) = (u32x4){0u, 0u, 0u, 0u};
    SCHED;
    u32 d[8];
    retile(k, r0, r1, d, wl, lane);
    const u32 call = (u32)__builtin_amdgcn_readfirstlane((int)prev) + 1u;
    stage_unit(a, wl, k, b < PUSH_TASKS, call & 1u, d, lane);
    char* w[NR];
#pragma unroll
    for (int p = 0; p < NR; ++p) w[p] = a.ws[p];
    asm volatile("" : "+s"(w[0]), "+s"(w[1]), "+s"(w[2]), "+s"(w[3]), "+s"(w[4]), "+s"(w[5]), "+s"(w[6]), "+s"(w[7]));
    asm volatile("s_waitcnt vmcnt(0)" ::: "memory");
    __syncthreads();
    if (wid == 0) publish_task(w, (u32)a.rank, b, call, lane);
    st32_agent_lane0((u32*)(a.me + DONE_OFF + ((size_t)slot_unit(k, call & 1u) << DONE_SHIFT)) + a.rank, call);
    return call;
}
__device__ __forceinline__ void stage_late(const Args& a, LAS char* wl, u32 b, u32 call, int lane) {
    pack_tables(wl, lane);
#pragma nounroll
    for (u32 i = 0; i < 8; ++i) {
        const u32 k = task_unit(b, i);
        u32x4 r0, r1; u32 d[8];
        load_unit(a, k, lane, r0, r1);
        *(LAS u32x4*)(wl + P0_VAR + lane * 16) = (u32x4){0u, 0u, 0u, 0u};
        retile(k, r0, r1, d, wl, lane);
        stage_unit(a, wl, k, false, call & 1u, d, lane);
    }
    char* w[NR];
#pragma unroll
    for (int p = 0; p < NR; ++p) w[p] = a.ws[p];
    asm volatile("s_waitcnt vmcnt(0)" ::: "memory");
    publish_task(w, (u32)a.rank, b, call, lane);
    const u32 own = ((task_unit(b, (u32)lane & 7u) << DONE_SHIFT) + (u32)a.rank * 4u);
    asm volatile("s_mov_b64 exec, 0xff\n\tglobal_store_dword %0, %1, %2 sc1\n\ts_mov_b64 exec, -1" ::"v"(own), "v"(call), "s"(a.me + DONE_OFF) : "memory");
}

constexpr int PL_WAVE = 768 + 2048 + 2560 + 64;
#define PULL_WPW 7
constexpr u32 PULL_WAVES = PULL_SLOTS * 8 * PULL_WPW;

__device__ __forceinline__ void pull(const Args& a, u32* cnt, int c, LAS char* lds) {
    const int tid = threadIdx.x, wid = __builtin_amdgcn_readfirstlane(tid >> 6), lane = tid & 63;
    const u32 gw = (u32)c * PULL_WPW + (u32)wid, rk = gw / 7u, link = PULL_WPW == 7 ? ((u32)wid + (u32)c) % 7u : gw % 7u;
    const u32 nw = (PULL_WAVES + 6u - link) / 7u;
    auto push_unit = [](u32 i) -> u32 { return (i >> 6) * 64u + ((i >> 1) & 1u) * 32u + ((i & 63u) >> 2) * 2u + (i & 1u); };
    static_assert(NPUSH <= 3 * (PULL_WAVES / 7) && NPUSH >= 2 * (PULL_WAVES / 7), "two or three pushed units per wave");
    const bool pa = wid < PULL_WPW, pc = pa && rk + 2u * nw < NPUSH;
    const u32 ka = push_unit(pa ? rk : 0u), kb = push_unit(pa ? rk + nw : 0u), kc = push_unit(pc ? rk + 2u * nw : 0u);
    u32x4 a0, a1, b0, b1, c0 = {}, c1 = {};
    load_unit(a, ka, lane, a0, a1);
    const u32 prev = __hip_atomic_load(cnt, __ATOMIC_RELAXED, __HIP_MEMORY_SCOPE_AGENT);
    load_unit(a, kb, lane, b0, b1);
    if (pc) load_unit(a, kc, lane, c0, c1);
    const u32 ord = ((const u32*)ORDER.item)[tid], ord2 = ((const u32*)ORDER.item)[512 + (tid & 127)];
    SCHED;
    const u32 s = ((u32)a.rank + 1u + link) & 7u;
    char* myws = a.me;
    char* peer = uniform_ptr(a.ws[s]);
    u32* tmo = (u32*)(myws + TMO_OFF);
    const u32 img = (u32)(size_t)(lds + 4096 + wid * PL_WAVE + 768), exp = img + 2048u, lut = exp + 2560u;
    if (lane < 16) {
        u32 dep = 0; u32 n = 0;
#pragma unroll
        for (int i = 0; i < 4; ++i) if ((lane >> i) & 1) { dep |= (3u ^ (4u + n)) << (8 * i); ++n; }
        *(LAS u32*)(size_t)(lut + lane * 4) = dep;
    }
    const u32 l16 = (u32)lane * 16u, l8 = (u32)lane * 8u;
    u32 pd[8];
    retile(ka, a0, a1, pd, (LAS char*)(size_t)exp, lane);
    const u32 call = (u32)__builtin_amdgcn_readfirstlane((int)prev) + 1u, par = call & 1u;
    const u32x4 rpush = make_rsrc(peer + STAGE_OFF + a.rank * 65536u, (u32)STAGE_BYTES - a.rank * 65536u);
    const u32x4 rpdone = make_rsrc(peer + DONE_OFF + a.rank * 4u, (NUNITS + 128u) << DONE_SHIFT);
    u32 vcall = call; asm volatile("" : "+v"(vcall));
#define PUSH(d, k) asm volatile("s_nop 4\n\tbuffer_store_dwordx4 %0, %2, %3, %4 offen sc0 sc1\n\tbuffer_store_dwordx4 %1, %2, %3, %4 offen offset:1024 sc0 sc1\n\ts_nop 1" \
                     ::"v"((u32x4){d[0], d[1], d[2], d[3]}), "v"((u32x4){d[4], d[5], d[6], d[7]}), "v"(l16), "s"(rpush), "s"(stage_off(slot_unit(k, par))) : "memory")
#define PUSHED(k) asm volatile("s_waitcnt vmcnt(0)\n\ts_mov_b64 exec, 1\n\tbuffer_store_dword %0, off, %1, %2 sc0 sc1\n\ts_mov_b64 exec, -1" \
                     ::"v"(vcall), "s"(rpdone), "s"(slot_unit(k, par) << DONE_SHIFT) : "memory")
    __builtin_amdgcn_s_waitcnt(0x0F70);
    if (pa) PUSH(pd, ka);
    retile(kb, b0, b1, pd, (LAS char*)(size_t)exp, lane);
    *(LAS u32*)(lds + tid * 4) = ord; *(LAS u32*)(lds + 2048 + (tid & 127) * 4) = ord2;
    asm volatile("s_waitcnt lgkmcnt(0)" ::: "memory");
    __builtin_amdgcn_s_barrier();
    asm volatile("" ::: "memory");
    if (wid == 0) st32_agent_lane0(cnt, call);
    if (wid >= PULL_WPW) {
        static_assert(PULL_WPW <= 7, "a spare wave per pulling workgroup");
        if (wid < 7) return;
        __builtin_amdgcn_s_sleep(LATE2_SLEEP);
        stage_late(a, lds + 4096 + 8 * PL_WAVE, GEMM_SLOTS * 8 + (u32)c, call, lane);
        GSTAMP(34);
        return;
    }
    STAMP(1);
    const char* dense = myws + RDY_OFF + par * RDY_HALF + RDY_DENSE + s * RDY_DENSE_SRC + l16;
#define GLANCE() asm volatile("global_load_dwordx4 v[252:255], %0, off sc0 sc1" :: "v"(dense) : "memory", "v252", "v253", "v254", "v255")
    PUSHED(ka);
    PUSH(pd, kb);
    u32 klast = kb;
    if (pc) { retile(kc, c0, c1, pd, (LAS char*)(size_t)exp, lane); PUSHED(kb); PUSH(pd, kc); klast = kc; }
    const u32x4 rbox = make_rsrc(peer + OUTBOX_OFF + par * SRC_BYTES, (u32)(RDY_OFF - OUTBOX_OFF - par * SRC_BYTES));
    const u32x4 rstage = make_rsrc(myws + STAGE_OFF + s * 65536u, (u32)STAGE_BYTES - s * 65536u);
    const u32x4 rdone = make_rsrc(myws + DONE_OFF + s * 4u, (NUNITS + 128u + 511u) << DONE_SHIFT);
    const u32 vdelta = (u32)(VBOX_OFF - OUTBOX_OFF) + par * (u32)VBOX_HALF - par * (u32)SRC_BYTES;
    const char* rdy = myws + RDY_OFF + par * RDY_HALF;
    u32* claim = (u32*)(myws + CLAIM_OFF) + link * 32u;
    const u32 tbase = nw - (call - 1u) * NTICKETS;
    u32 vlut = lut; asm volatile("" : "+v"(vlut));
    u32 known = 0;
    auto lookup = [&](u32 ticket) -> u32 { return (u32)__builtin_amdgcn_readfirstlane(*(LAS const unsigned short*)(lds + ticket * 2u)); };
    auto await = [&](u32 ticket, u32 item, bool refresh) -> bool {
        const u32 b = unit_task(item & 2047u);
        {
            if (refresh) {
                u32x4 f;
                asm volatile("global_load_dwordx4 %0, %1, off sc0 sc1\n\ts_waitcnt vmcnt(0)" : "=&v"(f) : "v"(rdy + RDY_DENSE + s * RDY_DENSE_SRC + l16) : "memory");
                const u64 bad = __ballot(f[0] != call || f[1] != call || f[2] != call || f[3] != call);
                known = bad ? (u32)__builtin_ctzll(bad) * 4u : 256u;
                if (b < known) return true;
            }
            const u64 t0 = __builtin_amdgcn_s_memrealtime();
            for (u32 spins = 0;;) {
                u32 f;
                asm volatile("s_mov_b64 exec, 1\n\tglobal_load_dword %0, %1, off sc0 sc1\n\ts_mov_b64 exec, -1\n\ts_waitcnt vmcnt(0)" : "=&v"(f) : "v"(rdy + s * RDY_SRC + b * 256u) : "memory");
                const u32 seq = (u32)__builtin_amdgcn_readfirstlane((int)f);
                if (seq == call) break;
                bool out = (int)(seq - call) > 0;
                if (out) st32_agent_lane0(tmo, call); else out = give_up(tmo, call, t0, spins);
                if (out) {
                    while (ticket < NTICKETS) {
                        u32 r;
                        asm volatile("s_mov_b64 exec, 1\n\tglobal_atomic_add %0, %1, %2, off sc0\n\ts_mov_b64 exec, -1\n\ts_waitcnt vmcnt(0)" : "=&v"(r) : "v"(claim), "v"(1u) : "memory");
                        ticket = (u32)__builtin_amdgcn_readfirstlane((int)r) + tbase;
                    }
                    return false;
                }
                __builtin_amdgcn_s_sleep(POLL_SLEEP);
            }
        }
        return true;
    };
#define ZONE "v240", "v241", "v242", "v243", "v244", "v245", "v246", "v247", "v248", "v249", "v250", "v252", "v253", "v254", "v255"
#define ISSUE(it) do { const u32 k_ = (it) & 2047u; \
        asm volatile("buffer_load_dwordx4 v[240:243], %0, %2, %3 offen sc0 sc1\n\ts_mov_b64 exec, %5\n\tbuffer_load_dwordx2 v[248:249], %1, %2, %3 offen offset:1024 sc0 sc1\n\t" \
                     "s_mov_b64 exec, 0xffffff\n\tbuffer_load_dwordx4 v[244:247], %0, %2, %4 offen offset:256 sc0 sc1\n\t" \
                     "s_mov_b64 exec, 1\n\tglobal_atomic_add v250, %6, %7, off sc0\n\ts_mov_b64 exec, -1" \
                     :: "v"(l16), "v"(l8), "s"(rbox), "s"(k_ << 11), "s"(vdelta + ((k_ >> 1) << 10)), \
                        "s"((k_ & 32u) ? 0xffffffffull : ~0ull), "v"(claim), "v"(1u) : "memory", ZONE); } while (0)
#define TAKE(n) asm volatile("s_waitcnt vmcnt(" #n ")\n\tv_mov_b64 %0, v[240:241]\n\tv_mov_b64 %1, v[242:243]\n\tv_mov_b64 %2, v[248:249]" : "=v"(A0), "=v"(A1), "=v"(B) :: "memory", ZONE)
    auto FLAG = [=](u32 k) { asm volatile("s_mov_b64 exec, 1\n\tbuffer_store_dword %0, off, %1, %2 sc1\n\ts_mov_b64 exec, -1" ::"v"(vcall), "s"(rdone), "s"(k << DONE_SHIFT) : "memory"); };
#define STORE2(d, so) asm volatile("buffer_store_dwordx4 %0, %2, %3, %4 offen sc1\n\tbuffer_store_dwordx4 %1, %2, %3, %4 offen offset:1024 sc1\n\ts_nop 1" \
                     ::"v"((u32x4){d[0], d[1], d[2], d[3]}), "v"((u32x4){d[4], d[5], d[6], d[7]}), "v"(l16), "s"(rstage), "s"(so) : "memory")
    static_assert(VMID - VWIN == 256 && 2 * VWIN == 24 * 16, "the asm above and below has these built in");
    u32 it = lookup(rk);
    u32x2 A0, A1, B;
    const u32 none = NUNITS + 128u + gw;
    u32 prv = none;
#define PST() do { } while (0)
    auto noted = [&]() {
        u32 t; u64 bad;
        asm volatile("v_xor_b32 v252, %2, v252\n\tv_xor_b32 v253, %2, v253\n\tv_xor_b32 v254, %2, v254\n\tv_xor_b32 v255, %2, v255\n\t"
                     "v_or3_b32 %0, v252, v253, v254\n\tv_or_b32 %0, %0, v255\n\tv_cmp_ne_u32 %1, 0, %0\n\ts_nop 1" : "=&v"(t), "=s"(bad) : "s"(call) : "memory", ZONE);
        known = bad ? (u32)__builtin_ctzll(bad) * 4u : 256u;
    };
    asm volatile("s_waitcnt vmcnt(0)\n\tglobal_load_dwordx4 v[252:255], %3, off sc0 sc1\n\ts_mov_b64 exec, 1\n\tbuffer_store_dword %0, off, %1, %2 sc0 sc1\n\t"
                 "s_mov_b64 exec, -1\n\ts_waitcnt vmcnt(1)" ::"v"(vcall), "s"(rpdone), "s"(slot_unit(klast, par) << DONE_SHIFT), "v"(dense) : "memory", ZONE);
    noted();
    if (unit_task(it & 2047u) >= known && !await(rk, it, false)) return;
    if (known < 256u) GLANCE();
    ISSUE(it);
    TAKE(0);
    if (known < 256u) noted();
    STAMP(2);
    for (;;) {
        const u32 k = it & 2047u, ks = slot_unit(k, par);
        u32 ticket = 0, hdr0, hdr1, nit = ~0u, d[8];
        asm volatile("v_readlane_b32 %0, v247, 11\n\tv_readlane_b32 %1, v244, 12" : "=&s"(hdr0), "=&s"(hdr1) :: "memory", ZONE);
        PST();
        bool glanced = false;
        bool ahead = false;
        auto advance = [&]() {
            asm volatile("s_waitcnt vmcnt(3)\n\tv_readfirstlane_b32 %0, v250" : "=s"(ticket) :: "memory", ZONE);
            ticket += tbase;
            if (ticket < NTICKETS) {
                nit = lookup(ticket);
                const u32 b = unit_task(nit & 2047u);
                if (__builtin_expect(b < known, 1)) {
                    if (__builtin_expect(b + 48u >= known && known < 256u, 0)) { GLANCE(); glanced = true; }
                    ISSUE(nit); ahead = true;
                }
            }
        };
        auto publish = [&](u32 u) {
            if (__builtin_expect(ahead, 1)) asm volatile("s_waitcnt vmcnt(4)" ::: "memory"); else asm volatile("s_waitcnt vmcnt(0)" ::: "memory");
            FLAG(u);
        };
        u32 last = ks;
        {
            const bool solo = (it & I_SOLO) != 0u;
            const u32 hdr = (k & 1u) ? hdr1 : hdr0, hdrs = solo ? hdr : hdr0 | hdr1;
            if (__builtin_expect((hdrs & H_RAW) != 0u, 0)) {
                last = solo ? ks : ks + 1u;
                asm volatile("s_nop 4\n\tbuffer_load_dwordx4 v[244:247], %0, %1, %2 offen sc0 sc1\n\ts_waitcnt vmcnt(0)\n\tds_write_b128 %3, v[244:247]\n\ts_waitcnt lgkmcnt(0)"
                             :: "v"(l16), "s"(rbox), "s"(vdelta + ((k >> 1) << 10)), "v"(img + l16) : "memory", ZONE);
                FLAG(prv);
#pragma nounroll
                for (u32 u = k;; ++u) {
                    u32x4 R0, R1; u32x2 C;
                    asm volatile("s_nop 4\n\tbuffer_load_dwordx4 %0, %3, %5, %6 offen sc0 sc1\n\tbuffer_load_dwordx4 %1, %3, %5, %6 offen offset:1024 sc0 sc1\n\t"
                                 "buffer_load_dwordx2 %2, %4, %5, %6 offen offset:1024 sc0 sc1\n\ts_waitcnt vmcnt(0)"
                                 : "=&v"(R0), "=&v"(R1), "=&v"(C) : "v"(l16), "v"(l8), "s"(rbox), "s"(u << 11) : "memory");
                    const u32 hu = (u & 1u) ? hdr1 : hdr0;
                    if (hu & H_RAW) {
#pragma unroll
                        for (int j = 0; j < 4; ++j) { d[j] = R0[j]; d[4 + j] = R1[j]; }
                    } else if (u & 32u) unpack_unit<1>(R0, C, hu, (u & 1u) != 0u, d, img + VMID, exp, vlut, lane, []() {});
                    else unpack_unit<0>(R0, C, hu, (u & 1u) != 0u, d, img + VMID, exp, vlut, lane, []() {});
                    if (solo || u != k) break;
                    STORE2(d, stage_off(ks));
                    asm volatile("s_waitcnt vmcnt(0)" ::: "memory");
                    FLAG(ks);
                }
                advance();
                FLAG(none);
            } else {
                u32 vat = img + 256u + l16;
                if (__builtin_expect((hdrs & H_BIG) != 0u, 0)) {
                    asm volatile("s_nop 4\n\tbuffer_load_dwordx4 v[244:247], %0, %1, %2 offen sc0 sc1\n\ts_waitcnt vmcnt(0)" :: "v"(l16), "s"(rbox), "s"(vdelta + ((k >> 1) << 10)) : "memory", ZONE);
                    vat = img + l16;
                }
                asm volatile("ds_write_b128 %0, v[244:247]" :: "v"(vat) : "memory", ZONE);
                auto body = [&](auto oc) {
                    constexpr int O = decltype(oc)::value;
                    if (__builtin_expect(!solo, 1)) {
                        asm volatile("buffer_load_dwordx4 v[240:243], %0, %2, %3 offen offset:2048 sc0 sc1\n\ts_mov_b64 exec, %4\n\t"
                                     "buffer_load_dwordx2 v[248:249], %1, %2, %3 offen offset:3072 sc0 sc1\n\ts_mov_b64 exec, -1"
                                     :: "v"(l16), "v"(l8), "s"(rbox), "s"(k << 11), "s"(O ? 0xffffffffull : ~0ull) : "memory", ZONE);
                        unpack_unit<O>((u32x4){A0[0], A0[1], A1[0], A1[1]}, B, hdr0, false, d, img + VMID, exp, vlut, lane,
                                       [&]() { asm volatile("s_waitcnt vmcnt(2)" ::: "memory"); FLAG(prv); });
                        WSTAMP(k >> 6); PST();
                        STORE2(d, stage_off(ks));
                        TAKE(3);
                        PST();
                        prv = ks; last = ks + 1u;
                    }
                    advance();
                    PST();
                    unpack_unit<O>((u32x4){A0[0], A0[1], A1[0], A1[1]}, B, (last & 1u) ? hdr1 : hdr0, (last & 1u) != 0u, d, img + VMID, exp, vlut, lane, [&]() { publish(prv); });
                };
                if (k & 32u) body(std::integral_constant<int, 1>{}); else body(std::integral_constant<int, 0>{});
            }
        }
        PST();
        STORE2(d, stage_off(last));
        if (__builtin_expect(ahead, 1)) {
            TAKE(4);
            if (__builtin_expect(glanced, 0)) noted();
            prv = last;
        } else {
            asm volatile("s_waitcnt vmcnt(0)" ::: "memory");
            FLAG(last);
            prv = none;
            if (nit == ~0u || !await(ticket, nit, false)) break;
            GLANCE();
            ISSUE(nit);
            TAKE(0);
            noted();
        }
        it = nit;
    }
#undef ISSUE
#undef PUSH
#undef PUSHED
#undef TAKE
#undef GLANCE
#undef PST
#undef ZONE
#undef STORE2
    GSTAMP(32);
    STAMP(3);
}

__device__ __forceinline__ void gemm(const Args& a, u32 call, int xcd, int slot, LAS char* lds) {
    const int tid = threadIdx.x, wid = __builtin_amdgcn_readfirstlane(tid >> 6), lane = tid & 63;
    const int li = (xcd & 1) * GEMM_SLOTS + slot, ti = (xcd >> 1) * 4 + (li & 3), tj = li >> 2;
    const int row0 = ti * BM, col0 = tj * BN > DIM - BN ? DIM - BN : tj * BN;
    const int wr = wid >> 2, wc = wid & 3, fr = lane & 15, fq = lane >> 4;
    char* myws = a.me;
    u32* tmo = (u32*)(myws + TMO_OFF);
    const __amdgpu_buffer_rsrc_t rf = __builtin_amdgcn_make_buffer_rsrc(myws + DONE_OFF, 0, (int)(DBG_OFF - DONE_OFF), 0x00020000);
    const __amdgpu_buffer_rsrc_t rs = __builtin_amdgcn_make_buffer_rsrc(myws + STAGE_OFF, 0, (int)STAGE_BYTES, 0x00020000);
    const int vx = (row0 + lane) * 16, vw = 8 * 65536 + (col0 + lane) * 16;
    const int xoff = 2 * fq * XS + (wr * 128 + fr) * 16, woff = XB + 2 * fq * WS + (wc * 64 + fr) * 16, woff4 = woff + (256 - wc * 48) * 16;
    const int alt = (int)(call & 1u) * NT;
    auto piece_of = [&](int v) -> int {
        int t, r = a.rank;
        if (v < OWN_FIRST) t = v * 8 + wid;
        else if (v == OWN_LAST) t = OWN_FIRST * 8 + wid;
        else {
            const u32 g = (u32)(v - OWN_FIRST - (v > OWN_LAST ? 1 : 0)) * 8u + (u32)wid, q = (g * 9363u) >> 16;
            t = (int)q; r = (int)(((u32)a.rank + 1u + g + q) & 7u);
        }
        return ((t + (t < 2 ? alt : 0)) << 4) | r;
    };
    const int fv = (lane < 2 ? ti * 2 + lane : lane < 5 ? 30 + (col0 >> 7) + lane : ti * 2) << DONE_SHIFT;
    LAS u32* snap = (LAS u32*)(lds + 2 * STG + wid * 256);
    LAS u32* bad = (LAS u32*)(lds + 2 * STG + 2048);
    bool dead = false;
    auto wait_piece = [&](int pc) {
        if (dead) return;
        const char* fl = myws + DONE_OFF + fv + ((pc >> 4) << (DONE_SHIFT + 6)) + (pc & 7) * 4;
        const u64 t0 = __builtin_amdgcn_s_memrealtime();
        for (u32 spins = 0;;) {
            u32 seen;
            asm volatile("s_mov_b64 exec, 31\n\tglobal_load_dword %0, %1, off sc0 sc1\n\ts_mov_b64 exec, -1\n\ts_waitcnt vmcnt(0)" : "=&v"(seen) : "v"(fl) : "memory");
            if ((__ballot(seen == call) & 31ull) == 31ull) break;
            if (give_up(tmo, call, t0, spins)) { dead = true; *(LAS volatile u32*)bad = call; break; }
            __builtin_amdgcn_s_sleep(GEMM_SLEEP);
        }
    };
#define DMA_X(pc, b, i) __builtin_amdgcn_raw_ptr_buffer_load_lds(rs, (LAS void*)(lds + (b) * STG + wid * XS), 16, vx, (pc) << 16, (i) * 1024, 17)
#define DMA_W(pc, b, i) __builtin_amdgcn_raw_ptr_buffer_load_lds(rs, (LAS void*)(lds + (b) * STG + XB + wid * WS + ((i) == 4 ? 4096 : 0)), 16, vw, ((pc) << 16) + ((i) == 4 ? 4096 : 0), ((i) & 3) * 1024, 17)
    v4f acc[8][5];
#pragma unroll
    for (int m = 0; m < 8; ++m)
#pragma unroll
        for (int n = 0; n < 5; ++n) acc[m][n] = (v4f){0.f, 0.f, 0.f, 0.f};
    const float scale = __builtin_bit_cast(float, __builtin_amdgcn_readfirstlane(__builtin_bit_cast(int, a.sx[0] * a.sw[0])));
    __builtin_amdgcn_s_waitcnt(0x0F70);
    int rscale = __builtin_amdgcn_readfirstlane(__float_as_int(__builtin_amdgcn_rcpf(scale)));
    u32x4 ro = make_rsrc(a.out + (size_t)(row0 + wr * 128) * DIM + col0, (u32)(128 * DIM - col0) * 4u);
    asm volatile("" : "+s"(rscale), "+s"(ro));
    if (tid == 0) *bad = 0u;
    asm volatile("s_waitcnt lgkmcnt(0)" ::: "memory");
    __builtin_amdgcn_s_barrier(); asm volatile("" ::: "memory");
    {
        const int pc = piece_of(0);
        wait_piece(pc);
        STAMP(2);
        DMA_W(pc, 0, 0); DMA_W(pc, 0, 1); DMA_W(pc, 0, 2); DMA_W(pc, 0, 3); DMA_W(pc, 0, 4); DMA_X(pc, 0, 0); DMA_X(pc, 0, 1); DMA_X(pc, 0, 2); DMA_X(pc, 0, 3);
    }
    asm volatile("s_waitcnt vmcnt(0) lgkmcnt(0)" ::: "memory");
    __builtin_amdgcn_s_barrier(); asm volatile("" ::: "memory");
    v8i wf[5], xa, xb, xc;
    xc = ldfrag(lds, xoff, XS); SCHED;
#pragma unroll
    for (int n = 0; n < 5; ++n) { wf[n] = ldfrag(lds, n < 4 ? woff + n * 256 : woff4, WS); SCHED; }
#define MF(m, n, xf) acc[m][n] = __builtin_amdgcn_mfma_scale_f32_16x16x128_f8f6f4(xf, wf[n], acc[m][n], 0, 1, 0, 0, 0, 0)
#define MM(m, xf) do { _Pragma("unroll") for (int n = 0; n < 5; ++n) MF(m, n, xf); } while (0)
#define LF(m, n, xf) asm volatile("v_mfma_f32_16x16x128_f8f6f4 %0, %1, %2, %0 blgp:1" : "+v"(acc[m][n]) : "v"(xf), "v"(wf[n]))
#define LM(m, xf) do { _Pragma("unroll") for (int n = 0; n < 5; ++n) LF(m, n, xf); } while (0)
#define PRIO(p) __builtin_amdgcn_s_setprio(p)
    u32 seen = 0;
    int u = piece_of(1);
    for (int t = 0; t < NT - 1; ++t) {
        const int b = t & 1;
        LAS const char* cur = lds + b * STG; LAS const char* nxt = lds + (b ^ 1) * STG;
        if (wid == 0) GSTAMP(t);
        if ((__ballot(seen == call) & 31ull) != 31ull) { wait_piece(u); __builtin_amdgcn_s_waitcnt(0xC07F); }
        if (wid == 0) GSTAMP(32 + t);
        xb = ldfrag(cur, xoff + 1 * 256, XS); SCHED;
        LF(0, 0, xc); SCHED; DMA_W(u, b ^ 1, 0); SCHED; LF(0, 1, xc); SCHED; DMA_W(u, b ^ 1, 1); SCHED; LF(0, 2, xc); SCHED; DMA_W(u, b ^ 1, 2); SCHED;
        LF(0, 3, xc); SCHED; DMA_W(u, b ^ 1, 3); SCHED; LF(0, 4, xc); SCHED; DMA_W(u, b ^ 1, 4); SCHED;
        PRIO(2);
        xa = ldfrag(cur, xoff + 2 * 256, XS); SCHED;
        LF(1, 0, xb); SCHED; DMA_X(u, b ^ 1, 0); SCHED; LF(1, 1, xb); SCHED; DMA_X(u, b ^ 1, 1); SCHED;
        LF(1, 2, xb); SCHED; DMA_X(u, b ^ 1, 2); SCHED; LF(1, 3, xb); SCHED; DMA_X(u, b ^ 1, 3); SCHED; LF(1, 4, xb); SCHED;
        xb = ldfrag(cur, xoff + 3 * 256, XS);
        {
            u = piece_of((t + 2) & 31);
            __builtin_amdgcn_raw_ptr_buffer_load_lds(rf, (LAS void*)snap, 4, fv, ((u >> 4) << (DONE_SHIFT + 6)) + (u & 7) * 4, 0, 17);
        }
        SCHED; LM(2, xa); SCHED;
        PRIO(1);
        xa = ldfrag(cur, xoff + 4 * 256, XS); SCHED; LM(3, xb); SCHED;
        xc = ldfrag(cur, xoff + 5 * 256, XS); SCHED; LM(4, xa); SCHED;
        PRIO(0);
        xa = ldfrag(cur, xoff + 6 * 256, XS); xb = ldfrag(cur, xoff + 7 * 256, XS); SCHED; LM(5, xc); SCHED;
        asm volatile("s_waitcnt vmcnt(0) lgkmcnt(0)" ::: "memory");
        __builtin_amdgcn_s_barrier(); asm volatile("" ::: "memory"); SCHED;
        PRIO(3);
#pragma unroll
        for (int n = 0; n < 5; ++n) {
            LF(6, n, xa); SCHED;
            if (n == 0) { seen = *(LAS const u32*)(snap + lane); xc = ldfrag(nxt, xoff, XS); }
            SCHED; LF(7, n, xb); SCHED; wf[n] = ldfrag(nxt, n < 4 ? woff + n * 256 : woff4, WS); SCHED;
        }
    }
    STAMP(3);
    const float inv = *(LAS const volatile u32*)bad == call ? __builtin_nanf("") : __int_as_float(rscale);
    const float nsl = scale * -1.44269504089f;
    const int v4 = (tj == 12 && wc == 0) ? 0x40000000 : (fq * 4 * DIM + wc * 64 + fr * 4) * 4, v1 = (fq * 4 * DIM + 256 + wc * 16 + fr) * 4;
    auto silu2 = [&](float a0, float a1, float& r0, float& r1) {
        const v2f u = (v2f){a0, a1} * nsl;
        v2f e = {__builtin_amdgcn_exp2f(__builtin_amdgcn_fmed3f(u[0], -86.5617024533f, 86.5617024533f)),
                 __builtin_amdgcn_exp2f(__builtin_amdgcn_fmed3f(u[1], -86.5617024533f, 86.5617024533f))};
        e = __builtin_elementwise_fma(e, (v2f){inv, inv}, (v2f){inv, inv});
        r0 = __builtin_amdgcn_rcpf(e[0]) * a0; r1 = __builtin_amdgcn_rcpf(e[1]) * a1;
        asm("" : "+v"(r0)); asm("" : "+v"(r1));
    };
#define EPI(m) do { _Pragma("unroll") for (int j = 0; j < 4; j += 2) { \
        float r0[5], r1[5]; \
        _Pragma("unroll") for (int n = 0; n < 5; ++n) silu2(acc[m][n][j], acc[m][n][j + 1], r0[n], r1[n]); \
        asm volatile("buffer_store_dwordx4 %0, %4, %6, %7 offen " OUT_FLAGS "\n\tbuffer_store_dwordx4 %1, %4, %6, %8 offen " OUT_FLAGS "\n\t" \
                     "buffer_store_dword %2, %5, %6, %7 offen " OUT_FLAGS "\n\tbuffer_store_dword %3, %5, %6, %8 offen " OUT_FLAGS \
                     :: "v"((v4f){r0[0], r0[1], r0[2], r0[3]}), "v"((v4f){r1[0], r1[1], r1[2], r1[3]}), "v"(r0[4]), "v"(r1[4]), "v"(v4), "v"(v1), \
                        "s"(ro), "s"(((m) * 16 + j) * DIM * 4), "s"(((m) * 16 + j + 1) * DIM * 4) : "memory"); \
    } } while (0)
    {
        LAS const char* cur = lds + ((NT - 1) & 1) * STG;
        xb = ldfrag(cur, xoff + 1 * 256, XS); SCHED; MM(0, xc); SCHED;
        xa = ldfrag(cur, xoff + 2 * 256, XS); SCHED; MM(1, xb); SCHED; EPI(0); SCHED; PRIO(2);
        xb = ldfrag(cur, xoff + 3 * 256, XS); SCHED; MM(2, xa); SCHED; EPI(1); SCHED;
        xa = ldfrag(cur, xoff + 4 * 256, XS); SCHED; MM(3, xb); SCHED; EPI(2); SCHED; PRIO(1);
        xb = ldfrag(cur, xoff + 5 * 256, XS); SCHED; MM(4, xa); SCHED; EPI(3); SCHED;
        xa = ldfrag(cur, xoff + 6 * 256, XS); SCHED; MM(5, xb); SCHED; EPI(4); SCHED; PRIO(0);
        xb = ldfrag(cur, xoff + 7 * 256, XS); SCHED; MM(6, xa); SCHED; EPI(5); SCHED;
        MM(7, xb); SCHED; EPI(6); SCHED; EPI(7);
    }
    STAMP(4);
}
}

__global__ void __launch_bounds__(512, 2) fused_allgather_gemm(Args a) {
    extern __shared__ __attribute__((aligned(16))) char lds_dyn[];
    LAS char* lds = (LAS char*)lds_dyn;
    asm volatile("" :: "s"(a.x), "s"(a.w), "s"(a.me), "s"(a.rank));
    STAMP(0);
    const int xcd = blockIdx.x & 7, slot = blockIdx.x >> 3;
    u32* cnt = (u32*)(a.me + CNT_OFF) + blockIdx.x;
    if (__builtin_expect(slot < GEMM_SLOTS, 1)) {
        const u32 k0 = task_unit(blockIdx.x, (u32)__builtin_amdgcn_readfirstlane(threadIdx.x >> 6));
        u32x4 r0, r1;
        if (blockIdx.x >= LATE_TASKS) __builtin_amdgcn_s_sleep(LATE_SLEEP);
        load_unit(a, k0, threadIdx.x & 63, r0, r1);
        const u32 prev = __hip_atomic_load(cnt, __ATOMIC_RELAXED, __HIP_MEMORY_SCOPE_AGENT);
        SCHED;
        const u32 call = stage_out(a, lds, prev, r0, r1);
        if (__builtin_amdgcn_readfirstlane(threadIdx.x) == 0) st32_agent_lane0(cnt, call);
        STAMP(1);
        gemm(a, call, xcd, slot, lds);
    } else {
        pull(a, cnt, (slot - GEMM_SLOTS) * 8 + xcd, lds);
    }
}

extern "C" void kernel_launch(int rank, int nranks, ncclComm_t comm, hipStream_t stream,
                              void* const* d_in, const int* in_sizes, int n_in,
                              void* d_out, int out_size, const int* in_dtypes, int out_dtype,
                              void* const* d_out_all, void* const* d_ws_all, size_t ws_bytes) {
    if (nranks != NR || (unsigned)rank >= (unsigned)NR || n_in != 4 || ws_bytes < WS_NEED || out_size != DIM * DIM ||
        in_sizes[0] != DIM * KS || in_sizes[1] != DIM * KS) {
        fprintf(stderr, "kernel_launch: unexpected problem shape\n");
        return;
    }
    static bool attr_set[NR] = {};
    if (!attr_set[rank]) {
        HIP_CHECK(hipFuncSetAttribute((const void*)fused_allgather_gemm, hipFuncAttributeMaxDynamicSharedMemorySize, LDS_BYTES));
        attr_set[rank] = true;
    }
    Args a;
    a.x = (const unsigned char*)d_in[0]; a.w = (const unsigned char*)d_in[1];
    a.sx = (const float*)d_in[2]; a.sw = (const float*)d_in[3];
    a.out = (float*)d_out;
    for (int p = 0; p < NR; ++p) a.ws[p] = (char*)d_ws_all[p];
    a.me = a.ws[rank]; a.rank = rank; a.pad = 0;
    fused_allgather_gemm<<<256, 512, LDS_BYTES, stream>>>(a);
    HIP_CHECK(hipGetLastError());
}
```

```cpp
#include <hip/hip_runtime.h>
#include <rccl/rccl.h>
#include <cstdio>
#include <type_traits>

#define HIP_CHECK(expr) do { hipError_t _e = (expr); if (_e != hipSuccess) { \
    fprintf(stderr, "HIP error %s at %s:%d\n", hipGetErrorString(_e), __FILE__, __LINE__); \
    return; } } while (0)

#define LAS __attribute__((address_space(3)))
#define GAS __attribute__((address_space(1)))
typedef unsigned u32;
typedef unsigned long long u64;
typedef int v2i __attribute__((ext_vector_type(2)));
typedef int v4i __attribute__((ext_vector_type(4)));
typedef int v8i __attribute__((ext_vector_type(8)));
typedef float v2f __attribute__((ext_vector_type(2)));
typedef float v4f __attribute__((ext_vector_type(4)));
typedef unsigned u32x2 __attribute__((ext_vector_type(2)));
typedef unsigned u32x4 __attribute__((ext_vector_type(4)));

namespace {
constexpr int NR = 8;
constexpr int DIM = 4096;
constexpr int KS = DIM / NR;
constexpr int NT = 32;
constexpr u32 NUNITS = NT * 64, NBLOCKS = NUNITS / 2;
constexpr size_t STAGE_OFF = 0;
constexpr size_t STAGE_BYTES = (size_t)(NT + 2) << 20;
constexpr size_t SRC_BYTES = (size_t)NUNITS * 2048;
constexpr size_t OUTBOX_OFF = STAGE_OFF + STAGE_BYTES;
constexpr size_t VBOX_OFF = OUTBOX_OFF + 2 * SRC_BYTES;
constexpr size_t VBOX_HALF = (size_t)NBLOCKS * 1024;
constexpr size_t RDY_OFF = VBOX_OFF + 2 * VBOX_HALF + 4096;
constexpr u32 RDY_SRC = 65536 + 2304, RDY_DENSE = NR * RDY_SRC, RDY_DENSE_SRC = 1280;
constexpr size_t RDY_HALF = RDY_DENSE + NR * RDY_DENSE_SRC;
#define DONE_SHIFT 7
constexpr size_t DONE_OFF = RDY_OFF + 2 * RDY_HALF;
constexpr size_t CLAIM_OFF = DONE_OFF + ((size_t)(NUNITS + 128 + 512) << DONE_SHIFT);
constexpr size_t CNT_OFF = CLAIM_OFF + NR * 128;
constexpr size_t TMO_OFF = CNT_OFF + 256 * 4;
constexpr size_t DBG_OFF = TMO_OFF + 128;
constexpr size_t WS_NEED = DBG_OFF + 256 * 64 + 256 * 8 * 80 * 8;
#define OUT_FLAGS "sc0 sc1"
#define LATE_TASKS 32
#define SHARE 0
#define OWN_FIRST 3
#define OWN_LAST 18
#define GEMM_SLEEP 4
#define LATE2_SLEEP 20
#define LATE_SLEEP 32
#define POLL_SLEEP 4
constexpr u32 VMID = 448, VWIN = 192;
constexpr u32 H_RAW = 0x80000000u, H_BIG = 0x40000000u;
constexpr u64 DEADLINE = 150000000ull;
constexpr int BM = 256, BN = 320, XS = BM * 16, WS = BN * 16, XB = 8 * XS, WB = 8 * WS, STG = XB + WB;
constexpr int LDS_BYTES = 2 * STG + 2048 + 64;
constexpr int GEMM_SLOTS = 26, PULL_SLOTS = 6;

#define STAG_A 0
#define PUSH_TASKS 12
#define SPLIT_HEAD 0
#define SPLIT_TAIL 0
constexpr u32 NITEMS = NBLOCKS + 256;
constexpr u32 I_SOLO = 0x2000u;
constexpr u32 NPUSH = PUSH_TASKS * 8;
static_assert(NPUSH <= 128, "pushed units stay inside K-tiles 0 and 1");
struct Order { unsigned short item[NITEMS]; u32 n; };
constexpr int lag16(int p) { return STAG_A * 16 * p / 15; }
constexpr Order make_order() {
    Order r = {};
    int n = 0;
    for (int key = 0; n < (int)NBLOCKS; ++key)
        for (int p = 0; p < 16; ++p) {
            const int d = key - lag16(p);
            if (d >= 0 && d % 16 == 0 && d / 16 < NT)
                for (int o = 0; o < 2; ++o, ++n) {
                    const u32 k = ((d / 16) * 32 + o * 16 + p) * 2;
                    if ((d / 16) * 8 + (p >> 1) < PUSH_TASKS) continue;
                    if (n < SPLIT_HEAD || n >= (int)NBLOCKS - SPLIT_TAIL) { r.item[r.n++] = (unsigned short)(k | I_SOLO); r.item[r.n++] = (unsigned short)((k + 1) | I_SOLO); }
                    else r.item[r.n++] = (unsigned short)k;
                }
        }
    return r;
}
__device__ constexpr Order ORDER = make_order();
constexpr u32 NTICKETS = make_order().n;
static_assert(NTICKETS <= NITEMS, "order table too small");

struct Args {
    const unsigned char* x; const unsigned char* w; const float* sx; const float* sw; float* out;
    char* me; char* ws[NR]; int rank; int pad;
};

__device__ __forceinline__ void st16_sys(void* p, u32x4 v) {
    asm volatile("global_store_dwordx4 %0, %1, off sc0 sc1\n\ts_nop 1" ::"v"(p), "v"(v) : "memory");
}
__device__ __forceinline__ void st16_agent(void* p, u32x4 v) {
    asm volatile("global_store_dwordx4 %0, %1, off sc1\n\ts_nop 1" ::"v"(p), "v"(v) : "memory");
}
__device__ __forceinline__ u32 ld32_agent(const void* p) {
    u32 v;
    asm volatile("global_load_dword %0, %1, off sc1\n\ts_waitcnt vmcnt(0)" : "=&v"(v) : "v"(p) : "memory");
    return v;
}
__device__ __forceinline__ void st32_agent_lane0(void* p, u32 v) {
    asm volatile("s_mov_b64 exec, 1\n\tglobal_store_dword %0, %1, off sc1\n\ts_mov_b64 exec, -1" ::"v"(p), "v"(v) : "memory");
}
__device__ __forceinline__ bool give_up(u32* tmo, u32 call, u64 t0, u32& spins) {
    if ((++spins & 15u) != 0u) return false;
    if ((u32)__builtin_amdgcn_readfirstlane((int)ld32_agent(tmo)) == call || __builtin_amdgcn_s_memrealtime() - t0 > DEADLINE) {
        st32_agent_lane0(tmo, call);
        return true;
    }
    return false;
}
template <class T> __device__ __forceinline__ T* uniform_ptr(T* p) {
    const u64 v = (u64)p;
    return (T*)(((u64)(u32)__builtin_amdgcn_readfirstlane((int)(u32)(v >> 32)) << 32) | (u32)__builtin_amdgcn_readfirstlane((int)(u32)v));
}
__device__ __forceinline__ v8i ldfrag(LAS const char* p, int off, int stride) {
    v4i a = *(LAS const v4i*)(p + off), b = *(LAS const v4i*)(p + off + stride);
    return __builtin_shufflevector(a, b, 0, 1, 2, 3, 4, 5, 6, 7);
}
#define SCHED __builtin_amdgcn_sched_barrier(0)
#define STAMP(i) do { } while (0)
#define GSTAMP(i) do { } while (0)
#define WSTAMP(i) do { } while (0)

__device__ __forceinline__ u32 wave_incl_scan(u32 x) {
    x += (u32)__builtin_amdgcn_update_dpp(0, (int)x, 0x111, 0xf, 0xf, false);
    x += (u32)__builtin_amdgcn_update_dpp(0, (int)x, 0x112, 0xf, 0xf, false);
    x += (u32)__builtin_amdgcn_update_dpp(0, (int)x, 0x114, 0xf, 0xf, false);
    x += (u32)__builtin_amdgcn_update_dpp(0, (int)x, 0x118, 0xf, 0xf, false);
    x += (u32)__builtin_amdgcn_update_dpp(0, (int)x, 0x142, 0xa, 0xf, false);
    x += (u32)__builtin_amdgcn_update_dpp(0, (int)x, 0x143, 0xc, 0xf, false);
    return x;
}
__device__ __forceinline__ u32 rotr(u32 x, u32 n) { return __builtin_amdgcn_alignbit(x, x, n); }
__device__ __forceinline__ u32 bfi(u32 m, u32 a, u32 b) { return __builtin_amdgcn_bitop3_b32(m, a, b, 0xca); }
__device__ __forceinline__ u32 and3(u32 a, u32 b, u32 c) { return __builtin_amdgcn_bitop3_b32(a, b, c, 0x80); }
__device__ __forceinline__ u32 lds32(u32 addr) { return *(LAS const u32*)(size_t)addr; }
__device__ __forceinline__ u32x4 make_rsrc(const void* p, u32 bytes) {
    const u64 v = (u64)p;
    return (u32x4){(u32)__builtin_amdgcn_readfirstlane((int)(u32)v), (u32)__builtin_amdgcn_readfirstlane((int)(u32)(v >> 32)) & 0xffffu, bytes, 0x00020000u};
}

constexpr int P0_VAR = 2560, P0_LUT = P0_VAR + 1536, P0_WAVE = P0_LUT + 128;

template <int O>
__device__ __forceinline__ void pack_unit(const u32 (&d)[8], char* slot, char* vslot, u32 h, LAS char* img, u32 lut, int lane) {
    constexpr int SH = O ? 2 : 3;
    constexpr u32 EM = O ? 0x1f1f1f1fu : 0x0f0f0f0fu;
    u32 ex[8], sum = 0;
#pragma unroll
    for (int j = 0; j < 8; ++j) ex[j] = (d[j] >> SH) & EM;
#pragma unroll
    for (int j = 0; j < 8; j += 2) sum = __builtin_amdgcn_udot4(ex[j], 0x01010101u, sum, false);
    const u32 tot = (u32)__builtin_amdgcn_readlane((int)wave_incl_scan(sum), 63);
    const u32 lo = tot > 50u ? (tot - 50u) >> 10 : 0u;
    const u32 bias = (0x80u - lo) * 0x01010101u;
    u32 P[2] = {0u, 0u}, X = 0, n2 = 0, n3 = 0, b3 = 0, q[8], n2j[8];
#pragma unroll
    for (int j = 0; j < 8; ++j) {
        const u32 a7 = ex[j] + bias;
        const u32 t = ((~a7 | (a7 - 0x03030303u)) & 0x80808080u) >> 7;
        P[j >> 2] |= ((a7 | t | (t << 1)) & 0x03030303u) << (2 * (j & 3));
        X = (X >> 1) | ((~(a7 + 0x0a0a0a0au) | (a7 - 0x04040404u)) & 0x80808080u);
        const u32 g7 = __builtin_amdgcn_perm(0x7f7f7f7fu, a7, lds32(__builtin_amdgcn_udot4(t, 0x20100804u, lut, false)));
        const u32 cz = and3(g7, (g7 | 0x80808080u) >> 5, 0x07070707u);
        const u32 f = (~(g7 + 0x02020202u) & 0x80808080u) >> 7;
        q[j] = __builtin_amdgcn_udot4(__builtin_amdgcn_perm(0x00010000u, 0x00000002u, cz) | f | (f << 1), 0x40100401u, 0u, false);
        const u32 c3 = __builtin_amdgcn_perm(0u, cz, lds32(__builtin_amdgcn_udot4(f, 0x20100804u, lut + 64u, false)));
        b3 |= __builtin_amdgcn_udot4(c3, 0x003e0801u, c3 >> 15, false) << (n3 & 31u);
        n3 = __builtin_amdgcn_udot4(f, 0x03030303u, n3, false);
        n2j[j] = n2;
        n2 = __builtin_amdgcn_udot4(t, 0x02020202u, n2, false);
    }
    const u32 qa = q[0] | (q[1] << n2j[1]) | (q[2] << n2j[2]) | (q[3] << n2j[3]);
    const u32 qb = q[4] | (q[5] << (n2j[5] - n2j[4])) | (q[6] << (n2j[6] - n2j[4])) | (q[7] << (n2j[7] - n2j[4]));
    const u64 b2 = (u64)qa | ((u64)qb << n2j[4]);
    const u32 both = n2 | (n3 << 16), incl = wave_incl_scan(both), pre = incl - both;
    const u32 total = (u32)__builtin_amdgcn_readlane((int)incl, 63);
    const u32 len2 = (((total & 0xffffu) + 31u) >> 5) << 2, len3 = (((total >> 16) + 31u) >> 5) << 2;
    const u32 used = 4u + len2 + len3;
    bool raw = __any(n3 > 30u) || used > VMID - 16u;
    u32 nexc = 0;
    if (!raw) {
        const u32 p2 = pre & 0xffffu, p3 = pre >> 16;
        LAS u32* w2 = (LAS u32*)(img + (h ? VMID + 4u : VMID - 4u - len2) + ((p2 >> 5) << 2));
        LAS u32* w3 = (LAS u32*)(img + (h ? VMID + 4u + len2 : VMID - used) + ((p3 >> 5) << 2));
        const u64 u2 = b2 << (p2 & 31u), u3 = (u64)b3 << (p3 & 31u);
        __hip_atomic_fetch_or(w2, (u32)u2, __ATOMIC_RELAXED, __HIP_MEMORY_SCOPE_WAVEFRONT);
        __hip_atomic_fetch_or(w2 + 1, (u32)(u2 >> 32), __ATOMIC_RELAXED, __HIP_MEMORY_SCOPE_WAVEFRONT);
        __hip_atomic_fetch_or(w2 + 2, (u32)(b2 >> 32) >> 1 >> (31u - (p2 & 31u)), __ATOMIC_RELAXED, __HIP_MEMORY_SCOPE_WAVEFRONT);
        __hip_atomic_fetch_or(w3, (u32)u3, __ATOMIC_RELAXED, __HIP_MEMORY_SCOPE_WAVEFRONT);
        __hip_atomic_fetch_or(w3 + 1, (u32)(u3 >> 32), __ATOMIC_RELAXED, __HIP_MEMORY_SCOPE_WAVEFRONT);
        u64 bal;
        while ((bal = __ballot(X != 0u)) != 0ull) {
            const u32 slotn = nexc + __builtin_amdgcn_mbcnt_hi((u32)(bal >> 32), __builtin_amdgcn_mbcnt_lo((u32)bal, 0u));
            nexc += (u32)__builtin_popcountll(bal);
            if (nexc > 64u || used + 2u * nexc > VMID - 16u) { raw = true; break; }
            if (X != 0u) {
                const u32 k = (u32)__builtin_ctz(X), j = k & 7u, i8 = k & 24u;
                const u32 e01 = (j & 1) ? ex[1] : ex[0], e23 = (j & 1) ? ex[3] : ex[2], e45 = (j & 1) ? ex[5] : ex[4], e67 = (j & 1) ? ex[7] : ex[6];
                const u32 e03 = (j & 2) ? e23 : e01, e47 = (j & 2) ? e67 : e45, ej = (j & 4) ? e47 : e03;
                const u32 n01 = (j & 1) ? n2j[1] : n2j[0], n23 = (j & 1) ? n2j[3] : n2j[2], n45 = (j & 1) ? n2j[5] : n2j[4], n67 = (j & 1) ? n2j[7] : n2j[6];
                const u32 n03 = (j & 2) ? n23 : n01, n47 = (j & 2) ? n67 : n45, nj = (j & 4) ? n47 : n03;
                const u32 Pj = ((j & 4) ? P[1] : P[0]) >> (2u * (j & 3u));
                const u32 pos = ((p2 + nj) >> 1) + (u32)__builtin_popcount(Pj & (Pj >> 1) & 0x01010101u & ((1u << i8) - 1u));
                *(LAS unsigned short*)(img + (h ? VMID + used + 2u * slotn : VMID - used - 2u - 2u * slotn)) = (unsigned short)(pos | (((ej >> i8) & 31u) << 11));
                X &= X - 1u;
            }
        }
    }
    if (raw) {
        st16_sys(slot + lane * 16, (u32x4){d[0], d[1], d[2], d[3]});
        st16_sys(slot + 1024 + lane * 16, (u32x4){d[4], d[5], d[6], d[7]});
        asm volatile("s_mov_b64 exec, 1\n\tglobal_store_dword %0, %1, off sc0 sc1\n\ts_mov_b64 exec, -1" ::"v"(vslot + (h ? VMID : VMID - 4u)), "v"(H_RAW) : "memory");
        return;
    }
    const u32 nvar = (used + 2u * nexc + 15u) >> 4;
    if (lane == 0) *(LAS u32*)(img + (h ? VMID : VMID - 4u)) = lo | (nexc << 5) | (len3 << 10) | (len2 << 17) | (nvar * 16u > VWIN ? H_BIG : 0u);
    const u32 first = h ? VMID : VMID - nvar * 16u;
    const u32x4 v = *(LAS const u32x4*)(img + first + lane * 16);
    if (O == 0) {
        u32x4 sm;
#pragma unroll
        for (int k = 0; k < 4; ++k) sm[k] = bfi(0x87878787u, d[2 * k], rotr(d[2 * k + 1], 28));
        st16_sys(slot + lane * 16, sm);
        asm volatile("global_store_dwordx2 %0, %1, off offset:1024 sc0 sc1\n\ts_nop 1" ::"v"(slot + lane * 8), "v"((u32x2){P[0], P[1]}) : "memory");
    } else {
        u32x4 sm;
        sm[0] = bfi(0x60606060u, d[6] << 5, bfi(0x83838383u, d[0], rotr(d[1], 29)));
        sm[1] = bfi(0x60606060u, d[7] << 5, bfi(0x83838383u, d[2], rotr(d[3], 29)));
        sm[2] = bfi(0x60606060u, bfi(0x40404040u, d[6] >> 1, d[7] >> 2), bfi(0x83838383u, d[4], rotr(d[5], 29)));
        sm[3] = P[0];
        st16_sys(slot + lane * 16, sm);
        const u32 up = __builtin_amdgcn_permlane32_swap(P[1], P[1], false, false)[1];
        if (lane < 32) asm volatile("global_store_dwordx2 %0, %1, off offset:1024 sc0 sc1\n\ts_nop 1" ::"v"(slot + lane * 8), "v"((u32x2){P[1], up}) : "memory");
    }
    if ((u32)lane < nvar) st16_sys(vslot + first + lane * 16, v);
}

__device__ __forceinline__ u32 spread3(u32 v) {
    const u32 x = bfi(0x3fu, v, v << 10);
    return bfi(0x00070007u, x, x << 5) & 0x07070707u;
}
__device__ __forceinline__ u32 spread2(u32 v) {
    const u32 x = bfi(0xfu, v, v << 12);
    return bfi(0x00030003u, x, x << 6);
}

template <int O, class F>
__device__ __forceinline__ void unpack_unit(const u32x4 A, const u32x2 B, const u32 hdr, const bool H, u32 (&d)[8], u32 mid, u32 exp, u32 lut, int lane, F&& midway) {
    constexpr int SH = O ? 2 : 3;
    const u32 P[2] = {O ? A[3] : B[0], O ? __builtin_amdgcn_permlane32_swap(B[0], B[1], false, false)[0] : B[1]};
    const u32 lo = hdr & 31u, nexc = (hdr >> 5) & 127u, len3 = (hdr >> 10) & 0x1fcu, len2 = (hdr >> 17) & 0x1fcu;
    const u32 s2 = H ? mid + 4u : mid - 4u - len2, s3 = H ? s2 + len2 : s2 - len3;
    u32 c = *(LAS const unsigned short*)(size_t)(s2 + (u32)lane * 2u);
    const u32 E[2] = {and3(P[0], P[0] >> 1, 0x55555555u), and3(P[1], P[1] >> 1, 0x55555555u)};
    const u32 cnt = (u32)__builtin_popcount(E[0]) + (u32)__builtin_popcount(E[1]);
    const u32 prim = ((lo * 0x01010101u + 0x00020100u) << SH);
    const u32 b8 = ((lo + 24u) & 31u) * 0x01010101u;
    const u32 t3lo = ((b8 + 0x03020100u) & 0x1f1f1f1fu) << SH, t3hi = ((b8 + 0x1f1e0504u) & 0x1f1f1f1fu) << SH;
    const u32 t2 = ((b8 + 0x000b0607u) & 0x001f1f1fu) << SH;
    u32 at = 0, m3 = 0;
    for (u32 base = 0;;) {
        const u32 c0 = spread2(c), c1 = spread2(c >> 8);
        const u32 f0 = and3(c0, c0 >> 1, 0x01010101u), f1 = and3(c1, c1 >> 1, 0x01010101u);
        const u32 k0 = __builtin_amdgcn_udot4(f0, 0x03030303u, 0u, false), k = __builtin_amdgcn_udot4(f1, 0x03030303u, k0, false);
        const u32 g0 = lds32(__builtin_amdgcn_udot4(f0, 0x20100804u, lut, false)), g1 = lds32(__builtin_amdgcn_udot4(f1, 0x20100804u, lut, false));
        const u32 both = k | (cnt << 16), incl = wave_incl_scan(both), pre = incl - both;
        if (base == 0) at = exp + (pre >> 16);
        const u32 bit = m3 + (pre & 0xffffu);
        LAS const volatile u32* wp = (LAS const volatile u32*)(size_t)(s3 + ((bit >> 5) << 2));
        const u32 w0 = wp[0], w1 = wp[1];
        const u32 w = __builtin_amdgcn_alignbit(w1, w0, bit);
        const u32 e0 = __builtin_amdgcn_perm(t3hi, t3lo, spread3(w)), e1 = __builtin_amdgcn_perm(t3hi, t3lo, spread3(w >> k0));
        *(LAS u32x2*)(size_t)(exp + base * 4u + (u32)lane * 8u) = (u32x2){__builtin_amdgcn_perm(e0, t2, (c0 & 0x03030303u) ^ g0), __builtin_amdgcn_perm(e1, t2, (c1 & 0x03030303u) ^ g1)};
        base += 128u;
        if (__builtin_expect(base >= len2, 1)) break;
        m3 += (u32)__builtin_amdgcn_readlane((int)incl, 63) & 0xffffu;
        c = *(LAS const unsigned short*)(size_t)(s2 + base + (u32)lane * 2u);
    }
    if (nexc) {
        if ((u32)lane < nexc) {
            const u32 e = *(LAS const unsigned short*)(size_t)(H ? s3 + len3 + 2u * (u32)lane : s3 - 2u - 2u * (u32)lane);
            *(LAS unsigned char*)(size_t)(exp + (e & 0x7ffu)) = (unsigned char)((e >> 11) << SH);
        }
    }
    midway();
    u32 ptr[8], dep[8], ex[8];
#pragma unroll
    for (int j = 0; j < 8; ++j) {
        const int g = j & 3;
        const u32 Eh = g < 2 ? E[j >> 2] : E[j >> 2] >> 4;
        const u32 m = Eh & ((g & 1) ? 0x04040404u : 0x01010101u);
        dep[j] = lds32(__builtin_amdgcn_udot4(m, (g & 1) ? 0x08040201u : 0x20100804u, lut, false));
        ptr[j] = at;
        at += (u32)__builtin_popcount(m);
    }
    u32x2 wn[8];
    asm volatile("ds_read2_b32 %0, %8 offset1:1\n\tds_read2_b32 %1, %9 offset1:1\n\tds_read2_b32 %2, %10 offset1:1\n\tds_read2_b32 %3, %11 offset1:1\n\t"
                 "ds_read2_b32 %4, %12 offset1:1\n\tds_read2_b32 %5, %13 offset1:1\n\tds_read2_b32 %6, %14 offset1:1\n\tds_read2_b32 %7, %15 offset1:1\n\t"
                 "s_waitcnt lgkmcnt(0)"
                 : "=&v"(wn[0]), "=&v"(wn[1]), "=&v"(wn[2]), "=&v"(wn[3]), "=&v"(wn[4]), "=&v"(wn[5]), "=&v"(wn[6]), "=&v"(wn[7])
                 : "v"(ptr[0] & ~3u), "v"(ptr[1] & ~3u), "v"(ptr[2] & ~3u), "v"(ptr[3] & ~3u), "v"(ptr[4] & ~3u), "v"(ptr[5] & ~3u), "v"(ptr[6] & ~3u), "v"(ptr[7] & ~3u)
                 : "memory");
#pragma unroll
    for (int j = 0; j < 8; ++j)
        ex[j] = __builtin_amdgcn_perm(__builtin_amdgcn_alignbyte(wn[j][1], wn[j][0], ptr[j]), prim, ((P[j >> 2] >> (2 * (j & 3))) & 0x03030303u) ^ dep[j]);
    if (O == 0) {
#pragma unroll
        for (int k = 0; k < 4; ++k) { d[2 * k] = (A[k] & 0x87878787u) | ex[2 * k]; d[2 * k + 1] = (rotr(A[k], 4) & 0x87878787u) | ex[2 * k + 1]; }
    } else {
#pragma unroll
        for (int k = 0; k < 3; ++k) { d[2 * k] = (A[k] & 0x83838383u) | ex[2 * k]; d[2 * k + 1] = (rotr(A[k], 3) & 0x83838383u) | ex[2 * k + 1]; }
        d[6] = (bfi(0x80808080u, A[2] << 1, A[0] >> 5) & 0x83838383u) | ex[6];
        d[7] = (bfi(0x80808080u, A[2] << 2, A[1] >> 5) & 0x83838383u) | ex[7];
    }
}

__host__ __device__ constexpr u32 task_unit(u32 b, u32 wid) { return (b >> 3) * 64u + (wid >> 2) * 32u + (b & 7u) * 4u + (wid & 3u); }
__host__ __device__ constexpr u32 unit_task(u32 k) { return (k >> 6) * 8u + ((k & 31u) >> 2); }
__device__ __forceinline__ u32 stage_off(u32 k) { return ((k >> 5) << 19) | ((k & 31u) << 11); }

__device__ __forceinline__ void load_unit(const Args& a, u32 k, int lane, u32x4& r0, u32x4& r1) {
    const u32 t = k >> 6, rows = (k & 31u) * 128u;
    const unsigned char* src = (k & 32u) ? a.w + (size_t)(16u * t + (lane >> 3)) * DIM + rows + (lane & 7) * 16
                                         : a.x + (size_t)(rows + lane) * KS + 16u * t;
    r0 = *(const u32x4*)src; r1 = *(const u32x4*)(src + 32768);
}
__device__ __forceinline__ void retile(u32 k, const u32x4 r0, const u32x4 r1, u32 (&d)[8], LAS char* wl, int lane) {
    if ((k & 32u) == 0u) {
#pragma unroll
        for (int j = 0; j < 4; ++j) { d[j] = r0[j]; d[4 + j] = r1[j]; }
    } else {
        *(LAS u32x4*)(wl + (lane >> 3) * 160 + (lane & 7) * 16) = r0;
        *(LAS u32x4*)(wl + (8 + (lane >> 3)) * 160 + (lane & 7) * 16) = r1;
        LAS char* tb = wl + ((lane & 15) >> 1) * 160 + (lane >> 4) * 16 + (lane & 1) * 8;
        asm volatile("" ::: "memory");
        const v2i c00 = __builtin_amdgcn_ds_read_tr8_b64_v2i32((LAS v2i*)tb);
        const v2i c01 = __builtin_amdgcn_ds_read_tr8_b64_v2i32((LAS v2i*)(tb + 8 * 160));
        const v2i c10 = __builtin_amdgcn_ds_read_tr8_b64_v2i32((LAS v2i*)(tb + 64));
        const v2i c11 = __builtin_amdgcn_ds_read_tr8_b64_v2i32((LAS v2i*)(tb + 8 * 160 + 64));
        const u32 g0 = (k & 31u) * 2u, g1 = g0 + 1u;
        const int mixed = ((lane & 15) * 4 + (lane >> 4)) * 4;
        const int s0 = g0 % 5u == 4u ? lane * 4 : mixed, s1 = (g1 % 5u == 4u || g1 == 63u) ? lane * 4 : mixed;
        d[0] = __builtin_amdgcn_ds_bpermute(s0, c00[0]); d[1] = __builtin_amdgcn_ds_bpermute(s0, c00[1]);
        d[2] = __builtin_amdgcn_ds_bpermute(s0, c01[0]); d[3] = __builtin_amdgcn_ds_bpermute(s0, c01[1]);
        d[4] = __builtin_amdgcn_ds_bpermute(s1, c10[0]); d[5] = __builtin_amdgcn_ds_bpermute(s1, c10[1]);
        d[6] = __builtin_amdgcn_ds_bpermute(s1, c11[0]); d[7] = __builtin_amdgcn_ds_bpermute(s1, c11[1]);
    }
}
__device__ __forceinline__ u32 slot_unit(u32 k, u32 par) { return k + (k < 128u ? par * NUNITS : 0u); }
__device__ __forceinline__ void pack_tables(LAS char* wl, int lane) {
    u32 sel = 0x04040404u; int n = 0;
#pragma unroll
    for (int i = 0; i < 4; ++i) if ((lane >> i) & 1) { sel = (sel & ~(0xffu << (8 * n))) | ((u32)i << (8 * n)); ++n; }
    if (lane & 16) sel |= (sel & 0x04040404u) << 1;
    if (lane < 32) *(LAS u32*)(wl + P0_LUT + lane * 4) = sel;
}
__device__ __forceinline__ void stage_unit(const Args& a, LAS char* wl, u32 k, bool plain, u32 par, const u32 (&d)[8], int lane) {
    char* own = a.me + STAGE_OFF + (size_t)a.rank * 65536 + stage_off(slot_unit(k, par)) + lane * 16;
    st16_agent(own, (u32x4){d[0], d[1], d[2], d[3]}); st16_agent(own + 1024, (u32x4){d[4], d[5], d[6], d[7]});
    if (!plain) {
        char* slot = a.me + OUTBOX_OFF + par * SRC_BYTES + (size_t)k * 2048;
        char* vslot = a.me + VBOX_OFF + par * VBOX_HALF + (size_t)(k >> 1) * 1024;
        if (k & 32u) pack_unit<1>(d, slot, vslot, k & 1u, wl + P0_VAR, (u32)(size_t)(wl + P0_LUT), lane);
        else pack_unit<0>(d, slot, vslot, k & 1u, wl + P0_VAR, (u32)(size_t)(wl + P0_LUT), lane);
    }
}
__device__ __forceinline__ void publish_task(char* const (&w)[NR], u32 rank, u32 b, u32 call, int lane) {
    const u32 off = (u32)RDY_OFF + (call & 1u) * (u32)RDY_HALF + (lane == 0 ? rank * RDY_SRC + b * 256u : RDY_DENSE + rank * RDY_DENSE_SRC + b * 4u);
    asm volatile("s_mov_b64 exec, 3\n\tglobal_store_dword %0, %1, %2 sc0 sc1\n\tglobal_store_dword %0, %1, %3 sc0 sc1\n\tglobal_store_dword %0, %1, %4 sc0 sc1\n\t"
                 "global_store_dword %0, %1, %5 sc0 sc1\n\tglobal_store_dword %0, %1, %6 sc0 sc1\n\tglobal_store_dword %0, %1, %7 sc0 sc1\n\t"
                 "global_store_dword %0, %1, %8 sc0 sc1\n\tglobal_store_dword %0, %1, %9 sc0 sc1\n\ts_mov_b64 exec, -1"
                 ::"v"(off), "v"(call), "s"(w[0]), "s"(w[1]), "s"(w[2]), "s"(w[3]), "s"(w[4]), "s"(w[5]), "s"(w[6]), "s"(w[7]) : "memory");
}
__device__ __forceinline__ u32 first_unit(u32 b, u32 wid) { return SHARE && b < PUSH_TASKS && wid < 4u ? task_unit(b + PUSH_TASKS, wid + 4u) : task_unit(b, wid); }
__device__ __forceinline__ u32 word_of(u32 k) { const u32 b = unit_task(k); return SHARE && (k & 32u) && b < 2u * PUSH_TASKS ? b - PUSH_TASKS : b; }
__device__ __forceinline__ u32 stage_out(const Args& a, LAS char* lds, const u32 prev, const u32x4 r0, const u32x4 r1) {
    const int tid = threadIdx.x, wid = __builtin_amdgcn_readfirstlane(tid >> 6), lane = tid & 63;
    const u32 b = blockIdx.x, k = first_unit(b, (u32)wid);
    const bool copies = b < PUSH_TASKS && (!SHARE || wid >= 4), idle = SHARE && b >= PUSH_TASKS && b < 2 * PUSH_TASKS && wid >= 4;
    LAS char* wl = lds + wid * P0_WAVE;
    u32x4 s0 = {}, s1 = {};
    if (SHARE && copies) load_unit(a, task_unit(b, (u32)wid - 4u), lane, s0, s1);
    pack_tables(wl, lane);
    *(LAS u32x4*)(wl + P0_VAR + lane * 16) = (u32x4){0u, 0u, 0u, 0u};
    SCHED;
    u32 d[8];
    retile(k, r0, r1, d, wl, lane);
    const u32 call = (u32)__builtin_amdgcn_readfirstlane((int)prev) + 1u, par = call & 1u;
    if (SHARE && copies) {
        char* own = a.me + STAGE_OFF + (size_t)a.rank * 65536 + stage_off(slot_unit(k - 32u, par)) + lane * 16;
        st16_agent(own, s0); st16_agent(own + 1024, s1);
    }
    if (!idle) stage_unit(a, wl, k, copies, par, d, lane);
    char* w[NR];
#pragma unroll
    for (int p = 0; p < NR; ++p) w[p] = a.ws[p];
    asm volatile("" : "+s"(w[0]), "+s"(w[1]), "+s"(w[2]), "+s"(w[3]), "+s"(w[4]), "+s"(w[5]), "+s"(w[6]), "+s"(w[7]));
    asm volatile("s_waitcnt vmcnt(0)" ::: "memory");
    __syncthreads();
    if (wid == 0) publish_task(w, (u32)a.rank, b, call, lane);
    if (!idle) st32_agent_lane0((u32*)(a.me + DONE_OFF + ((size_t)slot_unit(k, par) << DONE_SHIFT)) + a.rank, call);
    if (SHARE && copies) st32_agent_lane0((u32*)(a.me + DONE_OFF + ((size_t)slot_unit(k - 32u, par) << DONE_SHIFT)) + a.rank, call);
    return call;
}
__device__ __forceinline__ void stage_late(const Args& a, LAS char* wl, u32 b, u32 call, int lane) {
    pack_tables(wl, lane);
#pragma nounroll
    for (u32 i = 0; i < 8; ++i) {
        const u32 k = task_unit(b, i);
        u32x4 r0, r1; u32 d[8];
        load_unit(a, k, lane, r0, r1);
        *(LAS u32x4*)(wl + P0_VAR + lane * 16) = (u32x4){0u, 0u, 0u, 0u};
        retile(k, r0, r1, d, wl, lane);
        stage_unit(a, wl, k, false, call & 1u, d, lane);
    }
    char* w[NR];
#pragma unroll
    for (int p = 0; p < NR; ++p) w[p] = a.ws[p];
    asm volatile("s_waitcnt vmcnt(0)" ::: "memory");
    publish_task(w, (u32)a.rank, b, call, lane);
    const u32 own = ((task_unit(b, (u32)lane & 7u) << DONE_SHIFT) + (u32)a.rank * 4u);
    asm volatile("s_mov_b64 exec, 0xff\n\tglobal_store_dword %0, %1, %2 sc1\n\ts_mov_b64 exec, -1" ::"v"(own), "v"(call), "s"(a.me + DONE_OFF) : "memory");
}

constexpr int PL_WAVE = 768 + 2048 + 2560 + 64;
#define PULL_WPW 7
constexpr u32 PULL_WAVES = PULL_SLOTS * 8 * PULL_WPW;

__device__ __forceinline__ void pull(const Args& a, u32* cnt, int c, LAS char* lds) {
    const int tid = threadIdx.x, wid = __builtin_amdgcn_readfirstlane(tid >> 6), lane = tid & 63;
    const u32 gw = (u32)c * PULL_WPW + (u32)wid, rk = gw / 7u, link = PULL_WPW == 7 ? ((u32)wid + (u32)c) % 7u : gw % 7u;
    const u32 nw = (PULL_WAVES + 6u - link) / 7u;
    auto push_unit = [](u32 i) -> u32 { return (i >> 6) * 64u + ((i >> 1) & 1u) * 32u + ((i & 63u) >> 2) * 2u + (i & 1u); };
    static_assert(NPUSH <= 2 * (PULL_WAVES / 7) && NPUSH >= (PULL_WAVES + 6) / 7, "one or two pushed units per wave");
    const bool pa = wid < PULL_WPW, pb = pa && rk + nw < NPUSH;
    const u32 ka = push_unit(pa ? rk : 0u), kb = push_unit(pb ? rk + nw : 0u);
    u32x4 a0, a1, b0 = {}, b1 = {};
    load_unit(a, ka, lane, a0, a1);
    const u32 prev = __hip_atomic_load(cnt, __ATOMIC_RELAXED, __HIP_MEMORY_SCOPE_AGENT);
    if (pb) load_unit(a, kb, lane, b0, b1);
    const u32 ord = ((const u32*)ORDER.item)[tid], ord2 = ((const u32*)ORDER.item)[512 + (tid & 127)];
    SCHED;
    const u32 s = ((u32)a.rank + 1u + link) & 7u;
    char* myws = a.me;
    char* peer = uniform_ptr(a.ws[s]);
    u32* tmo = (u32*)(myws + TMO_OFF);
    const u32 img = (u32)(size_t)(lds + 4096 + wid * PL_WAVE + 768), exp = img + 2048u, lut = exp + 2560u;
    if (lane < 16) {
        u32 dep = 0; u32 n = 0;
#pragma unroll
        for (int i = 0; i < 4; ++i) if ((lane >> i) & 1) { dep |= (3u ^ (4u + n)) << (8 * i); ++n; }
        *(LAS u32*)(size_t)(lut + lane * 4) = dep;
    }
    const u32 l16 = (u32)lane * 16u, l8 = (u32)lane * 8u;
    u32 pd[8];
    retile(ka, a0, a1, pd, (LAS char*)(size_t)exp, lane);
    const u32 call = (u32)__builtin_amdgcn_readfirstlane((int)prev) + 1u, par = call & 1u;
    const u32x4 rpush = make_rsrc(peer + STAGE_OFF + a.rank * 65536u, (u32)STAGE_BYTES - a.rank * 65536u);
    const u32x4 rpdone = make_rsrc(peer + DONE_OFF + a.rank * 4u, (NUNITS + 128u) << DONE_SHIFT);
    u32 vcall = call; asm volatile("" : "+v"(vcall));
#define PUSH(d, k) asm volatile("s_nop 4\n\tbuffer_store_dwordx4 %0, %2, %3, %4 offen sc0 sc1\n\tbuffer_store_dwordx4 %1, %2, %3, %4 offen offset:1024 sc0 sc1\n\ts_nop 1" \
                     ::"v"((u32x4){d[0], d[1], d[2], d[3]}), "v"((u32x4){d[4], d[5], d[6], d[7]}), "v"(l16), "s"(rpush), "s"(stage_off(slot_unit(k, par))) : "memory")
#define PUSHED(k) asm volatile("s_waitcnt vmcnt(0)\n\ts_mov_b64 exec, 1\n\tbuffer_store_dword %0, off, %1, %2 sc0 sc1\n\ts_mov_b64 exec, -1" \
                     ::"v"(vcall), "s"(rpdone), "s"(slot_unit(k, par) << DONE_SHIFT) : "memory")
    __builtin_amdgcn_s_waitcnt(0x0F70);
    if (pa) PUSH(pd, ka);
    if (pb) retile(kb, b0, b1, pd, (LAS char*)(size_t)exp, lane);
    *(LAS u32*)(lds + tid * 4) = ord; *(LAS u32*)(lds + 2048 + (tid & 127) * 4) = ord2;
    asm volatile("s_waitcnt lgkmcnt(0)" ::: "memory");
    __builtin_amdgcn_s_barrier();
    asm volatile("" ::: "memory");
    if (wid == 0) st32_agent_lane0(cnt, call);
    if (wid >= PULL_WPW) {
        static_assert(PULL_WPW <= 7, "a spare wave per pulling workgroup");
        if (wid < 7) return;
        __builtin_amdgcn_s_sleep(LATE2_SLEEP);
        stage_late(a, lds + 4096 + 8 * PL_WAVE, GEMM_SLOTS * 8 + (u32)c, call, lane);
        GSTAMP(34);
        return;
    }
    STAMP(1);
    const char* dense = myws + RDY_OFF + par * RDY_HALF + RDY_DENSE + s * RDY_DENSE_SRC + l16;
#define GLANCE() asm volatile("global_load_dwordx4 v[252:255], %0, off sc0 sc1" :: "v"(dense) : "memory", "v252", "v253", "v254", "v255")
    u32 klast = ka;
    if (pb) { PUSHED(ka); PUSH(pd, kb); klast = kb; }
    const u32x4 rbox = make_rsrc(peer + OUTBOX_OFF + par * SRC_BYTES, (u32)(RDY_OFF - OUTBOX_OFF - par * SRC_BYTES));
    const u32x4 rstage = make_rsrc(myws + STAGE_OFF + s * 65536u, (u32)STAGE_BYTES - s * 65536u);
    const u32x4 rdone = make_rsrc(myws + DONE_OFF + s * 4u, (NUNITS + 128u + 511u) << DONE_SHIFT);
    const u32 vdelta = (u32)(VBOX_OFF - OUTBOX_OFF) + par * (u32)VBOX_HALF - par * (u32)SRC_BYTES;
    const char* rdy = myws + RDY_OFF + par * RDY_HALF;
    u32* claim = (u32*)(myws + CLAIM_OFF) + link * 32u;
    const u32 tbase = nw - (call - 1u) * NTICKETS;
    u32 vlut = lut; asm volatile("" : "+v"(vlut));
    u32 known = 0;
    auto lookup = [&](u32 ticket) -> u32 { return (u32)__builtin_amdgcn_readfirstlane(*(LAS const unsigned short*)(lds + ticket * 2u)); };
    auto await = [&](u32 ticket, u32 item, bool refresh) -> bool {
        const u32 b = word_of(item & 2047u);
        {
            if (refresh) {
                u32x4 f;
                asm volatile("global_load_dwordx4 %0, %1, off sc0 sc1\n\ts_waitcnt vmcnt(0)" : "=&v"(f) : "v"(rdy + RDY_DENSE + s * RDY_DENSE_SRC + l16) : "memory");
                const u64 bad = __ballot(f[0] != call || f[1] != call || f[2] != call || f[3] != call);
                known = bad ? (u32)__builtin_ctzll(bad) * 4u : 256u;
                if (b < known) return true;
            }
            const u64 t0 = __builtin_amdgcn_s_memrealtime();
            for (u32 spins = 0;;) {
                u32 f;
                asm volatile("s_mov_b64 exec, 1\n\tglobal_load_dword %0, %1, off sc0 sc1\n\ts_mov_b64 exec, -1\n\ts_waitcnt vmcnt(0)" : "=&v"(f) : "v"(rdy + s * RDY_SRC + b * 256u) : "memory");
                const u32 seq = (u32)__builtin_amdgcn_readfirstlane((int)f);
                if (seq == call) break;
                bool out = (int)(seq - call) > 0;
                if (out) st32_agent_lane0(tmo, call); else out = give_up(tmo, call, t0, spins);
                if (out) {
                    while (ticket < NTICKETS) {
                        u32 r;
                        asm volatile("s_mov_b64 exec, 1\n\tglobal_atomic_add %0, %1, %2, off sc0\n\ts_mov_b64 exec, -1\n\ts_waitcnt vmcnt(0)" : "=&v"(r) : "v"(claim), "v"(1u) : "memory");
                        ticket = (u32)__builtin_amdgcn_readfirstlane((int)r) + tbase;
                    }
                    return false;
                }
                __builtin_amdgcn_s_sleep(POLL_SLEEP);
            }
        }
        return true;
    };
#define ZONE "v240", "v241", "v242", "v243", "v244", "v245", "v246", "v247", "v248", "v249", "v250", "v252", "v253", "v254", "v255"
#define ISSUE(it) do { const u32 k_ = (it) & 2047u; \
        asm volatile("buffer_load_dwordx4 v[240:243], %0, %2, %3 offen sc0 sc1\n\ts_mov_b64 exec, %5\n\tbuffer_load_dwordx2 v[248:249], %1, %2, %3 offen offset:1024 sc0 sc1\n\t" \
                     "s_mov_b64 exec, 0xffffff\n\tbuffer_load_dwordx4 v[244:247], %0, %2, %4 offen offset:256 sc0 sc1\n\t" \
                     "s_mov_b64 exec, 1\n\tglobal_atomic_add v250, %6, %7, off sc0\n\ts_mov_b64 exec, -1" \
                     :: "v"(l16), "v"(l8), "s"(rbox), "s"(k_ << 11), "s"(vdelta + ((k_ >> 1) << 10)), \
                        "s"((k_ & 32u) ? 0xffffffffull : ~0ull), "v"(claim), "v"(1u) : "memory", ZONE); } while (0)
#define TAKE(n) asm volatile("s_waitcnt vmcnt(" #n ")\n\tv_mov_b64 %0, v[240:241]\n\tv_mov_b64 %1, v[242:243]\n\tv_mov_b64 %2, v[248:249]" : "=v"(A0), "=v"(A1), "=v"(B) :: "memory", ZONE)
    auto FLAG = [=](u32 k) { asm volatile("s_mov_b64 exec, 1\n\tbuffer_store_dword %0, off, %1, %2 sc1\n\ts_mov_b64 exec, -1" ::"v"(vcall), "s"(rdone), "s"(k << DONE_SHIFT) : "memory"); };
#define STORE2(d, so) asm volatile("buffer_store_dwordx4 %0, %2, %3, %4 offen sc1\n\tbuffer_store_dwordx4 %1, %2, %3, %4 offen offset:1024 sc1\n\ts_nop 1" \
                     ::"v"((u32x4){d[0], d[1], d[2], d[3]}), "v"((u32x4){d[4], d[5], d[6], d[7]}), "v"(l16), "s"(rstage), "s"(so) : "memory")
    static_assert(VMID - VWIN == 256 && 2 * VWIN == 24 * 16, "the asm above and below has these built in");
    u32 it = lookup(rk);
    u32x2 A0, A1, B;
    const u32 none = NUNITS + 128u + gw;
    u32 prv = none;
#define PST() do { } while (0)
    auto noted = [&]() {
        u32 t; u64 bad;
        asm volatile("v_xor_b32 v252, %2, v252\n\tv_xor_b32 v253, %2, v253\n\tv_xor_b32 v254, %2, v254\n\tv_xor_b32 v255, %2, v255\n\t"
                     "v_or3_b32 %0, v252, v253, v254\n\tv_or_b32 %0, %0, v255\n\tv_cmp_ne_u32 %1, 0, %0\n\ts_nop 1" : "=&v"(t), "=s"(bad) : "s"(call) : "memory", ZONE);
        known = bad ? (u32)__builtin_ctzll(bad) * 4u : 256u;
    };
    asm volatile("s_waitcnt vmcnt(0)\n\tglobal_load_dwordx4 v[252:255], %3, off sc0 sc1\n\ts_mov_b64 exec, 1\n\tbuffer_store_dword %0, off, %1, %2 sc0 sc1\n\t"
                 "s_mov_b64 exec, -1\n\ts_waitcnt vmcnt(1)" ::"v"(vcall), "s"(rpdone), "s"(slot_unit(klast, par) << DONE_SHIFT), "v"(dense) : "memory", ZONE);
    noted();
    if (word_of(it & 2047u) >= known && !await(rk, it, false)) return;
    if (known < 256u) GLANCE();
    ISSUE(it);
    TAKE(0);
    if (known < 256u) noted();
    STAMP(2);
    for (;;) {
        const u32 k = it & 2047u, ks = slot_unit(k, par);
        u32 ticket = 0, hdr0, hdr1, nit = ~0u, d[8];
        asm volatile("v_readlane_b32 %0, v247, 11\n\tv_readlane_b32 %1, v244, 12" : "=&s"(hdr0), "=&s"(hdr1) :: "memory", ZONE);
        PST();
        bool glanced = false;
        bool ahead = false;
        auto advance = [&]() {
            asm volatile("s_waitcnt vmcnt(3)\n\tv_readfirstlane_b32 %0, v250" : "=s"(ticket) :: "memory", ZONE);
            ticket += tbase;
            if (ticket < NTICKETS) {
                nit = lookup(ticket);
                const u32 b = unit_task(nit & 2047u);
                if (__builtin_expect(b < known, 1)) {
                    if (__builtin_expect(b + 48u >= known && known < 256u, 0)) { GLANCE(); glanced = true; }
                    ISSUE(nit); ahead = true;
                }
            }
        };
        auto publish = [&](u32 u) {
            if (__builtin_expect(ahead, 1)) asm volatile("s_waitcnt vmcnt(4)" ::: "memory"); else asm volatile("s_waitcnt vmcnt(0)" ::: "memory");
            FLAG(u);
        };
        u32 last = ks;
        {
            const bool solo = (it & I_SOLO) != 0u;
            const u32 hdr = (k & 1u) ? hdr1 : hdr0, hdrs = solo ? hdr : hdr0 | hdr1;
            if (__builtin_expect((hdrs & H_RAW) != 0u, 0)) {
                last = solo ? ks : ks + 1u;
                asm volatile("s_nop 4\n\tbuffer_load_dwordx4 v[244:247], %0, %1, %2 offen sc0 sc1\n\ts_waitcnt vmcnt(0)\n\tds_write_b128 %3, v[244:247]\n\ts_waitcnt lgkmcnt(0)"
                             :: "v"(l16), "s"(rbox), "s"(vdelta + ((k >> 1) << 10)), "v"(img + l16) : "memory", ZONE);
                FLAG(prv);
#pragma nounroll
                for (u32 u = k;; ++u) {
                    u32x4 R0, R1; u32x2 C;
                    asm volatile("s_nop 4\n\tbuffer_load_dwordx4 %0, %3, %5, %6 offen sc0 sc1\n\tbuffer_load_dwordx4 %1, %3, %5, %6 offen offset:1024 sc0 sc1\n\t"
                                 "buffer_load_dwordx2 %2, %4, %5, %6 offen offset:1024 sc0 sc1\n\ts_waitcnt vmcnt(0)"
                                 : "=&v"(R0), "=&v"(R1), "=&v"(C) : "v"(l16), "v"(l8), "s"(rbox), "s"(u << 11) : "memory");
                    const u32 hu = (u & 1u) ? hdr1 : hdr0;
                    if (hu & H_RAW) {
#pragma unroll
                        for (int j = 0; j < 4; ++j) { d[j] = R0[j]; d[4 + j] = R1[j]; }
                    } else if (u & 32u) unpack_unit<1>(R0, C, hu, (u & 1u) != 0u, d, img + VMID, exp, vlut, lane, []() {});
                    else unpack_unit<0>(R0, C, hu, (u & 1u) != 0u, d, img + VMID, exp, vlut, lane, []() {});
                    if (solo || u != k) break;
                    STORE2(d, stage_off(ks));
                    asm volatile("s_waitcnt vmcnt(0)" ::: "memory");
                    FLAG(ks);
                }
                advance();
                FLAG(none);
            } else {
                u32 vat = img + 256u + l16;
                if (__builtin_expect((hdrs & H_BIG) != 0u, 0)) {
                    asm volatile("s_nop 4\n\tbuffer_load_dwordx4 v[244:247], %0, %1, %2 offen sc0 sc1\n\ts_waitcnt vmcnt(0)" :: "v"(l16), "s"(rbox), "s"(vdelta + ((k >> 1) << 10)) : "memory", ZONE);
                    vat = img + l16;
                }
                asm volatile("ds_write_b128 %0, v[244:247]" :: "v"(vat) : "memory", ZONE);
                auto body = [&](auto oc) {
                    constexpr int O = decltype(oc)::value;
                    if (__builtin_expect(!solo, 1)) {
                        asm volatile("buffer_load_dwordx4 v[240:243], %0, %2, %3 offen offset:2048 sc0 sc1\n\ts_mov_b64 exec, %4\n\t"
                                     "buffer_load_dwordx2 v[248:249], %1, %2, %3 offen offset:3072 sc0 sc1\n\ts_mov_b64 exec, -1"
                                     :: "v"(l16), "v"(l8), "s"(rbox), "s"(k << 11), "s"(O ? 0xffffffffull : ~0ull) : "memory", ZONE);
                        unpack_unit<O>((u32x4){A0[0], A0[1], A1[0], A1[1]}, B, hdr0, false, d, img + VMID, exp, vlut, lane,
                                       [&]() { asm volatile("s_waitcnt vmcnt(2)" ::: "memory"); FLAG(prv); });
                        WSTAMP(k >> 6); PST();
                        STORE2(d, stage_off(ks));
                        TAKE(3);
                        PST();
                        prv = ks; last = ks + 1u;
                    }
                    advance();
                    PST();
                    unpack_unit<O>((u32x4){A0[0], A0[1], A1[0], A1[1]}, B, (last & 1u) ? hdr1 : hdr0, (last & 1u) != 0u, d, img + VMID, exp, vlut, lane, [&]() { publish(prv); });
                };
                if (k & 32u) body(std::integral_constant<int, 1>{}); else body(std::integral_constant<int, 0>{});
            }
        }
        PST();
        STORE2(d, stage_off(last));
        if (__builtin_expect(ahead, 1)) {
            TAKE(4);
            if (__builtin_expect(glanced, 0)) noted();
            prv = last;
        } else {
            asm volatile("s_waitcnt vmcnt(0)" ::: "memory");
            FLAG(last);
            prv = none;
            if (nit == ~0u || !await(ticket, nit, false)) break;
            GLANCE();
            ISSUE(nit);
            TAKE(0);
            noted();
        }
        it = nit;
    }
#undef ISSUE
#undef PUSH
#undef PUSHED
#undef TAKE
#undef GLANCE
#undef PST
#undef ZONE
#undef STORE2
    GSTAMP(32);
    STAMP(3);
}

__device__ __forceinline__ void gemm(const Args& a, u32 call, int xcd, int slot, LAS char* lds) {
    const int tid = threadIdx.x, wid = __builtin_amdgcn_readfirstlane(tid >> 6), lane = tid & 63;
    const int li = (xcd & 1) * GEMM_SLOTS + slot, ti = (xcd >> 1) * 4 + (li & 3), tj = li >> 2;
    const int row0 = ti * BM, col0 = tj * BN > DIM - BN ? DIM - BN : tj * BN;
    const int wr = wid >> 2, wc = wid & 3, fr = lane & 15, fq = lane >> 4;
    char* myws = a.me;
    u32* tmo = (u32*)(myws + TMO_OFF);
    const __amdgpu_buffer_rsrc_t rf = __builtin_amdgcn_make_buffer_rsrc(myws + DONE_OFF, 0, (int)(DBG_OFF - DONE_OFF), 0x00020000);
    const __amdgpu_buffer_rsrc_t rs = __builtin_amdgcn_make_buffer_rsrc(myws + STAGE_OFF, 0, (int)STAGE_BYTES, 0x00020000);
    const int vx = (row0 + lane) * 16, vw = 8 * 65536 + (col0 + lane) * 16;
    const int xoff = 2 * fq * XS + (wr * 128 + fr) * 16, woff = XB + 2 * fq * WS + (wc * 64 + fr) * 16, woff4 = woff + (256 - wc * 48) * 16;
    const int alt = (int)(call & 1u) * NT;
    auto piece_of = [&](int v) -> int {
        int t, r = a.rank;
        if (v < OWN_FIRST) t = v * 8 + wid;
        else if (v == OWN_LAST) t = OWN_FIRST * 8 + wid;
        else {
            const u32 g = (u32)(v - OWN_FIRST - (v > OWN_LAST ? 1 : 0)) * 8u + (u32)wid, q = (g * 9363u) >> 16;
            t = (int)q; r = (int)(((u32)a.rank + 1u + g + q) & 7u);
        }
        return ((t + (t < 2 ? alt : 0)) << 4) | r;
    };
    const int fv = (lane < 2 ? ti * 2 + lane : lane < 5 ? 30 + (col0 >> 7) + lane : ti * 2) << DONE_SHIFT;
    LAS u32* snap = (LAS u32*)(lds + 2 * STG + wid * 256);
    LAS u32* bad = (LAS u32*)(lds + 2 * STG + 2048);
    bool dead = false;
    auto wait_piece = [&](int pc) {
        if (dead) return;
        const char* fl = myws + DONE_OFF + fv + ((pc >> 4) << (DONE_SHIFT + 6)) + (pc & 7) * 4;
        const u64 t0 = __builtin_amdgcn_s_memrealtime();
        for (u32 spins = 0;;) {
            u32 seen;
            asm volatile("s_mov_b64 exec, 31\n\tglobal_load_dword %0, %1, off sc0 sc1\n\ts_mov_b64 exec, -1\n\ts_waitcnt vmcnt(0)" : "=&v"(seen) : "v"(fl) : "memory");
            if ((__ballot(seen == call) & 31ull) == 31ull) break;
            if (give_up(tmo, call, t0, spins)) { dead = true; *(LAS volatile u32*)bad = call; break; }
            __builtin_amdgcn_s_sleep(GEMM_SLEEP);
        }
    };
#define DMA_X(pc, b, i) __builtin_amdgcn_raw_ptr_buffer_load_lds(rs, (LAS void*)(lds + (b) * STG + wid * XS), 16, vx, (pc) << 16, (i) * 1024, 17)
#define DMA_W(pc, b, i) __builtin_amdgcn_raw_ptr_buffer_load_lds(rs, (LAS void*)(lds + (b) * STG + XB + wid * WS + ((i) == 4 ? 4096 : 0)), 16, vw, ((pc) << 16) + ((i) == 4 ? 4096 : 0), ((i) & 3) * 1024, 17)
    v4f acc[8][5];
#pragma unroll
    for (int m = 0; m < 8; ++m)
#pragma unroll
        for (int n = 0; n < 5; ++n) acc[m][n] = (v4f){0.f, 0.f, 0.f, 0.f};
    const float scale = __builtin_bit_cast(float, __builtin_amdgcn_readfirstlane(__builtin_bit_cast(int, a.sx[0] * a.sw[0])));
    __builtin_amdgcn_s_waitcnt(0x0F70);
    int rscale = __builtin_amdgcn_readfirstlane(__float_as_int(__builtin_amdgcn_rcpf(scale)));
    u32x4 ro = make_rsrc(a.out + (size_t)(row0 + wr * 128) * DIM + col0, (u32)(128 * DIM - col0) * 4u);
    asm volatile("" : "+s"(rscale), "+s"(ro));
    if (tid == 0) *bad = 0u;
    asm volatile("s_waitcnt lgkmcnt(0)" ::: "memory");
    __builtin_amdgcn_s_barrier(); asm volatile("" ::: "memory");
    {
        const int pc = piece_of(0);
        wait_piece(pc);
        STAMP(2);
        DMA_W(pc, 0, 0); DMA_W(pc, 0, 1); DMA_W(pc, 0, 2); DMA_W(pc, 0, 3); DMA_W(pc, 0, 4); DMA_X(pc, 0, 0); DMA_X(pc, 0, 1); DMA_X(pc, 0, 2); DMA_X(pc, 0, 3);
    }
    asm volatile("s_waitcnt vmcnt(0) lgkmcnt(0)" ::: "memory");
    __builtin_amdgcn_s_barrier(); asm volatile("" ::: "memory");
    v8i wf[5], xa, xb, xc;
    xc = ldfrag(lds, xoff, XS); SCHED;
#pragma unroll
    for (int n = 0; n < 5; ++n) { wf[n] = ldfrag(lds, n < 4 ? woff + n * 256 : woff4, WS); SCHED; }
#define MF(m, n, xf) acc[m][n] = __builtin_amdgcn_mfma_scale_f32_16x16x128_f8f6f4(xf, wf[n], acc[m][n], 0, 1, 0, 0, 0, 0)
#define MM(m, xf) do { _Pragma("unroll") for (int n = 0; n < 5; ++n) MF(m, n, xf); } while (0)
#define LF(m, n, xf) asm volatile("v_mfma_f32_16x16x128_f8f6f4 %0, %1, %2, %0 blgp:1" : "+v"(acc[m][n]) : "v"(xf), "v"(wf[n]))
#define LM(m, xf) do { _Pragma("unroll") for (int n = 0; n < 5; ++n) LF(m, n, xf); } while (0)
#define PRIO(p) __builtin_amdgcn_s_setprio(p)
    u32 seen = 0;
    int u = piece_of(1);
    for (int t = 0; t < NT - 1; ++t) {
        const int b = t & 1;
        LAS const char* cur = lds + b * STG; LAS const char* nxt = lds + (b ^ 1) * STG;
        if (wid == 0) GSTAMP(t);
        if ((__ballot(seen == call) & 31ull) != 31ull) { wait_piece(u); __builtin_amdgcn_s_waitcnt(0xC07F); }
        if (wid == 0) GSTAMP(32 + t);
        xb = ldfrag(cur, xoff + 1 * 256, XS); SCHED;
        LF(0, 0, xc); SCHED; DMA_W(u, b ^ 1, 0); SCHED; LF(0, 1, xc); SCHED; DMA_W(u, b ^ 1, 1); SCHED; LF(0, 2, xc); SCHED; DMA_W(u, b ^ 1, 2); SCHED;
        LF(0, 3, xc); SCHED; DMA_W(u, b ^ 1, 3); SCHED; LF(0, 4, xc); SCHED; DMA_W(u, b ^ 1, 4); SCHED;
        PRIO(2);
        xa = ldfrag(cur, xoff + 2 * 256, XS); SCHED;
        LF(1, 0, xb); SCHED; DMA_X(u, b ^ 1, 0); SCHED; LF(1, 1, xb); SCHED; DMA_X(u, b ^ 1, 1); SCHED;
        LF(1, 2, xb); SCHED; DMA_X(u, b ^ 1, 2); SCHED; LF(1, 3, xb); SCHED; DMA_X(u, b ^ 1, 3); SCHED; LF(1, 4, xb); SCHED;
        xb = ldfrag(cur, xoff + 3 * 256, XS);
        {
            u = piece_of((t + 2) & 31);
            __builtin_amdgcn_raw_ptr_buffer_load_lds(rf, (LAS void*)snap, 4, fv, ((u >> 4) << (DONE_SHIFT + 6)) + (u & 7) * 4, 0, 17);
        }
        SCHED; LM(2, xa); SCHED;
        PRIO(1);
        xa = ldfrag(cur, xoff + 4 * 256, XS); SCHED; LM(3, xb); SCHED;
        xc = ldfrag(cur, xoff + 5 * 256, XS); SCHED; LM(4, xa); SCHED;
        PRIO(0);
        xa = ldfrag(cur, xoff + 6 * 256, XS); xb = ldfrag(cur, xoff + 7 * 256, XS); SCHED; LM(5, xc); SCHED;
        asm volatile("s_waitcnt vmcnt(0) lgkmcnt(0)" ::: "memory");
        __builtin_amdgcn_s_barrier(); asm volatile("" ::: "memory"); SCHED;
        PRIO(3);
#pragma unroll
        for (int n = 0; n < 5; ++n) {
            LF(6, n, xa); SCHED;
            if (n == 0) { seen = *(LAS const u32*)(snap + lane); xc = ldfrag(nxt, xoff, XS); }
            SCHED; LF(7, n, xb); SCHED; wf[n] = ldfrag(nxt, n < 4 ? woff + n * 256 : woff4, WS); SCHED;
        }
    }
    STAMP(3);
    const float inv = *(LAS const volatile u32*)bad == call ? __builtin_nanf("") : __int_as_float(rscale);
    const float nsl = scale * -1.44269504089f;
    const int v4 = (tj == 12 && wc == 0) ? 0x40000000 : (fq * 4 * DIM + wc * 64 + fr * 4) * 4, v1 = (fq * 4 * DIM + 256 + wc * 16 + fr) * 4;
    auto silu2 = [&](float a0, float a1, float& r0, float& r1) {
        const v2f u = (v2f){a0, a1} * nsl;
        v2f e = {__builtin_amdgcn_exp2f(__builtin_amdgcn_fmed3f(u[0], -86.5617024533f, 86.5617024533f)),
                 __builtin_amdgcn_exp2f(__builtin_amdgcn_fmed3f(u[1], -86.5617024533f, 86.5617024533f))};
        e = __builtin_elementwise_fma(e, (v2f){inv, inv}, (v2f){inv, inv});
        r0 = __builtin_amdgcn_rcpf(e[0]) * a0; r1 = __builtin_amdgcn_rcpf(e[1]) * a1;
        asm("" : "+v"(r0)); asm("" : "+v"(r1));
    };
#define EPI(m) do { _Pragma("unroll") for (int j = 0; j < 4; j += 2) { \
        float r0[5], r1[5]; \
        _Pragma("unroll") for (int n = 0; n < 5; ++n) silu2(acc[m][n][j], acc[m][n][j + 1], r0[n], r1[n]); \
        asm volatile("buffer_store_dwordx4 %0, %4, %6, %7 offen " OUT_FLAGS "\n\tbuffer_store_dwordx4 %1, %4, %6, %8 offen " OUT_FLAGS "\n\t" \
                     "buffer_store_dword %2, %5, %6, %7 offen " OUT_FLAGS "\n\tbuffer_store_dword %3, %5, %6, %8 offen " OUT_FLAGS \
                     :: "v"((v4f){r0[0], r0[1], r0[2], r0[3]}), "v"((v4f){r1[0], r1[1], r1[2], r1[3]}), "v"(r0[4]), "v"(r1[4]), "v"(v4), "v"(v1), \
                        "s"(ro), "s"(((m) * 16 + j) * DIM * 4), "s"(((m) * 16 + j + 1) * DIM * 4) : "memory"); \
    } } while (0)
    {
        LAS const char* cur = lds + ((NT - 1) & 1) * STG;
        xb = ldfrag(cur, xoff + 1 * 256, XS); SCHED; MM(0, xc); SCHED;
        xa = ldfrag(cur, xoff + 2 * 256, XS); SCHED; MM(1, xb); SCHED; EPI(0); SCHED; PRIO(2);
        xb = ldfrag(cur, xoff + 3 * 256, XS); SCHED; MM(2, xa); SCHED; EPI(1); SCHED;
        xa = ldfrag(cur, xoff + 4 * 256, XS); SCHED; MM(3, xb); SCHED; EPI(2); SCHED; PRIO(1);
        xb = ldfrag(cur, xoff + 5 * 256, XS); SCHED; MM(4, xa); SCHED; EPI(3); SCHED;
        xa = ldfrag(cur, xoff + 6 * 256, XS); SCHED; MM(5, xb); SCHED; EPI(4); SCHED; PRIO(0);
        xb = ldfrag(cur, xoff + 7 * 256, XS); SCHED; MM(6, xa); SCHED; EPI(5); SCHED;
        MM(7, xb); SCHED; EPI(6); SCHED; EPI(7);
    }
    STAMP(4);
}
}

__global__ void __launch_bounds__(512, 2) fused_allgather_gemm(Args a) {
    extern __shared__ __attribute__((aligned(16))) char lds_dyn[];
    LAS char* lds = (LAS char*)lds_dyn;
    asm volatile("" :: "s"(a.x), "s"(a.w), "s"(a.me), "s"(a.rank));
    STAMP(0);
    const int xcd = blockIdx.x & 7, slot = blockIdx.x >> 3;
    u32* cnt = (u32*)(a.me + CNT_OFF) + blockIdx.x;
    if (__builtin_expect(slot < GEMM_SLOTS, 1)) {
        const u32 k0 = first_unit(blockIdx.x, (u32)__builtin_amdgcn_readfirstlane(threadIdx.x >> 6));
        u32x4 r0, r1;
        if (blockIdx.x >= LATE_TASKS) __builtin_amdgcn_s_sleep(LATE_SLEEP);
        load_unit(a, k0, threadIdx.x & 63, r0, r1);
        const u32 prev = __hip_atomic_load(cnt, __ATOMIC_RELAXED, __HIP_MEMORY_SCOPE_AGENT);
        SCHED;
        const u32 call = stage_out(a, lds, prev, r0, r1);
        if (__builtin_amdgcn_readfirstlane(threadIdx.x) == 0) st32_agent_lane0(cnt, call);
        STAMP(1);
        gemm(a, call, xcd, slot, lds);
    } else {
        pull(a, cnt, (slot - GEMM_SLOTS) * 8 + xcd, lds);
    }
}

extern "C" void kernel_launch(int rank, int nranks, ncclComm_t comm, hipStream_t stream,
                              void* const* d_in, const int* in_sizes, int n_in,
                              void* d_out, int out_size, const int* in_dtypes, int out_dtype,
                              void* const* d_out_all, void* const* d_ws_all, size_t ws_bytes) {
    if (nranks != NR || (unsigned)rank >= (unsigned)NR || n_in != 4 || ws_bytes < WS_NEED || out_size != DIM * DIM ||
        in_sizes[0] != DIM * KS || in_sizes[1] != DIM * KS) {
        fprintf(stderr, "kernel_launch: unexpected problem shape\n");
        return;
    }
    static bool attr_set[NR] = {};
    if (!attr_set[rank]) {
        HIP_CHECK(hipFuncSetAttribute((const void*)fused_allgather_gemm, hipFuncAttributeMaxDynamicSharedMemorySize, LDS_BYTES));
        attr_set[rank] = true;
    }
    Args a;
    a.x = (const unsigned char*)d_in[0]; a.w = (const unsigned char*)d_in[1];
    a.sx = (const float*)d_in[2]; a.sw = (const float*)d_in[3];
    a.out = (float*)d_out;
    for (int p = 0; p < NR; ++p) a.ws[p] = (char*)d_ws_all[p];
    a.me = a.ws[rank]; a.rank = rank; a.pad = 0;
    fused_allgather_gemm<<<256, 512, LDS_BYTES, stream>>>(a);
    HIP_CHECK(hipGetLastError());
}
```
